```python
import numpy as np

kernel_name = "gpu_ag_gemm_m4096_k4096_n4096_e4m3e5m2_dqsilu"


ABSMAX_RTOL = 0.05

COMPUTE_DTYPE = "float32"
EPILOGUE = "dequant_silu"
IN_QUANT = None
OUT_QUANT = None
IN_FP8 = "e4m3e5m2"
IN_DTYPES = {"x": "e4m3", "w_mat": "e5m2",
             "scale_x": "float32", "scale_w": "float32"}
WORLD_SIZE = 8

def _snap_fp8(x, mant_bits, max_abs, min_normal):
    a = np.abs(x.astype(np.float64))
    _m, _e = np.frexp(a)
    q = np.where(a >= min_normal, np.ldexp(1.0, _e - 1 - mant_bits),
                 min_normal * 2.0 ** -mant_bits)
    snapped = np.minimum(np.round(a / q) * q, max_abs)
    return (np.sign(x) * snapped).astype(np.float32)

def setup_inputs():

    x = _snap_fp8(np.random.randn(4096, 4096).astype(np.float32), 3, 448.0, 2.0 ** -6)
    w_mat = _snap_fp8(np.random.randn(4096, 4096).astype(np.float32), 2, 57344.0, 2.0 ** -14)
    scale_x = (0.02 * (1.0 + np.random.rand(1))).astype(np.float32)
    scale_w = (0.02 * (1.0 + np.random.rand(1))).astype(np.float32)
    return {"x": x, "w_mat": w_mat, "scale_x": scale_x, "scale_w": scale_w}

def reference(x, w_mat, scale_x, scale_w):
    acc = (x @ w_mat).astype(np.float32)
    y = (acc * (scale_x * scale_w)).astype(np.float32)
    return (y / (1.0 + np.exp(-np.clip(y, -60.0, 60.0)))).astype(np.float32)
```

```cpp
#include <hip/hip_runtime.h>
#include <rccl/rccl.h>
#include <cstdio>
#include <cstdint>

#define HIP_CHECK(expr) do { hipError_t _e = (expr); if (_e != hipSuccess) { \
    fprintf(stderr, "HIP error %s at %s:%d\n", hipGetErrorString(_e), __FILE__, __LINE__); \
    return; } } while (0)

namespace {

typedef unsigned u32x4 __attribute__((ext_vector_type(4)));
typedef int i32x8 __attribute__((ext_vector_type(8)));
typedef float f32x4 __attribute__((ext_vector_type(4)));

constexpr int NR = 8;
constexpr int KD = 4096;
constexpr int MLOC = 512;
constexpr int NLOC = 512;
constexpr unsigned SHARD = MLOC * KD;
constexpr unsigned GROUP = 16 * KD;
constexpr unsigned SLOT = NR * SHARD;
constexpr int NSLOT = 3;
constexpr size_t CTRL_OFF = (size_t)NSLOT * SLOT;
constexpr size_t CTRL_BYTES = 64 * 1024;
constexpr int NBLK = 256;
constexpr int NWAVE = 9;
constexpr int NTHR = NWAVE * 64;
constexpr unsigned LDS_W = 32 * KD;
constexpr unsigned LDS_BYTES = LDS_W + 16;
constexpr unsigned long long DEADLINE = 50000000ull;
constexpr int AUX_SYS = 17;
#define PWIN 4

struct Params {
  const uint8_t* x;
  const uint8_t* w;
  const float* sx;
  const float* sw;
  float* out;
  char* ws[NR];
  int rank;
};

__device__ __forceinline__ __amdgpu_buffer_rsrc_t mk_rsrc(const void* p, unsigned bytes) {
  return __builtin_amdgcn_make_buffer_rsrc(const_cast<void*>(p), 0, bytes, 0x00020000);
}
__device__ __forceinline__ unsigned min3u(unsigned a, unsigned b, unsigned c) {
  return min(min(a, b), c);
}

__global__ void __launch_bounds__(NTHR) ag_gemm_kernel(Params p) {
  extern __shared__ __attribute__((aligned(16))) char smem[];
  const int wid = __builtin_amdgcn_readfirstlane(threadIdx.x >> 6);
  const int lane = threadIdx.x & 63;
  const int bid = blockIdx.x;
  const int grp = bid & 7;
  const int cu = bid >> 3;
  char* myws = p.ws[p.rank];
  unsigned* ctrl = (unsigned*)(myws + CTRL_OFF);
  unsigned* cnt = ctrl + 64 + bid * 16 + wid;
  unsigned* tmo = ctrl;

  if (wid == 0) {
    if (grp != 0) {
      __builtin_amdgcn_s_setprio(3);
      const unsigned craw = __hip_atomic_load(cnt, __ATOMIC_RELAXED, __HIP_MEMORY_SCOPE_AGENT);
      const __amdgpu_buffer_rsrc_t rx = mk_rsrc(p.x, SHARD);
      const unsigned vsrc = (unsigned)(lane & 15) * KD + (unsigned)(lane >> 4) * 16 + (unsigned)(cu >> 3) * GROUP + (unsigned)(cu & 7) * 64;
      const unsigned vdst = (unsigned)lane * 16 + (unsigned)(cu >> 3) * GROUP + (unsigned)(cu & 7) * 1024;
      auto gsel = [](int it) -> unsigned { return (unsigned)((it >> 4) * 8 + 4 * (it & 1)) * GROUP; };
      auto csel = [](int it) -> unsigned { return (unsigned)((it >> 1) & 7) * 8; };
      u32x4 r[PWIN];
#pragma unroll
      for (int i = 0; i < PWIN; ++i) r[i] = __builtin_amdgcn_raw_buffer_load_b128(rx, vsrc, gsel(i) + csel(i) * 64, 0);
      __builtin_amdgcn_s_barrier();
      const unsigned call = __builtin_amdgcn_readfirstlane(craw) + 1;
      const unsigned dst = (unsigned)(p.rank + grp) & 7u;
      const __amdgpu_buffer_rsrc_t rd = mk_rsrc(p.ws[dst] + (size_t)(call % NSLOT) * SLOT + (size_t)p.rank * SHARD, SHARD);
      for (int it0 = 0; it0 < 64; it0 += PWIN) {
#pragma unroll
        for (int i = 0; i < PWIN; ++i) {
          const int it = it0 + i;
          u32x4 v = r[i];
#pragma unroll
          for (int d = 0; d < 4; ++d) v[d] = v[d] ? v[d] : 0x80808080u;
          __builtin_amdgcn_raw_buffer_store_b128(v, rd, vdst, gsel(it) + csel(it) * 1024, AUX_SYS);
          const int nx = (it + PWIN) & 63;
          r[i] = __builtin_amdgcn_raw_buffer_load_b128(rx, vsrc, gsel(nx) + csel(nx) * 64, 0);
        }
      }
      if (lane == 0) __hip_atomic_store(cnt, call, __ATOMIC_RELAXED, __HIP_MEMORY_SCOPE_AGENT);
    } else {
      __builtin_amdgcn_s_barrier();
    }
    return;
  }

  const int cw = wid - 1;
  volatile unsigned* lds_cnt = (volatile unsigned*)(smem + LDS_W);
  const unsigned craw = __hip_atomic_load(cnt, __ATOMIC_RELAXED, __HIP_MEMORY_SCOPE_AGENT);
  if (cw == 0 && lane == 0) *lds_cnt = 0;
  asm volatile("s_waitcnt lgkmcnt(0)" ::: "memory");
  __builtin_amdgcn_s_barrier();
  const int slab = cu & 15, half = cu >> 4;
  {
    const int kb = cw * 32 + (lane & 31), nh = lane >> 5;
    const __amdgpu_buffer_rsrc_t rw = mk_rsrc(p.w, KD * NLOC);
    const unsigned vo = (unsigned)kb * 16 * NLOC + (unsigned)slab * 32 + (unsigned)nh * 16;
    u32x4 r[16];
#pragma unroll
    for (int j = 0; j < 16; ++j) r[j] = __builtin_amdgcn_raw_buffer_load_b128(rw, vo, j * NLOC, 0);
    const int u = kb & 7;
    char* base = smem + (kb >> 3) * 4096 + nh * 2048 + (u >> 2) * 1024 + (u & 3) * 256;
#pragma unroll
    for (int d = 0; d < 4; ++d) {
      u32x4 o[4];
#pragma unroll
      for (int e = 0; e < 4; ++e) {
        const unsigned a0 = r[4 * e][d], a1 = r[4 * e + 1][d], a2 = r[4 * e + 2][d], a3 = r[4 * e + 3][d];
        const unsigned t0 = __builtin_amdgcn_perm(a1, a0, 0x05010400u), t1 = __builtin_amdgcn_perm(a1, a0, 0x07030602u);
        const unsigned t2 = __builtin_amdgcn_perm(a3, a2, 0x05010400u), t3 = __builtin_amdgcn_perm(a3, a2, 0x07030602u);
        o[0][e] = __builtin_amdgcn_perm(t2, t0, 0x05040100u);
        o[1][e] = __builtin_amdgcn_perm(t2, t0, 0x07060302u);
        o[2][e] = __builtin_amdgcn_perm(t3, t1, 0x05040100u);
        o[3][e] = __builtin_amdgcn_perm(t3, t1, 0x07060302u);
      }
#pragma unroll
      for (int e = 0; e < 4; ++e) *(u32x4*)(base + (((4 * d + e) ^ u) * 16)) = o[e];
    }
    asm volatile("s_waitcnt lgkmcnt(0)" ::: "memory");
    if (lane == 0) __hip_atomic_fetch_add((unsigned*)(smem + LDS_W), 1u, __ATOMIC_RELAXED, __HIP_MEMORY_SCOPE_WORKGROUP);
  }
  const unsigned call = __builtin_amdgcn_readfirstlane(craw) + 1;
  const float scale = p.sx[0] * p.sw[0];
  const bool remote = grp != 0;
  const unsigned src = (unsigned)(p.rank + NR - grp) & 7u;
  const char* xbase = remote ? myws + (size_t)(call % NSLOT) * SLOT + (size_t)src * SHARD : (const char*)p.x;
  const unsigned vx = remote ? (unsigned)lane * 16 : (unsigned)(lane & 15) * KD + (unsigned)(lane >> 4) * 16;
  const unsigned ps = remote ? 1024u : 64u;
  const unsigned vh = (unsigned)(lane & 7) * 1024 + 1020;
  const int li = lane & 15, lg = lane >> 4;
  const char* wl0 = smem + lg * 256 + ((li ^ lg) * 16);
  const char* wl1 = smem + 1024 + lg * 256 + ((li ^ (lg | 4)) * 16);
  while (*lds_cnt < 8u) __builtin_amdgcn_s_sleep(1);
  asm volatile("" ::: "memory");

  bool dead = false;
  for (int rb = cw >> 2; rb < 4; rb += 2) {
    const int G = rb * 8 + half * 4 + (cw & 3);
    const __amdgpu_buffer_rsrc_t rx = mk_rsrc(xbase + (size_t)G * GROUP, GROUP);
    f32x4 acc0 = {0.f, 0.f, 0.f, 0.f}, acc1 = {0.f, 0.f, 0.f, 0.f};
    bool hint_ok = !remote;
    for (int c = 0; c < 8; ++c) {
      const unsigned long long t0 = __builtin_amdgcn_s_memrealtime();
      if (!hint_ok && !dead && !(rb == 3 && c == 7)) {
        for (;;) {
          const unsigned h = __builtin_amdgcn_raw_buffer_load_b32(rx, vh, c * 8192, AUX_SYS);
          if (__builtin_amdgcn_ballot_w64(h == 0u) == 0ull) break;
          if (__builtin_amdgcn_readfirstlane(__hip_atomic_load(tmo, __ATOMIC_RELAXED, __HIP_MEMORY_SCOPE_AGENT)) == call ||
              __builtin_amdgcn_s_memrealtime() - t0 > DEADLINE) { dead = true; break; }
          __builtin_amdgcn_s_sleep(4);
          asm volatile("" ::: "memory");
        }
      }
      u32x4 xv[8];
      unsigned hn;
      for (;;) {
#pragma unroll
        for (int q = 0; q < 8; ++q) xv[q] = __builtin_amdgcn_raw_buffer_load_b128(rx, vx, (unsigned)(c * 8 + q) * ps, AUX_SYS);
        hn = __builtin_amdgcn_raw_buffer_load_b32(rx, vh, ((c + 1) & 7) * 8192, AUX_SYS);
        if (!remote || dead) break;
        unsigned m = 0xffffffffu;
#pragma unroll
        for (int q = 0; q < 8; ++q) m = min3u(min3u(m, xv[q][0], xv[q][1]), xv[q][2], xv[q][3]);
        if (__builtin_amdgcn_ballot_w64(m == 0u) == 0ull) break;
        if (__builtin_amdgcn_readfirstlane(__hip_atomic_load(tmo, __ATOMIC_RELAXED, __HIP_MEMORY_SCOPE_AGENT)) == call ||
            __builtin_amdgcn_s_memrealtime() - t0 > DEADLINE) { dead = true; break; }
        asm volatile("" ::: "memory");
      }
      hint_ok = !remote || __builtin_amdgcn_ballot_w64(hn == 0u) == 0ull;
#pragma unroll
      for (int ks = 0; ks < 4; ++ks) {
        const int off = (c * 4 + ks) * 4096;
        const u32x4 a00 = *(const u32x4*)(wl0 + off), a01 = *(const u32x4*)(wl1 + off);
        const u32x4 a10 = *(const u32x4*)(wl0 + off + 2048), a11 = *(const u32x4*)(wl1 + off + 2048);
        const i32x8 b = {(int)xv[2 * ks][0], (int)xv[2 * ks][1], (int)xv[2 * ks][2], (int)xv[2 * ks][3],
                         (int)xv[2 * ks + 1][0], (int)xv[2 * ks + 1][1], (int)xv[2 * ks + 1][2], (int)xv[2 * ks + 1][3]};
        const i32x8 a0 = {(int)a00[0], (int)a00[1], (int)a00[2], (int)a00[3], (int)a01[0], (int)a01[1], (int)a01[2], (int)a01[3]};
        const i32x8 a1 = {(int)a10[0], (int)a10[1], (int)a10[2], (int)a10[3], (int)a11[0], (int)a11[1], (int)a11[2], (int)a11[3]};
        acc0 = __builtin_amdgcn_mfma_scale_f32_16x16x128_f8f6f4(a0, b, acc0,   1,   0, 0, 0, 0, 0);
        acc1 = __builtin_amdgcn_mfma_scale_f32_16x16x128_f8f6f4(a1, b, acc1, 1, 0, 0, 0, 0, 0);
      }
    }
    if (dead && lane == 0) __hip_atomic_store(tmo, call, __ATOMIC_RELAXED, __HIP_MEMORY_SCOPE_AGENT);
    f32x4 o0, o1;
#pragma unroll
    for (int r = 0; r < 4; ++r) {
      const float y0 = acc0[r] * scale, y1 = acc1[r] * scale;
      const float c0 = __builtin_amdgcn_fmed3f(y0, -60.f, 60.f), c1 = __builtin_amdgcn_fmed3f(y1, -60.f, 60.f);
      o0[r] = y0 * __builtin_amdgcn_rcpf(1.f + __expf(-c0));
      o1[r] = y1 * __builtin_amdgcn_rcpf(1.f + __expf(-c1));
      if (dead) { o0[r] = __builtin_nanf(""); o1[r] = __builtin_nanf(""); }
    }
    const __amdgpu_buffer_rsrc_t ro = mk_rsrc(p.out + ((size_t)src * MLOC + (size_t)G * 16) * NLOC, 16 * NLOC * 4);
    const unsigned vo = (unsigned)li * (NLOC * 4) + (unsigned)slab * 128 + (unsigned)lg * 16;
    __builtin_amdgcn_raw_buffer_store_b128(__builtin_bit_cast(u32x4, o0), ro, vo, 0, AUX_SYS);
    __builtin_amdgcn_raw_buffer_store_b128(__builtin_bit_cast(u32x4, o1), ro, vo, 64, AUX_SYS);
  }
  if (!remote) {
    const __amdgpu_buffer_rsrc_t rz = mk_rsrc(myws + (size_t)((call + 2) % NSLOT) * SLOT, SLOT);
    const u32x4 z = {0u, 0u, 0u, 0u};
    const unsigned vz = (unsigned)(cu * 8 + cw) * 8192 + (unsigned)lane * 16;
    for (int s = 1; s < NR; ++s) {
      const unsigned so = ((unsigned)(p.rank + s) & 7u) * SHARD;
#pragma unroll
      for (int u = 0; u < 8; ++u) __builtin_amdgcn_raw_buffer_store_b128(z, rz, vz, so + u * 1024, AUX_SYS);
    }
  }
  if (lane == 0) __hip_atomic_store(cnt, call, __ATOMIC_RELAXED, __HIP_MEMORY_SCOPE_AGENT);
}

}

extern "C" void kernel_launch(int rank, int nranks, ncclComm_t comm,
                              hipStream_t stream,
                              void* const* d_in, const int* in_sizes, int n_in,
                              void* d_out, int out_size,
                              const int* in_dtypes, int out_dtype,
                              void* const* d_out_all,
                              void* const* d_ws_all, size_t ws_bytes) {
  (void)comm; (void)in_dtypes; (void)out_dtype; (void)d_out_all;
  if (nranks != NR || (unsigned)rank >= (unsigned)NR || n_in < 4 || in_sizes[0] != MLOC * KD || in_sizes[1] != KD * NLOC ||
      out_size != NR * MLOC * NLOC || ws_bytes < CTRL_OFF + CTRL_BYTES) {
    fprintf(stderr, "kernel_launch: unexpected problem shape\n");
    return;
  }
  static bool attr_done[NR];
  if (!attr_done[rank]) {
    HIP_CHECK(hipFuncSetAttribute((const void*)ag_gemm_kernel, hipFuncAttributeMaxDynamicSharedMemorySize, LDS_BYTES));
    attr_done[rank] = true;
  }
  Params p;
  p.x = (const uint8_t*)d_in[0];
  p.w = (const uint8_t*)d_in[1];
  p.sx = (const float*)d_in[2];
  p.sw = (const float*)d_in[3];
  p.out = (float*)d_out;
  for (int i = 0; i < NR; ++i) p.ws[i] = (char*)d_ws_all[i];
  p.rank = rank;
  hipLaunchKernelGGL(ag_gemm_kernel, dim3(NBLK), dim3(NTHR), LDS_BYTES, stream, p);
  HIP_CHECK(hipGetLastError());
}
```

```cpp
#include <hip/hip_runtime.h>
#include <rccl/rccl.h>
#include <cstdio>
#include <cstdint>

#define HIP_CHECK(expr) do { hipError_t _e = (expr); if (_e != hipSuccess) { \
    fprintf(stderr, "HIP error %s at %s:%d\n", hipGetErrorString(_e), __FILE__, __LINE__); \
    return; } } while (0)

namespace {

typedef unsigned u32x4 __attribute__((ext_vector_type(4)));
typedef int i32x8 __attribute__((ext_vector_type(8)));
typedef float f32x4 __attribute__((ext_vector_type(4)));

constexpr int NR = 8;
constexpr int KD = 4096;
constexpr int MLOC = 512;
constexpr int NLOC = 512;
constexpr unsigned SHARD = MLOC * KD;
constexpr unsigned GROUP = 16 * KD;
constexpr unsigned SLOT = NR * SHARD;
constexpr int NSLOT = 3;
constexpr size_t CTRL_OFF = (size_t)NSLOT * SLOT;
constexpr size_t CTRL_BYTES = 64 * 1024;
constexpr int NBLK = 256;
constexpr int NWAVE = 8;
constexpr int NTHR = NWAVE * 64;
constexpr int NPROD = 32;
constexpr unsigned LDS_BYTES = 32 * KD;
constexpr unsigned long long DEADLINE = 50000000ull;
constexpr int AUX_SYS = 17;
#define PWIN 2

#define STAMP(i) do {} while (0)
#define STAMP_FLUSH() do {} while (0)

struct Params {
  const uint8_t* x;
  const uint8_t* w;
  const float* sx;
  const float* sw;
  float* out;
  char* ws[NR];
  int rank;
};

__device__ __forceinline__ __amdgpu_buffer_rsrc_t mk_rsrc(const void* p, unsigned bytes) {
  return __builtin_amdgcn_make_buffer_rsrc(const_cast<void*>(p), 0, bytes, 0x00020000);
}
__device__ __forceinline__ unsigned min3u(unsigned a, unsigned b, unsigned c) {
  return min(min(a, b), c);
}

__global__ void __launch_bounds__(NTHR) ag_gemm_kernel(Params p) {
  extern __shared__ __attribute__((aligned(16))) char smem[];
  const int wid = __builtin_amdgcn_readfirstlane(threadIdx.x >> 6);
  const int lane = threadIdx.x & 63;
  const int bid = blockIdx.x;
  char* myws = p.ws[p.rank];
  unsigned* ctrl = (unsigned*)(myws + CTRL_OFF);
  unsigned* cnt = ctrl + 64 + bid * NWAVE + wid;
  unsigned* tmo = ctrl;
  STAMP(0);

  if (bid < NPROD) {
    if (wid == NR - 1) return;
    const int cu = bid;
    const unsigned craw = __hip_atomic_load(cnt, __ATOMIC_RELAXED, __HIP_MEMORY_SCOPE_AGENT);
    const __amdgpu_buffer_rsrc_t rx = mk_rsrc(p.x, SHARD);
    const unsigned vsrc = (unsigned)(lane & 15) * KD + (unsigned)(lane >> 4) * 16 + (unsigned)(cu >> 3) * GROUP + (unsigned)(cu & 7) * 64;
    const unsigned vdst = (unsigned)lane * 16 + (unsigned)(cu >> 3) * GROUP + (unsigned)(cu & 7) * 1024;
    auto gsel = [](int it) -> unsigned { return (unsigned)((it >> 4) * 8 + 4 * (it & 1)) * GROUP; };
    auto csel = [](int it) -> unsigned { return (unsigned)((it >> 1) & 7) * 8; };
    u32x4 r[PWIN];
#pragma unroll
    for (int i = 0; i < PWIN; ++i) r[i] = __builtin_amdgcn_raw_buffer_load_b128(rx, vsrc, gsel(i) + csel(i) * 64, 0);
    const unsigned call = __builtin_amdgcn_readfirstlane(craw) + 1;
    const unsigned dst = (unsigned)(p.rank + 1 + wid) & 7u;
    const __amdgpu_buffer_rsrc_t rd = mk_rsrc(p.ws[dst] + (size_t)(call % NSLOT) * SLOT + (size_t)p.rank * SHARD, SHARD);
    STAMP(1);
#pragma unroll 1
    for (int it0 = 0; it0 < 64; it0 += PWIN) {
      if (it0 == 16) STAMP(2);
      if (it0 == 32) STAMP(3);
      if (it0 == 48) STAMP(4);
#pragma unroll
      for (int i = 0; i < PWIN; ++i) {
        const int it = it0 + i;
        u32x4 v = r[i];
#pragma unroll
        for (int d = 0; d < 4; ++d) v[d] = v[d] ? v[d] : 0x80808080u;
        __builtin_amdgcn_raw_buffer_store_b128(v, rd, vdst, gsel(it) + csel(it) * 1024, AUX_SYS);
        __builtin_amdgcn_sched_barrier(0);
        const int nx = (it + PWIN) & 63;
        r[i] = __builtin_amdgcn_raw_buffer_load_b128(rx, vsrc, gsel(nx) + csel(nx) * 64, 0);
        __builtin_amdgcn_sched_barrier(0);
      }
    }
    STAMP(5);
    STAMP(6);
    if (lane == 0) __hip_atomic_store(cnt, call, __ATOMIC_RELAXED, __HIP_MEMORY_SCOPE_AGENT);
    STAMP_FLUSH();
    return;
  }

  const int ci = (bid & 7) * 28 + (bid >> 3) - NPROD / 8;
  const int strm = ci >> 4, slab = ci & 15;
  const int half = strm & 1;
  const int cw = wid;
  const unsigned craw = __hip_atomic_load(cnt, __ATOMIC_RELAXED, __HIP_MEMORY_SCOPE_AGENT);
  {
    const int kb = cw * 32 + (lane >> 1), nh = lane & 1;
    const __amdgpu_buffer_rsrc_t rw = mk_rsrc(p.w, KD * NLOC);
    const unsigned vo = (unsigned)kb * 16 * NLOC + (unsigned)slab * 32 + (unsigned)nh * 16;
    u32x4 r[16];
#pragma unroll
    for (int j = 0; j < 16; ++j) r[j] = __builtin_amdgcn_raw_buffer_load_b128(rw, vo, j * NLOC, 0);
    const int u = kb & 7;
    const int sw = u ^ (nh << 2);
    char* base = smem + (kb >> 3) * 4096 + nh * 2048 + (u >> 2) * 1024 + (u & 3) * 256;
#pragma unroll
    for (int d = 0; d < 4; ++d) {
      u32x4 o[4];
#pragma unroll
      for (int e = 0; e < 4; ++e) {
        const unsigned a0 = r[4 * e][d], a1 = r[4 * e + 1][d], a2 = r[4 * e + 2][d], a3 = r[4 * e + 3][d];
        const unsigned t0 = __builtin_amdgcn_perm(a1, a0, 0x05010400u), t1 = __builtin_amdgcn_perm(a1, a0, 0x07030602u);
        const unsigned t2 = __builtin_amdgcn_perm(a3, a2, 0x05010400u), t3 = __builtin_amdgcn_perm(a3, a2, 0x07030602u);
        o[0][e] = __builtin_amdgcn_perm(t2, t0, 0x05040100u);
        o[1][e] = __builtin_amdgcn_perm(t2, t0, 0x07060302u);
        o[2][e] = __builtin_amdgcn_perm(t3, t1, 0x05040100u);
        o[3][e] = __builtin_amdgcn_perm(t3, t1, 0x07060302u);
      }
#pragma unroll
      for (int e = 0; e < 4; ++e) *(u32x4*)(base + (((4 * d + e) ^ sw) * 16)) = o[e];
    }
  }
  const unsigned call = __builtin_amdgcn_readfirstlane(craw) + 1;
  {
    const __amdgpu_buffer_rsrc_t rz = mk_rsrc(myws + (size_t)((call + 2) % NSLOT) * SLOT, SLOT);
    const u32x4 z = {0u, 0u, 0u, 0u};
    const int zi = ci * NWAVE + cw;
    const unsigned so = ((unsigned)(p.rank + 1 + (zi >> 8)) & 7u) * SHARD + (unsigned)(zi & 255) * 8192;
#pragma unroll
    for (int u = 0; u < 8; ++u) __builtin_amdgcn_raw_buffer_store_b128(z, rz, (unsigned)lane * 16, so + u * 1024, AUX_SYS);
  }
  const float scale = p.sx[0] * p.sw[0];
  const unsigned rsrc_rank = (unsigned)(p.rank + NR - 1 - (strm >> 1)) & 7u;
  const char* xrem = myws + (size_t)(call % NSLOT) * SLOT + (size_t)rsrc_rank * SHARD;
  const unsigned vh = (unsigned)(lane & 7) * 1024 + 1020;
  const int li = lane & 15, lg = lane >> 4;
  const char* wlA = smem + lg * 256 + ((li ^ lg) * 16);
  const char* wlB = smem + lg * 256 + ((li ^ (lg | 4)) * 16);
  asm volatile("s_waitcnt lgkmcnt(0)" ::: "memory");
  __builtin_amdgcn_s_barrier();
  asm volatile("" ::: "memory");
  STAMP(1);

  bool dead = false;
  const int lgrp = (cw - 4) * 14 + strm;
  for (int task = (cw >= 4 && lgrp < 32) ? -1 : 0; task < 2; ++task) {
    const bool remote = task >= 0;
    const int rb = (cw >> 2) + 2 * task;
    const int G = remote ? rb * 8 + half * 4 + (cw & 3) : lgrp;
    const unsigned src = remote ? rsrc_rank : (unsigned)p.rank;
    const __amdgpu_buffer_rsrc_t rx = mk_rsrc((remote ? xrem : (const char*)p.x) + (size_t)G * GROUP, GROUP);
    const unsigned vx = remote ? (unsigned)lane * 16 : (unsigned)(lane & 15) * KD + (unsigned)(lane >> 4) * 16;
    const unsigned ps = remote ? 1024u : 64u;
    f32x4 acc0 = {0.f, 0.f, 0.f, 0.f}, acc1 = {0.f, 0.f, 0.f, 0.f};
    bool hint_ok = !remote;
    for (int c = 0; c < 8; ++c) {
      const unsigned long long t0 = __builtin_amdgcn_s_memrealtime();
      if (!hint_ok && !dead && !(rb == 3 && c == 7)) {
        for (;;) {
          const unsigned h = __builtin_amdgcn_raw_buffer_load_b32(rx, vh, c * 8192, AUX_SYS);
          if (__builtin_amdgcn_ballot_w64(h == 0u) == 0ull) break;
          if (__builtin_amdgcn_readfirstlane(__hip_atomic_load(tmo, __ATOMIC_RELAXED, __HIP_MEMORY_SCOPE_AGENT)) == call ||
              __builtin_amdgcn_s_memrealtime() - t0 > DEADLINE) { dead = true; break; }
          __builtin_amdgcn_s_sleep(4);
          asm volatile("" ::: "memory");
        }
      }
      u32x4 xv[8];
      unsigned hn;
      for (;;) {
#pragma unroll
        for (int q = 0; q < 8; ++q) xv[q] = __builtin_amdgcn_raw_buffer_load_b128(rx, vx, (unsigned)(c * 8 + q) * ps, AUX_SYS);
        hn = __builtin_amdgcn_raw_buffer_load_b32(rx, vh, ((c + 1) & 7) * 8192, AUX_SYS);
        if (!remote || dead) break;
        unsigned m = 0xffffffffu;
#pragma unroll
        for (int q = 0; q < 8; ++q) m = min3u(min3u(m, xv[q][0], xv[q][1]), xv[q][2], xv[q][3]);
        if (__builtin_amdgcn_ballot_w64(m == 0u) == 0ull) break;
        if (__builtin_amdgcn_readfirstlane(__hip_atomic_load(tmo, __ATOMIC_RELAXED, __HIP_MEMORY_SCOPE_AGENT)) == call ||
            __builtin_amdgcn_s_memrealtime() - t0 > DEADLINE) { dead = true; break; }
        asm volatile("" ::: "memory");
      }
      hint_ok = !remote || __builtin_amdgcn_ballot_w64(hn == 0u) == 0ull;
#pragma unroll
      for (int ks = 0; ks < 4; ++ks) {
        const int off = (c * 4 + ks) * 4096;
        const u32x4 a00 = *(const u32x4*)(wlA + off), a01 = *(const u32x4*)(wlB + off + 1024);
        const u32x4 a10 = *(const u32x4*)(wlB + off + 2048), a11 = *(const u32x4*)(wlA + off + 3072);
        const i32x8 b = {(int)xv[2 * ks][0], (int)xv[2 * ks][1], (int)xv[2 * ks][2], (int)xv[2 * ks][3],
                         (int)xv[2 * ks + 1][0], (int)xv[2 * ks + 1][1], (int)xv[2 * ks + 1][2], (int)xv[2 * ks + 1][3]};
        const i32x8 a0 = {(int)a00[0], (int)a00[1], (int)a00[2], (int)a00[3], (int)a01[0], (int)a01[1], (int)a01[2], (int)a01[3]};
        const i32x8 a1 = {(int)a10[0], (int)a10[1], (int)a10[2], (int)a10[3], (int)a11[0], (int)a11[1], (int)a11[2], (int)a11[3]};
        acc0 = __builtin_amdgcn_mfma_scale_f32_16x16x128_f8f6f4(a0, b, acc0,   1,   0, 0, 0, 0, 0);
        acc1 = __builtin_amdgcn_mfma_scale_f32_16x16x128_f8f6f4(a1, b, acc1, 1, 0, 0, 0, 0, 0);
      }
    }
    STAMP(3 + task);
    if (dead && lane == 0) __hip_atomic_store(tmo, call, __ATOMIC_RELAXED, __HIP_MEMORY_SCOPE_AGENT);
    f32x4 o0, o1;
#pragma unroll
    for (int r = 0; r < 4; ++r) {
      const float y0 = acc0[r] * scale, y1 = acc1[r] * scale;
      const float c0 = __builtin_amdgcn_fmed3f(y0, -60.f, 60.f), c1 = __builtin_amdgcn_fmed3f(y1, -60.f, 60.f);
      o0[r] = y0 * __builtin_amdgcn_rcpf(1.f + __expf(-c0));
      o1[r] = y1 * __builtin_amdgcn_rcpf(1.f + __expf(-c1));
      if (dead) { o0[r] = __builtin_nanf(""); o1[r] = __builtin_nanf(""); }
    }
    const __amdgpu_buffer_rsrc_t ro = mk_rsrc(p.out + ((size_t)src * MLOC + (size_t)G * 16) * NLOC, 16 * NLOC * 4);
    const unsigned vo = (unsigned)li * (NLOC * 4) + (unsigned)slab * 128 + (unsigned)lg * 16;
    __builtin_amdgcn_raw_buffer_store_b128(__builtin_bit_cast(u32x4, o0), ro, vo, 0, AUX_SYS);
    __builtin_amdgcn_raw_buffer_store_b128(__builtin_bit_cast(u32x4, o1), ro, vo, 64, AUX_SYS);
  }
  STAMP(5);
  STAMP(6);
  if (lane == 0) __hip_atomic_store(cnt, call, __ATOMIC_RELAXED, __HIP_MEMORY_SCOPE_AGENT);
  STAMP_FLUSH();
}

}

extern "C" void kernel_launch(int rank, int nranks, ncclComm_t comm,
                              hipStream_t stream,
                              void* const* d_in, const int* in_sizes, int n_in,
                              void* d_out, int out_size,
                              const int* in_dtypes, int out_dtype,
                              void* const* d_out_all,
                              void* const* d_ws_all, size_t ws_bytes) {
  (void)comm; (void)in_dtypes; (void)out_dtype; (void)d_out_all;
  if (nranks != NR || (unsigned)rank >= (unsigned)NR || n_in < 4 || in_sizes[0] != MLOC * KD || in_sizes[1] != KD * NLOC ||
      out_size != NR * MLOC * NLOC || ws_bytes < CTRL_OFF + CTRL_BYTES) {
    fprintf(stderr, "kernel_launch: unexpected problem shape\n");
    return;
  }
  static bool attr_done[NR];
  if (!attr_done[rank]) {
    HIP_CHECK(hipFuncSetAttribute((const void*)ag_gemm_kernel, hipFuncAttributeMaxDynamicSharedMemorySize, LDS_BYTES));
    attr_done[rank] = true;
  }
  Params p;
  p.x = (const uint8_t*)d_in[0];
  p.w = (const uint8_t*)d_in[1];
  p.sx = (const float*)d_in[2];
  p.sw = (const float*)d_in[3];
  p.out = (float*)d_out;
  for (int i = 0; i < NR; ++i) p.ws[i] = (char*)d_ws_all[i];
  p.rank = rank;
  hipLaunchKernelGGL(ag_gemm_kernel, dim3(NBLK), dim3(NTHR), LDS_BYTES, stream, p);
  HIP_CHECK(hipGetLastError());
}
```

```cpp
#include <hip/hip_runtime.h>
#include <rccl/rccl.h>
#include <cstdio>
#include <cstdint>

#define HIP_CHECK(expr) do { hipError_t _e = (expr); if (_e != hipSuccess) { \
    fprintf(stderr, "HIP error %s at %s:%d\n", hipGetErrorString(_e), __FILE__, __LINE__); \
    return; } } while (0)

namespace {

typedef unsigned u32x4 __attribute__((ext_vector_type(4)));
typedef int i32x8 __attribute__((ext_vector_type(8)));
typedef float f32x4 __attribute__((ext_vector_type(4)));

constexpr int NR = 8;
constexpr int KD = 4096;
constexpr int MLOC = 512;
constexpr int NLOC = 512;
constexpr unsigned SHARD = MLOC * KD;
constexpr unsigned GROUP = 16 * KD;
constexpr unsigned SLOT = NR * SHARD;
constexpr int NSLOT = 3;
constexpr size_t CTRL_OFF = (size_t)NSLOT * SLOT;
constexpr size_t CTRL_BYTES = 64 * 1024;
constexpr int NBLK = 256;
constexpr int NWAVE = 8;
constexpr int NTHR = NWAVE * 64;
constexpr int NPROD = 32;
constexpr int NPIECE = SHARD / 1024;
#define NSTAT 48
#define AHEAD 6
constexpr int NDYN = NPIECE - NSTAT * NPROD;
constexpr int NMAX = 120;
#define EARLY0 (3 * 512 + 4 * 64)
#define HSLEEP 2
#define EARLY 4
constexpr unsigned LDS_BYTES = 32 * KD;
constexpr unsigned long long DEADLINE = 50000000ull;
constexpr int AUX_SYS = 17;
constexpr int AUX_AGENT = 16;
#define PWIN 2

#define STAMP(i) do {} while (0)
#define STAMP_FLUSH() do {} while (0)

struct Params {
  const uint8_t* x;
  char* myws;
  char* dst[NR];
  const uint8_t* w;
  const float* sx;
  const float* sw;
  float* out;
  int rank;
};

__device__ __forceinline__ __amdgpu_buffer_rsrc_t mk_rsrc(const void* p, unsigned bytes) {
  return __builtin_amdgcn_make_buffer_rsrc(const_cast<void*>(p), 0, bytes, 0x00020000);
}
__device__ __forceinline__ unsigned min3u(unsigned a, unsigned b, unsigned c) {
  return min(min(a, b), c);
}

typedef __attribute__((address_space(3))) int lds_int;
__device__ __forceinline__ void lds_st(lds_int* a, int v) { __hip_atomic_store(a, v, __ATOMIC_RELAXED, __HIP_MEMORY_SCOPE_WORKGROUP); }
__device__ __forceinline__ int lds_ld(lds_int* a) { return __hip_atomic_load(a, __ATOMIC_RELAXED, __HIP_MEMORY_SCOPE_WORKGROUP); }
__device__ __forceinline__ unsigned grp_off(int s) { return (unsigned)(((s >> 9) << 3) | ((s >> 3) & 7)) * GROUP; }
__device__ __forceinline__ unsigned pc_idx(int s) { return (unsigned)(((s >> 3) & 56) | (s & 7)); }
__device__ __forceinline__ unsigned src_off(int s) { return grp_off(s) + pc_idx(s) * 64; }
__device__ __forceinline__ unsigned dst_off(int s) { return grp_off(s) + pc_idx(s) * 1024; }

__global__ void __launch_bounds__(NTHR) ag_gemm_kernel(Params p) {
  extern __shared__ __attribute__((aligned(16))) char smem[];
  const int wid = __builtin_amdgcn_readfirstlane(threadIdx.x >> 6);
  const int lane = threadIdx.x & 63;
  const int bid = blockIdx.x;
  char* myws = p.myws;
  const uint8_t* px = p.x;
  char* dbase = p.dst[wid];
  asm volatile("" : "+s"(myws), "+s"(px), "+s"(dbase));
  const __amdgpu_buffer_rsrc_t rc = mk_rsrc(myws + CTRL_OFF, CTRL_BYTES);
  const unsigned cnt = 256 + (unsigned)(bid * NWAVE + wid) * 4;
  STAMP(0);

  const int xcd = bid & 7, cu = bid >> 3;
  const __amdgpu_buffer_rsrc_t rxs = mk_rsrc(px, SHARD);
  if (__builtin_expect(bid < NPROD, 1)) {
    lds_int* pos = (lds_int*)smem;
    lds_int* done = pos + 192;
    const int s5 = (cu & 1) | (xcd << 1) | ((cu >> 1) << 4);
    if (__builtin_expect(wid == NR - 1, 0)) {
      const unsigned craw = __builtin_amdgcn_raw_buffer_load_b32(rc, 0, cnt, AUX_AGENT);
      lds_st(pos + lane, -1); lds_st(pos + lane + 64, -1); lds_st(pos + lane + 128, -1); lds_st(done + lane, lane == 7 ? 0x7fffffff : 0);
      asm volatile("s_waitcnt lgkmcnt(0)" ::: "memory");
      __builtin_amdgcn_s_barrier();
      const unsigned call = __builtin_amdgcn_readfirstlane(craw) + 1;
      __builtin_amdgcn_raw_buffer_store_b32(call, rc, 0, cnt, AUX_AGENT);
      if (bid == 0) __builtin_amdgcn_raw_buffer_store_b32(0u, rc, 0, 64 + ((call + 1) & 1) * 64, AUX_AGENT);
      typedef __attribute__((address_space(1))) unsigned gu32;
      gu32* tick = (gu32*)(myws + CTRL_OFF + 64 + (call & 1) * 64);
      int n = NSTAT;
      for (;;) {
        int d = lds_ld(done + (lane & 7));
        d = min(d, __builtin_amdgcn_update_dpp(0x7fffffff, d, 0x111, 0xf, 0xf, false));
        d = min(d, __builtin_amdgcn_update_dpp(0x7fffffff, d, 0x112, 0xf, 0xf, false));
        d = min(d, __builtin_amdgcn_update_dpp(0x7fffffff, d, 0x114, 0xf, 0xf, false));
        const int lim = min(__builtin_amdgcn_readlane(d, 7) + AHEAD, NMAX);
        const int k = lim - n;
        if (k <= 0) {
          if (n >= NMAX) { lds_st(pos + n + (lane & 3), NPIECE); break; }
          if (k < -4) __builtin_amdgcn_s_sleep(64); else __builtin_amdgcn_s_sleep(2);
          continue;
        }
        unsigned t = 0;
        if (lane == 0) t = __hip_atomic_fetch_add(tick, (unsigned)k, __ATOMIC_RELAXED, __HIP_MEMORY_SCOPE_AGENT);
        const int t0 = (int)min(__builtin_amdgcn_readfirstlane(t), (unsigned)NDYN);
        const bool last = t0 + k >= NDYN;
        if (lane < (last ? k + 4 : k)) lds_st(pos + n + lane, min(NSTAT * NPROD + t0 + lane, NPIECE));
        if (last) break;
        n += k;
      }
      STAMP(5); STAMP(6);
      STAMP_FLUSH();
      return;
    }
    const unsigned vsrc = (unsigned)(lane & 15) * KD + (unsigned)(lane >> 4) * 16;
    const unsigned vdst = (unsigned)lane * 16;
    u32x4 r[2];
    r[0] = __builtin_amdgcn_raw_buffer_load_b128(rxs, vsrc, src_off(s5), 0);
    __builtin_amdgcn_sched_barrier(0);
    const unsigned craw = __builtin_amdgcn_raw_buffer_load_b32(rc, 0, cnt, AUX_AGENT);
    __builtin_amdgcn_sched_barrier(0);
    r[1] = __builtin_amdgcn_raw_buffer_load_b128(rxs, vsrc, src_off(s5 + NPROD), 0);
    __builtin_amdgcn_sched_barrier(0);
    __builtin_amdgcn_s_barrier();
    __builtin_amdgcn_sched_barrier(0);
    const unsigned call = __builtin_amdgcn_readfirstlane(craw) + 1;
    const __amdgpu_buffer_rsrc_t rd = mk_rsrc(dbase + (size_t)(call % NSLOT) * SLOT, SHARD);
    STAMP(1);
    int sp[2] = {s5, s5 + NPROD};
#pragma unroll 1
    for (int n0 = 0; sp[0] < NPIECE; n0 += 2) {
#pragma unroll
      for (int i = 0; i < 2; ++i) {
        const int n = n0 + i;
        const int s = sp[i];
        u32x4 v = r[i];
#pragma unroll
        for (int d = 0; d < 4; ++d) v[d] = v[d] ? v[d] : 0x80808080u;
        __builtin_amdgcn_raw_buffer_store_b128(v, rd, s < NPIECE ? vdst : SHARD, dst_off(s & (NPIECE - 1)), AUX_SYS);
        __builtin_amdgcn_sched_barrier(0);
        lds_st(done + wid, n + 1);
        int sn = s5 + (n + 2) * NPROD;
        if (n + 2 >= NSTAT) {
          for (;;) {
            sn = __builtin_amdgcn_readfirstlane(lds_ld(pos + n + 2));
            if (sn >= 0) break;
            __builtin_amdgcn_s_sleep(1);
          }
        }
        r[i] = __builtin_amdgcn_raw_buffer_load_b128(rxs, vsrc, src_off(sn & (NPIECE - 1)), 0);
        sp[i] = sn;
        __builtin_amdgcn_sched_barrier(0);
      }
    }
    STAMP(5);
    STAMP(6);
    __builtin_amdgcn_raw_buffer_store_b32(call, rc, 0, cnt, AUX_AGENT);
    STAMP_FLUSH();
    return;
  }

  const int ci = xcd * 28 + cu - NPROD / 8;
  const int strm = ci >> 4, slab = ci & 15;
  const int half = strm & 1;
  const int cw = wid;
  const unsigned craw = __builtin_amdgcn_raw_buffer_load_b32(rc, 0, cnt, AUX_AGENT);
  unsigned warm = 0;
  {
    const int wi = cw * 28 + cu - NPROD / 8;
    if (wi < 64) {
      const int g = (wi >> 4) * 8 + (wi & 1) * 4 + (lane >> 5) * 2 + (xcd >> 2);
      warm = __builtin_amdgcn_raw_buffer_load_b32(rxs, (unsigned)(g * 16 + ((lane >> 1) & 15)) * KD + (unsigned)(lane & 1) * 64,
                                                  (unsigned)(((wi >> 1) & 7) * 512 + (xcd & 3) * 128), 0);
    }
  }
  __builtin_amdgcn_sched_barrier(0);
  {
    const int kb = cw * 32 + (lane >> 1), nh = lane & 1;
    const __amdgpu_buffer_rsrc_t rw = mk_rsrc(p.w, KD * NLOC);
    const unsigned vo = (unsigned)kb * 16 * NLOC + (unsigned)slab * 32 + (unsigned)nh * 16;
    u32x4 r[16];
#pragma unroll
    for (int j = 0; j < 16; ++j) r[j] = __builtin_amdgcn_raw_buffer_load_b128(rw, vo, j * NLOC, 0);
    const int u = kb & 7;
    const int sw = u ^ (nh << 2);
    char* base = smem + (kb >> 3) * 4096 + nh * 2048 + (u >> 2) * 1024 + (u & 3) * 256;
#pragma unroll
    for (int d = 0; d < 4; ++d) {
      u32x4 o[4];
#pragma unroll
      for (int e = 0; e < 4; ++e) {
        const unsigned a0 = r[4 * e][d], a1 = r[4 * e + 1][d], a2 = r[4 * e + 2][d], a3 = r[4 * e + 3][d];
        const unsigned t0 = __builtin_amdgcn_perm(a1, a0, 0x05010400u), t1 = __builtin_amdgcn_perm(a1, a0, 0x07030602u);
        const unsigned t2 = __builtin_amdgcn_perm(a3, a2, 0x05010400u), t3 = __builtin_amdgcn_perm(a3, a2, 0x07030602u);
        o[0][e] = __builtin_amdgcn_perm(t2, t0, 0x05040100u);
        o[1][e] = __builtin_amdgcn_perm(t2, t0, 0x07060302u);
        o[2][e] = __builtin_amdgcn_perm(t3, t1, 0x05040100u);
        o[3][e] = __builtin_amdgcn_perm(t3, t1, 0x07060302u);
      }
#pragma unroll
      for (int e = 0; e < 4; ++e) *(u32x4*)(base + (((4 * d + e) ^ sw) * 16)) = o[e];
    }
  }
  asm volatile("" :: "v"(warm));
  const unsigned call = __builtin_amdgcn_readfirstlane(craw) + 1;
  __builtin_amdgcn_raw_buffer_store_b32(call, rc, 0, cnt, AUX_AGENT);
  const __amdgpu_buffer_rsrc_t rz = mk_rsrc(myws + (size_t)((call + 2) % NSLOT) * SLOT +
      (size_t)((unsigned)(p.rank + 1 + (ci >> 5)) & 7u) * SHARD + (size_t)((ci & 31) * 4 + (cw & 3)) * 16384, 16384);
  const u32x4 zero4 = {0u, 0u, 0u, 0u};
  const float scale = p.sx[0] * p.sw[0];
  const unsigned rsrc_rank = (unsigned)(p.rank + NR - 1 - (strm >> 1)) & 7u;
  const char* xrem = myws + (size_t)(call % NSLOT) * SLOT + (size_t)rsrc_rank * SHARD;
  const __amdgpu_buffer_rsrc_t rh = mk_rsrc(xrem, SHARD);
  const unsigned vh = (unsigned)(lane & 7) * 1024 + 1020;
  const int li = lane & 15, lg = lane >> 4;
  const char* wlA = smem + lg * 256 + ((li ^ lg) * 16);
  const char* wlB = smem + lg * 256 + ((li ^ (lg | 4)) * 16);
  asm volatile("s_waitcnt lgkmcnt(0)" ::: "memory");
  __builtin_amdgcn_s_barrier();
  asm volatile("" ::: "memory");
  STAMP(1);

  bool dead = false;
  const unsigned long long t0 = __builtin_amdgcn_s_memrealtime();
  auto expired = [&]() -> bool {
    return __builtin_amdgcn_readfirstlane(__builtin_amdgcn_raw_buffer_load_b32(rc, 0, 0, AUX_AGENT)) == call ||
           __builtin_amdgcn_s_memrealtime() - t0 > DEADLINE;
  };
  const int lgrp = (cw - 4) * 14 + strm;
  for (int task = (cw >= 4 && lgrp < 32) ? -1 : 0; task < 2; ++task) {
    const bool remote = task >= 0;
    const int rb = (cw >> 2) + 2 * task;
    const int g8 = half * 4 + (cw & 3);
    const int G = remote ? rb * 8 + g8 : lgrp;
    const unsigned src = remote ? rsrc_rank : (unsigned)p.rank;
    const __amdgpu_buffer_rsrc_t rx = mk_rsrc((remote ? xrem : (const char*)px) + (size_t)G * GROUP, GROUP);
    const unsigned vx = remote ? (unsigned)lane * 16 : (unsigned)(lane & 15) * KD + (unsigned)(lane >> 4) * 16;
    const int so = rb * 512 + g8 * 8;
    auto hint_off = [&](int c) -> unsigned { const int q = so + c * 64; return dst_off(min(q >= EARLY0 ? q - EARLY * 8 : q, NPIECE - 8)); };
    if (task == 1) {
      const int wi = cw * 28 + cu - NPROD / 8;
      if (wi < NDYN / 4)
        warm = __builtin_amdgcn_raw_buffer_load_b32(rxs, (unsigned)(lane >> 2) * KD + (unsigned)(lane & 3) * 64, src_off(NSTAT * NPROD + wi * 4), 0);
    }
    f32x4 acc0 = {0.f, 0.f, 0.f, 0.f}, acc1 = {0.f, 0.f, 0.f, 0.f};
    bool hint_ok = !remote;
#pragma unroll 1
    for (int c = 0; c < 8; ++c) {
      u32x4 wf[16];
#pragma unroll
      for (int ks = 0; ks < 4; ++ks) {
        const int off = (c * 4 + ks) * 4096;
        wf[4 * ks] = *(const u32x4*)(wlA + off); wf[4 * ks + 1] = *(const u32x4*)(wlB + off + 1024);
        wf[4 * ks + 2] = *(const u32x4*)(wlB + off + 2048); wf[4 * ks + 3] = *(const u32x4*)(wlA + off + 3072);
      }
      asm volatile("" ::: "memory");
      if (!hint_ok && !dead) {
        const unsigned ho = hint_off(c);
        for (unsigned n = 1;; ++n) {
          const unsigned h = __builtin_amdgcn_raw_buffer_load_b32(rh, vh, ho, AUX_SYS);
          if (__builtin_amdgcn_ballot_w64(h == 0u) == 0ull) break;
          __builtin_amdgcn_s_sleep(HSLEEP);
          if ((n & 63) == 0 && expired()) { dead = true; break; }
          asm volatile("" ::: "memory");
        }
      }
      u32x4 xv[8];
      unsigned hn = 1;
      if (remote) {
        const unsigned hno = hint_off(c + 1);
        for (unsigned n = 1;; ++n) {
          hn = __builtin_amdgcn_raw_buffer_load_b32(rh, vh, hno, AUX_SYS);
          __builtin_amdgcn_sched_barrier(0);
#pragma unroll
          for (int q = 0; q < 8; ++q) xv[q] = __builtin_amdgcn_raw_buffer_load_b128(rx, vx, (unsigned)(c * 8 + q) * 1024, AUX_SYS);
          unsigned m = 0xffffffffu;
#pragma unroll
          for (int q = 0; q < 8; ++q) m = min3u(min3u(m, xv[q][0], xv[q][1]), xv[q][2], xv[q][3]);
          if (__builtin_amdgcn_ballot_w64(m == 0u) == 0ull || dead) break;
          if (n > 8) __builtin_amdgcn_s_sleep(16);
          if ((n & 63) == 0 && expired()) { dead = true; break; }
          asm volatile("" ::: "memory");
        }
      } else {
#pragma unroll
        for (int q = 0; q < 8; ++q) xv[q] = __builtin_amdgcn_raw_buffer_load_b128(rx, vx, (unsigned)(c * 8 + q) * 64, 0);
      }
#pragma unroll
      for (int ks = 0; ks < 4; ++ks) {
        const u32x4 a00 = wf[4 * ks], a01 = wf[4 * ks + 1], a10 = wf[4 * ks + 2], a11 = wf[4 * ks + 3];
        const i32x8 b = {(int)xv[2 * ks][0], (int)xv[2 * ks][1], (int)xv[2 * ks][2], (int)xv[2 * ks][3],
                         (int)xv[2 * ks + 1][0], (int)xv[2 * ks + 1][1], (int)xv[2 * ks + 1][2], (int)xv[2 * ks + 1][3]};
        const i32x8 a0 = {(int)a00[0], (int)a00[1], (int)a00[2], (int)a00[3], (int)a01[0], (int)a01[1], (int)a01[2], (int)a01[3]};
        const i32x8 a1 = {(int)a10[0], (int)a10[1], (int)a10[2], (int)a10[3], (int)a11[0], (int)a11[1], (int)a11[2], (int)a11[3]};
        acc0 = __builtin_amdgcn_mfma_scale_f32_16x16x128_f8f6f4(a0, b, acc0,   1,   0, 0, 0, 0, 0);
        acc1 = __builtin_amdgcn_mfma_scale_f32_16x16x128_f8f6f4(a1, b, acc1, 1, 0, 0, 0, 0, 0);
      }
      hint_ok = __builtin_amdgcn_ballot_w64(hn == 0u) == 0ull;
      __builtin_amdgcn_sched_barrier(0);
      if (cw < 4) __builtin_amdgcn_raw_buffer_store_b128(zero4, rz, (unsigned)lane * 16, (unsigned)(task * 8 + c) * 1024, AUX_SYS);
    }
    STAMP(3 + task);
    if (dead) __builtin_amdgcn_raw_buffer_store_b32(call, rc, 0, 0, AUX_AGENT);
    f32x4 o0, o1;
#pragma unroll
    for (int r = 0; r < 4; ++r) {
      const float y0 = acc0[r] * scale, y1 = acc1[r] * scale;
      const float c0 = __builtin_amdgcn_fmed3f(y0, -60.f, 60.f), c1 = __builtin_amdgcn_fmed3f(y1, -60.f, 60.f);
      o0[r] = y0 * __builtin_amdgcn_rcpf(1.f + __expf(-c0));
      o1[r] = y1 * __builtin_amdgcn_rcpf(1.f + __expf(-c1));
      if (dead) { o0[r] = __builtin_nanf(""); o1[r] = __builtin_nanf(""); }
    }
    const __amdgpu_buffer_rsrc_t ro = mk_rsrc(p.out + ((size_t)src * MLOC + (size_t)G * 16) * NLOC, 16 * NLOC * 4);
    const unsigned vo = (unsigned)li * (NLOC * 4) + (unsigned)slab * 128 + (unsigned)lg * 16;
    __builtin_amdgcn_raw_buffer_store_b128(__builtin_bit_cast(u32x4, o0), ro, vo, 0, AUX_SYS);
    __builtin_amdgcn_raw_buffer_store_b128(__builtin_bit_cast(u32x4, o1), ro, vo, 64, AUX_SYS);
  }
  asm volatile("" :: "v"(warm));
  STAMP(5);
  STAMP(6);
  STAMP_FLUSH();
}

}

extern "C" void kernel_launch(int rank, int nranks, ncclComm_t comm,
                              hipStream_t stream,
                              void* const* d_in, const int* in_sizes, int n_in,
                              void* d_out, int out_size,
                              const int* in_dtypes, int out_dtype,
                              void* const* d_out_all,
                              void* const* d_ws_all, size_t ws_bytes) {
  (void)comm; (void)in_dtypes; (void)out_dtype; (void)d_out_all;
  if (nranks != NR || (unsigned)rank >= (unsigned)NR || n_in < 4 || in_sizes[0] != MLOC * KD || in_sizes[1] != KD * NLOC ||
      out_size != NR * MLOC * NLOC || ws_bytes < CTRL_OFF + CTRL_BYTES) {
    fprintf(stderr, "kernel_launch: unexpected problem shape\n");
    return;
  }
  static bool attr_done[NR];
  if (!attr_done[rank]) {
    HIP_CHECK(hipFuncSetAttribute((const void*)ag_gemm_kernel, hipFuncAttributeMaxDynamicSharedMemorySize, LDS_BYTES));
    attr_done[rank] = true;
  }
  Params p;
  p.x = (const uint8_t*)d_in[0];
  p.w = (const uint8_t*)d_in[1];
  p.sx = (const float*)d_in[2];
  p.sw = (const float*)d_in[3];
  p.out = (float*)d_out;
  p.myws = (char*)d_ws_all[rank];
  p.rank = rank;
  for (int j = 0; j < NR; ++j) p.dst[j] = (char*)d_ws_all[(rank + 1 + j) & 7] + (size_t)rank * SHARD;
  hipLaunchKernelGGL(ag_gemm_kernel, dim3(NBLK), dim3(NTHR), LDS_BYTES, stream, p);
  HIP_CHECK(hipGetLastError());
}
```

```cpp
#include <hip/hip_runtime.h>
#include <rccl/rccl.h>
#include <cstdio>
#include <cstdint>

#define HIP_CHECK(expr) do { hipError_t _e = (expr); if (_e != hipSuccess) { \
    fprintf(stderr, "HIP error %s at %s:%d\n", hipGetErrorString(_e), __FILE__, __LINE__); \
    return; } } while (0)

namespace {

typedef unsigned u32x4 __attribute__((ext_vector_type(4)));
typedef unsigned u32x2 __attribute__((ext_vector_type(2)));
typedef int i32x8 __attribute__((ext_vector_type(8)));
typedef float f32x4 __attribute__((ext_vector_type(4)));
typedef float f32x2 __attribute__((ext_vector_type(2)));
typedef unsigned long long u64;

constexpr int NR = 8;
constexpr int KD = 4096;
constexpr int MLOC = 512;
constexpr int NLOC = 512;
constexpr unsigned SHARD = MLOC * KD;
constexpr unsigned GROUP = 16 * KD;
constexpr unsigned SLOT = NR * SHARD;
constexpr int NSLOT = 3;
constexpr size_t CTRL_OFF = (size_t)NSLOT * SLOT;
constexpr unsigned CTRL_BYTES = 64 * 1024;
constexpr unsigned C_TICK = 64, C_CNT = 256, C_FLAG = 16384;
constexpr int NBLK = 256;
constexpr int NWAVE = 8;
constexpr int NTHR = NWAVE * 64;
constexpr unsigned LDS_BYTES = 96 * 1024;
constexpr u64 DEADLINE = 50000000ull;
constexpr int AUX_SYS = 17;
constexpr int AUX_AGENT = 16;

#define PHI 4
#define PSI 2
constexpr int NNO = 32 - PHI - PSI;
constexpr int NI = NNO + PSI;
constexpr int NX = PHI + PSI;
constexpr int NIP = 16;
constexpr int NUNIT = NI * 32;
#define NDYNP 14
constexpr int NSTAT = NUNIT / NIP - NDYNP;
constexpr int NDYN = NUNIT - NSTAT * NIP;
constexpr int NMAX = 120;
#define AHEAD 6
#define EARLY 4
#define EARLY0 (NSTAT * NIP * 2 + 192)
#define HSLEEP 2
constexpr int NZG = 3 * NI + 4 * (8 + NX);
static_assert(NNO >= 8 && PHI >= 1, "W quarters live in the unused groups of a cross source");

#define STAMP(i) do {} while (0)
#define STAMP_FLUSH() do {} while (0)
#define STAMP_DRAIN() do {} while (0)

struct Params {
  const uint8_t* x;
  char* myws;
  char* ws[NR];
  const uint8_t* w;
  const float* sx;
  const float* sw;
  float* out[NR];
  unsigned rk;
  unsigned wq;
  u64 task[2][14][2];
};

__device__ __forceinline__ __amdgpu_buffer_rsrc_t mk_rsrc(const void* p, unsigned bytes) {
  return __builtin_amdgcn_make_buffer_rsrc(const_cast<void*>(p), 0, bytes, 0x00020000);
}
__device__ __forceinline__ unsigned min3u(unsigned a, unsigned b, unsigned c) { return min(min(a, b), c); }
__device__ __forceinline__ unsigned min4(unsigned m, u32x4 v) { return min3u(min3u(m, v[0], v[1]), v[2], v[3]); }
__device__ __forceinline__ u32x4 canon(u32x4 v) {
#pragma unroll
  for (int d = 0; d < 4; ++d) v[d] = v[d] ? v[d] : 0x80808080u;
  return v;
}
typedef __attribute__((address_space(3))) int lds_int;
__device__ __forceinline__ void lds_st(lds_int* a, int v) { __hip_atomic_store(a, v, __ATOMIC_RELAXED, __HIP_MEMORY_SCOPE_WORKGROUP); }
__device__ __forceinline__ int lds_ld(lds_int* a) { return __hip_atomic_load(a, __ATOMIC_RELAXED, __HIP_MEMORY_SCOPE_WORKGROUP); }
__device__ __forceinline__ unsigned src_off(int s) { return (unsigned)(s >> 6) * GROUP + (unsigned)(s & 63) * 64; }
__device__ __forceinline__ void tr16(const unsigned (&r)[16], u32x4 (&o)[4]) {
#pragma unroll
  for (int d = 0; d < 4; ++d) {
    const unsigned a0 = r[4 * d], a1 = r[4 * d + 1], a2 = r[4 * d + 2], a3 = r[4 * d + 3];
    const unsigned t0 = __builtin_amdgcn_perm(a1, a0, 0x05010400u), t1 = __builtin_amdgcn_perm(a1, a0, 0x07030602u);
    const unsigned t2 = __builtin_amdgcn_perm(a3, a2, 0x05010400u), t3 = __builtin_amdgcn_perm(a3, a2, 0x07030602u);
    o[0][d] = __builtin_amdgcn_perm(t2, t0, 0x05040100u);
    o[1][d] = __builtin_amdgcn_perm(t2, t0, 0x07060302u);
    o[2][d] = __builtin_amdgcn_perm(t3, t1, 0x05040100u);
    o[3][d] = __builtin_amdgcn_perm(t3, t1, 0x07060302u);
  }
}

__global__ void __launch_bounds__(NTHR) ag_gemm_kernel(Params p) {
  extern __shared__ __attribute__((aligned(16))) char smem[];
  const int wid = __builtin_amdgcn_readfirstlane(threadIdx.x >> 6);
  const int lane = threadIdx.x & 63;
  const int bid = blockIdx.x;
  const int xcd = bid & 7, cu = bid >> 3;
  const int drole = xcd < 4 ? 1 + (wid >= 3 ? wid - 3 : wid) % 3 : xcd;
  char* myws = p.myws;
  const uint8_t* px = p.x;
  char* pws = p.ws[drole];
  unsigned rk = p.rk;
  asm volatile("" : "+s"(myws), "+s"(px), "+s"(pws), "+s"(rk));
  const unsigned rank = rk & 7u;
  const __amdgpu_buffer_rsrc_t rc = mk_rsrc(myws + CTRL_OFF, CTRL_BYTES);
  const unsigned cnt = C_CNT + (unsigned)(bid * NWAVE + wid) * 4;
  const __amdgpu_buffer_rsrc_t rxs = mk_rsrc(px, SHARD);
  const int li = lane & 15, lg = lane >> 4;
  const unsigned vsrc = (unsigned)li * KD + (unsigned)lg * 16;
  const unsigned vlin = (unsigned)lane * 16;
  STAMP(0);

  if (__builtin_expect(cu < 4 && xcd < 4, 1)) {
    lds_int* pos = (lds_int*)smem;
    lds_int* done = pos + 192;
    const int pidx = cu * 4 + xcd;
    if (__builtin_expect(wid >= 6, 0)) {
      if (wid == 6) { __builtin_amdgcn_s_barrier(); return; }
      const unsigned craw = __builtin_amdgcn_raw_buffer_load_b32(rc, 0, cnt, AUX_AGENT);
      lds_st(pos + lane, -1); lds_st(pos + lane + 64, -1); lds_st(pos + lane + 128, -1); lds_st(done + lane, lane >= 6 ? 0x7fffffff : 0);
      asm volatile("s_waitcnt lgkmcnt(0)" ::: "memory");
      __builtin_amdgcn_s_barrier();
      const unsigned call = __builtin_amdgcn_readfirstlane(craw) + 1;
      __builtin_amdgcn_raw_buffer_store_b32(call, rc, 0, cnt, AUX_AGENT);
      if (bid == 0) __builtin_amdgcn_raw_buffer_store_b32(0u, rc, 0, C_TICK + ((call + 1) & 1) * 64, AUX_AGENT);
      typedef __attribute__((address_space(1))) unsigned gu32;
      gu32* tick = (gu32*)(myws + CTRL_OFF + C_TICK + (call & 1) * 64);
      int n = NSTAT;
      for (;;) {
        int d = lds_ld(done + (lane & 7));
        d = min(d, __builtin_amdgcn_update_dpp(0x7fffffff, d, 0x111, 0xf, 0xf, false));
        d = min(d, __builtin_amdgcn_update_dpp(0x7fffffff, d, 0x112, 0xf, 0xf, false));
        d = min(d, __builtin_amdgcn_update_dpp(0x7fffffff, d, 0x114, 0xf, 0xf, false));
        const int lim = min(__builtin_amdgcn_readlane(d, 7) + AHEAD, NMAX);
        const int k = lim - n;
        if (k <= 0) {
          if (n >= NMAX) { lds_st(pos + n + (lane & 3), NUNIT); break; }
          if (k < -4) __builtin_amdgcn_s_sleep(64); else __builtin_amdgcn_s_sleep(2);
          continue;
        }
        unsigned t = 0;
        if (lane == 0) t = __hip_atomic_fetch_add(tick, (unsigned)k, __ATOMIC_RELAXED, __HIP_MEMORY_SCOPE_AGENT);
        const int t0 = (int)min(__builtin_amdgcn_readfirstlane(t), (unsigned)NDYN);
        const bool last = t0 + k >= NDYN;
        if (lane < (last ? k + 4 : k)) lds_st(pos + n + lane, min(NSTAT * NIP + t0 + lane, NUNIT));
        if (last) break;
        n += k;
      }
      STAMP(6);
      STAMP_FLUSH();
      return;
    }
    const int h = wid >= 3;
    u32x4 r[2];
    r[0] = __builtin_amdgcn_raw_buffer_load_b128(rxs, vsrc, src_off(2 * pidx + h), 0);
    __builtin_amdgcn_sched_barrier(0);
    const unsigned craw = __builtin_amdgcn_raw_buffer_load_b32(rc, 0, cnt, AUX_AGENT);
    __builtin_amdgcn_sched_barrier(0);
    r[1] = __builtin_amdgcn_raw_buffer_load_b128(rxs, vsrc, src_off(2 * (pidx + NIP) + h), 0);
    __builtin_amdgcn_sched_barrier(0);
    __builtin_amdgcn_s_barrier();
    __builtin_amdgcn_sched_barrier(0);
    const unsigned call = __builtin_amdgcn_readfirstlane(craw) + 1;
    const __amdgpu_buffer_rsrc_t rd = mk_rsrc(pws + (size_t)(call % NSLOT) * SLOT + (size_t)rank * SHARD, SHARD);
    int up[2] = {pidx, pidx + NIP};
#pragma unroll 1
    for (int n0 = 0; up[0] < NUNIT; n0 += 2) {
#pragma unroll
      for (int i = 0; i < 2; ++i) {
        const int n = n0 + i;
        const int u = up[i];
        __builtin_amdgcn_raw_buffer_store_b128(canon(r[i]), rd, u < NUNIT ? vlin : SHARD, (unsigned)(2 * u + h) * 1024, AUX_SYS);
        __builtin_amdgcn_sched_barrier(0);
        lds_st(done + wid, n + 1);
        int un = pidx + (n + 2) * NIP;
        if (n + 2 >= NSTAT) {
          for (;;) {
            un = __builtin_amdgcn_readfirstlane(lds_ld(pos + n + 2));
            if (un >= 0) break;
            __builtin_amdgcn_s_sleep(1);
          }
        }
        r[i] = __builtin_amdgcn_raw_buffer_load_b128(rxs, vsrc, src_off(2 * un + h), 0);
        up[i] = un;
        __builtin_amdgcn_sched_barrier(0);
      }
    }
    STAMP(5);
    STAMP_DRAIN();
    STAMP(6);
    __builtin_amdgcn_raw_buffer_store_b32(call, rc, 0, cnt, AUX_AGENT);
    STAMP_FLUSH();
    return;
  }

  if (cu < 4) {
    constexpr int NPW = NX * 2;
    const int s5 = cu * 2 + (wid & 1) + (wid >> 1) * 8;
    auto spos = [&](int n) -> int { const int g = n >> 1; return (g < PHI ? NI + g : NNO + g - PHI) * 64 + (n & 1) * 32 + s5; };
    u32x4 r[NPW];
    r[0] = __builtin_amdgcn_raw_buffer_load_b128(rxs, vsrc, src_off(spos(0)), 0);
    __builtin_amdgcn_sched_barrier(0);
    const unsigned craw = __builtin_amdgcn_raw_buffer_load_b32(rc, 0, cnt, AUX_AGENT);
    __builtin_amdgcn_sched_barrier(0);
    r[1] = __builtin_amdgcn_raw_buffer_load_b128(rxs, vsrc, src_off(spos(1)), 0);
    __builtin_amdgcn_sched_barrier(0);
    const unsigned call = __builtin_amdgcn_readfirstlane(craw) + 1;
    const __amdgpu_buffer_rsrc_t rd = mk_rsrc(pws + (size_t)(call % NSLOT) * SLOT + (size_t)rank * SHARD, SHARD);
    __builtin_amdgcn_raw_buffer_store_b128(canon(r[0]), rd, vlin, (unsigned)spos(0) * 1024, AUX_SYS);
    STAMP(2);
    __builtin_amdgcn_raw_buffer_store_b128(canon(r[1]), rd, vlin, (unsigned)spos(1) * 1024, AUX_SYS);
    __builtin_amdgcn_sched_barrier(0);
#pragma unroll
    for (int n = 2; n < NPW; ++n) r[n] = __builtin_amdgcn_raw_buffer_load_b128(rxs, vsrc, src_off(spos(n)), 0);
    asm volatile("s_waitcnt vmcnt(0)" ::: "memory");
    STAMP(3);
#pragma unroll
    for (int n = 2; n < NPW; ++n) {
      __builtin_amdgcn_raw_buffer_store_b128(canon(r[n]), rd, vlin, (unsigned)spos(n) * 1024, AUX_SYS);
      asm volatile("s_waitcnt vmcnt(1)" ::: "memory");
    }
    STAMP_DRAIN();
    STAMP(4);
    __builtin_amdgcn_raw_buffer_store_b32(call, rc, 0, cnt, AUX_AGENT);
    const __amdgpu_buffer_rsrc_t rz = mk_rsrc(myws + (size_t)((call + 2) % NSLOT) * SLOT, SLOT);
    const u32x4 zero4 = {0u, 0u, 0u, 0u};
    const int zw = ((xcd - 4) * 4 + cu) * NWAVE + wid;
#pragma unroll 1
    for (int G = zw >> 6; G < NZG; G += 2) {
      int role, g;
      if (G < 3 * NI) { role = 1 + G / NI; g = G % NI; }
      else { const int q = G - 3 * NI; role = 4 + q / (8 + NX); g = q % (8 + NX); if (g >= 8) g += NNO - 8; }
      __builtin_amdgcn_raw_buffer_store_b128(zero4, rz, vlin, ((rk >> (4 * role)) & 7u) * SHARD + (unsigned)g * GROUP + (unsigned)(zw & 63) * 1024, AUX_SYS);
      asm volatile("s_waitcnt vmcnt(3)" ::: "memory");
    }
    STAMP_DRAIN();
    STAMP(5);
    if (cu == 0 && wid == 0) {
      const u64 t0 = __builtin_amdgcn_s_memrealtime();
      const unsigned fo = C_FLAG + ((rk >> (4 * xcd)) & 7u) * 256;
      for (unsigned n = 1;; ++n) {
        const unsigned f = __builtin_amdgcn_raw_buffer_load_b32(rc, (unsigned)min(lane, 27) * 4, fo, AUX_SYS);
        if (__builtin_amdgcn_ballot_w64(f != call) == 0ull) break;
        if ((n & 63) == 0 && __builtin_amdgcn_s_memrealtime() - t0 > DEADLINE) {
          __builtin_amdgcn_raw_buffer_store_b32(call, rc, 0, 0, AUX_AGENT);
          p.out[0][(size_t)rank * MLOC * NLOC + lane] = __builtin_nanf("");
          break;
        }
        __builtin_amdgcn_s_sleep(1);
        asm volatile("" ::: "memory");
      }
    }
    STAMP(6);
    STAMP_FLUSH();
    return;
  }

  const bool cross = xcd >= 4;
  const int c28 = cu - 4;
  const int ci = xcd * 28 + c28;
  const int strm = cross ? c28 >> 1 : ci >> 3;
  const int slab = cross ? c28 & 1 : ci & 7;
  const unsigned craw = __builtin_amdgcn_raw_buffer_load_b32(rc, 0, cnt, AUX_AGENT);
  const __amdgpu_buffer_rsrc_t rw = mk_rsrc(p.w, KD * NLOC);
  const unsigned vw = (unsigned)lg * 16 * NLOC + (unsigned)li * 4;
  u64 tl = p.task[cross][strm][0], th = p.task[cross][strm][1];
  bool dead = false;
  const u64 t0 = __builtin_amdgcn_s_memrealtime();
  unsigned call;
  auto expired = [&]() -> bool {
    return __builtin_amdgcn_readfirstlane(__builtin_amdgcn_raw_buffer_load_b32(rc, 0, 0, AUX_AGENT)) == call ||
           __builtin_amdgcn_s_memrealtime() - t0 > DEADLINE;
  };
  u32x4 wf[8][4];
  if (!cross) {
    unsigned warm = 0;
    {
      const int wi = wid * 28 + c28;
      if (wi < NI * 4)
        warm = __builtin_amdgcn_raw_buffer_load_b32(rxs, (unsigned)(lane >> 4) * KD + (unsigned)((lane >> 1) & 7) * 512 + (unsigned)(lane & 1) * 64,
                                                    (unsigned)wi * 4 * KD + (unsigned)xcd * 128, 0);
    }
    __builtin_amdgcn_sched_barrier(0);
    const unsigned vo = vw + (unsigned)wid * 512 * NLOC + (unsigned)slab * 64;
    unsigned raw[3][16];
#pragma unroll
    for (int u = 0; u < 10; ++u) {
      if (u < 8) {
#pragma unroll
        for (int i = 0; i < 16; ++i) raw[u % 3][i] = __builtin_amdgcn_raw_buffer_load_b32(rw, vo, (unsigned)(u * 64 + i) * NLOC, 0);
      }
      __builtin_amdgcn_sched_barrier(0);
      if (u >= 2) tr16(raw[(u - 2) % 3], wf[u - 2]);
      __builtin_amdgcn_sched_barrier(0);
    }
    asm volatile("" :: "v"(warm));
    call = __builtin_amdgcn_readfirstlane(craw) + 1;
  } else {
    const int id = wid * 28 + c28;
    if (id < 128) {
      unsigned raw[16];
      const unsigned so = (unsigned)(id >> 1) * 64 * NLOC + ((p.wq >> (4 * (xcd - 4))) & 3u) * 128 + (unsigned)(id & 1) * 64;
#pragma unroll
      for (int i = 0; i < 16; ++i) raw[i] = __builtin_amdgcn_raw_buffer_load_b32(rw, vw + (unsigned)i * NLOC, so, 0);
      u32x4 o[4];
      tr16(raw, o);
      call = __builtin_amdgcn_readfirstlane(craw) + 1;
      const __amdgpu_buffer_rsrc_t rd = mk_rsrc(pws + (size_t)(call % NSLOT) * SLOT + (size_t)rank * SHARD, SHARD);
#pragma unroll
      for (int e = 0; e < 4; ++e)
        __builtin_amdgcn_raw_buffer_store_b128(canon(o[e]), rd, vlin + (unsigned)e * 1024, (unsigned)((id & 1) * 64 + (id >> 1)) * 4096, AUX_SYS);
      STAMP(2);
    }
    call = __builtin_amdgcn_readfirstlane(craw) + 1;
    const __amdgpu_buffer_rsrc_t rq = mk_rsrc(myws + (size_t)(call % NSLOT) * SLOT + (size_t)((rk >> (4 * xcd)) & 7u) * SHARD +
                                              (size_t)(slab * 64 + wid * 8) * 4096, 32768);
    for (unsigned n = 1;; ++n) {
      const unsigned hh = __builtin_amdgcn_raw_buffer_load_b32(rq, (unsigned)(lane & 31) * 1024 + 1020, 0, AUX_SYS);
      if (__builtin_amdgcn_ballot_w64(hh == 0u) == 0ull) break;
      __builtin_amdgcn_s_sleep(8);
      if ((n & 63) == 0 && expired()) { dead = true; break; }
      asm volatile("" ::: "memory");
    }
    STAMP(3);
    for (unsigned n = 1;; ++n) {
      unsigned m = 0xffffffffu;
#pragma unroll
      for (int q = 0; q < 8; ++q)
#pragma unroll
        for (int e = 0; e < 4; ++e) wf[q][e] = __builtin_amdgcn_raw_buffer_load_b128(rq, vlin + (unsigned)e * 1024, (unsigned)q * 4096, AUX_SYS);
#pragma unroll
      for (int q = 0; q < 8; ++q)
#pragma unroll
        for (int e = 0; e < 4; ++e) m = min4(m, wf[q][e]);
      if (__builtin_amdgcn_ballot_w64(m == 0u) == 0ull || dead) break;
      __builtin_amdgcn_s_sleep(8);
      if ((n & 63) == 0 && expired()) { dead = true; break; }
      asm volatile("" ::: "memory");
    }
  }
  STAMP(1);
  const float scale = p.sx[0] * p.sw[0];
  const __amdgpu_buffer_rsrc_t rs = mk_rsrc(myws + (size_t)(call % NSLOT) * SLOT, SLOT);
  const __amdgpu_buffer_rsrc_t ro = mk_rsrc(p.out[cross ? xcd : 0], NR * MLOC * NLOC * 4 - (cross ? 3 * 512 : 0));
  const unsigned vh = (unsigned)(lane & 7) * 1024 + 1020;
  const int orow = 2 * wid + (lane >> 5), hq = lane & 31;
  const unsigned vout = (unsigned)orow * (NLOC * 4) + (unsigned)slab * 256 + (unsigned)hq * 8;
  char* const lw = smem + wid * 4096 + li * 256;
  const char* const lr = smem + orow * 256 + (((hq >> 1) ^ orow) * 16) + (hq & 1) * 8;

  auto next = [&]() -> int {
    const int c = (int)(tl & 255u);
    tl = (tl >> 8) | (th << 56); th = (th >> 8) | (63ull << 56);
    if (c == 63) return -1;
    int role = c >> 5;
    if (cross && role >= 4) role = 4 + ((role + xcd) & 3);
    return (int)(((rk >> (4 * role)) & 7u) << 8) | c;
  };
  u32x4 xv[8];
  auto issue = [&](int code) {
    if ((code & 0xe0) == 0) {
      const unsigned so = (unsigned)(code & 31) * GROUP + (unsigned)wid * 512;
#pragma unroll
      for (int q = 0; q < 8; ++q) xv[q] = __builtin_amdgcn_raw_buffer_load_b128(rxs, vsrc + (unsigned)q * 64, so, 0);
    } else {
      const unsigned so = (unsigned)(code >> 8) * SHARD + (unsigned)(code & 31) * GROUP + (unsigned)wid * 8192;
#pragma unroll
      for (int q = 0; q < 4; ++q) xv[q] = __builtin_amdgcn_raw_buffer_load_b128(rs, vlin + (unsigned)q * 1024, so, AUX_SYS);
#pragma unroll
      for (int q = 4; q < 8; ++q) xv[q] = __builtin_amdgcn_raw_buffer_load_b128(rs, vlin + (unsigned)(q - 4) * 1024, so + 4096, AUX_SYS);
    }
  };
  int code = next();
  issue(code);
  __builtin_amdgcn_sched_barrier(0);
  __builtin_amdgcn_raw_buffer_store_b32(call, rc, 0, cnt, AUX_AGENT);
  __builtin_amdgcn_sched_barrier(0);
  int buf = 0;
#pragma unroll 1
  for (;;) {
    if (code & 0xe0) {
      unsigned m = 0xffffffffu;
#pragma unroll
      for (int q = 0; q < 8; ++q) m = min4(m, xv[q]);
      if (__builtin_expect(__builtin_amdgcn_ballot_w64(m == 0u) != 0ull && !dead, 0)) {
        const int s = (code & 31) * 64 + wid * 8;
        const unsigned ho = (unsigned)(code >> 8) * SHARD + (unsigned)((code & 0x80) == 0 && s >= EARLY0 ? s - EARLY * 8 : s) * 1024;
        for (unsigned n = 1;; ++n) {
          const unsigned hh = __builtin_amdgcn_raw_buffer_load_b32(rs, vh, ho, AUX_SYS);
          if (__builtin_amdgcn_ballot_w64(hh == 0u) == 0ull) break;
          __builtin_amdgcn_s_sleep(HSLEEP);
          if ((n & 63) == 0 && expired()) { dead = true; break; }
          asm volatile("" ::: "memory");
        }
        for (unsigned n = 1;; ++n) {
          issue(code);
          m = 0xffffffffu;
#pragma unroll
          for (int q = 0; q < 8; ++q) m = min4(m, xv[q]);
          if (__builtin_amdgcn_ballot_w64(m == 0u) == 0ull || dead) break;
          if (n > 8) __builtin_amdgcn_s_sleep(16);
          if ((n & 63) == 0 && expired()) { dead = true; break; }
          asm volatile("" ::: "memory");
        }
        if (dead) __builtin_amdgcn_raw_buffer_store_b32(call, rc, 0, 0, AUX_AGENT);
      }
    }
    f32x4 acc[4];
#pragma unroll
    for (int e = 0; e < 4; ++e) acc[e] = f32x4{0.f, 0.f, 0.f, 0.f};
#pragma unroll
    for (int ks = 0; ks < 4; ++ks) {
      const i32x8 b = {(int)xv[2 * ks][0], (int)xv[2 * ks][1], (int)xv[2 * ks][2], (int)xv[2 * ks][3],
                       (int)xv[2 * ks + 1][0], (int)xv[2 * ks + 1][1], (int)xv[2 * ks + 1][2], (int)xv[2 * ks + 1][3]};
#pragma unroll
      for (int e = 0; e < 4; ++e) {
        const u32x4 a0 = wf[2 * ks][e], a1 = wf[2 * ks + 1][e];
        const i32x8 a = {(int)a0[0], (int)a0[1], (int)a0[2], (int)a0[3], (int)a1[0], (int)a1[1], (int)a1[2], (int)a1[3]};
        acc[e] = __builtin_amdgcn_mfma_scale_f32_16x16x128_f8f6f4(a, b, acc[e],   1,   0, 0, 0, 0, 0);
      }
    }
    __builtin_amdgcn_sched_barrier(0);
    const int ncode = next();
    if (ncode >= 0) issue(ncode);
    __builtin_amdgcn_sched_barrier(0);
#pragma unroll
    for (int r = 0; r < 4; ++r) {
      f32x4 v = {acc[0][r], acc[1][r], acc[2][r], acc[3][r]};
      if (dead) v[0] = __builtin_nanf("");
      *(f32x4*)(lw + buf * 32768 + (((lg * 4 + r) ^ li) * 16)) = v;
    }
    asm volatile("s_waitcnt lgkmcnt(0)" ::: "memory");
    __builtin_amdgcn_s_barrier();
    asm volatile("" ::: "memory");
    f32x2 pr[8];
#pragma unroll
    for (int v = 0; v < 8; ++v) pr[v] = *(const f32x2*)(lr + buf * 32768 + v * 4096);
    const f32x2 sum = ((pr[0] + pr[1]) + (pr[2] + pr[3])) + ((pr[4] + pr[5]) + (pr[6] + pr[7]));
    f32x2 o;
#pragma unroll
    for (int r = 0; r < 2; ++r) {
      const float y = sum[r] * scale;
      o[r] = y * __builtin_amdgcn_rcpf(1.f + __expf(-__builtin_amdgcn_fmed3f(y, -60.f, 60.f)));
    }
    __builtin_amdgcn_raw_buffer_store_b64(__builtin_bit_cast(u32x2, o), ro, vout, ((unsigned)(code >> 8) * MLOC + (unsigned)(code & 31) * 16) * (NLOC * 4), AUX_SYS);
    if (ncode < 0) break;
    code = ncode;
    buf ^= 1;
  }
  STAMP(5);
  if (cross) {
    asm volatile("s_waitcnt vmcnt(0)" ::: "memory");
    __builtin_amdgcn_s_barrier();
    if (wid == 0) __builtin_amdgcn_raw_buffer_store_b32(call, mk_rsrc(pws + CTRL_OFF, CTRL_BYTES), 0, C_FLAG + rank * 256 + (unsigned)c28 * 4, AUX_SYS);
  }
  STAMP_DRAIN();
  STAMP(6);
  STAMP_FLUSH();
}

void build_tasks(Params& p) {
  for (int t = 0; t < 2; ++t) {
    uint8_t lst[14][16];
    int len[14];
    const int ni = t ? NNO : NI;
    uint8_t fill[64];
    int nf = 0;
    for (int g = 0; g < (t ? NNO : 32); ++g) fill[nf++] = (uint8_t)g;
    for (int i = 0; i < (t ? PHI : NX); ++i)
      for (int r = t ? 5 : 4; r < 8; ++r) fill[nf++] = (uint8_t)(r << 5 | (i < PHI ? NI + i : NNO + i - PHI));
    const int total = 3 * ni + nf;
    int room[14], used = 0;
    for (int j = 0; j < 14; ++j) {
      const int own = j < 12 ? (ni - (j & 3) + 3) / 4 : 0;
      room[j] = (total + 13) / 14 - own;
      used += room[j];
      len[j] = 0;
    }
    for (int j = 0; used > nf; j = (j + 1) % 12) if (room[j] > 0) { --room[j]; --used; }
    int f = 0;
    for (int round = 0; round < 16; ++round)
      for (int j = 0; j < 12; ++j) if (round < room[j]) lst[j][len[j]++] = fill[f++];
    for (int round = 0; round < 16; ++round)
      for (int j = 12; j < 14; ++j) if (round < room[j]) lst[j][len[j]++] = fill[f++];
    for (int j = 0; j < 12; ++j)
      for (int g = j & 3; g < ni; g += 4) lst[j][len[j]++] = (uint8_t)((1 + (j >> 2)) << 5 | g);
    for (int j = 0; j < 14; ++j) {
      u64 v[2] = {0, 0};
      for (int i = 0; i < 16; ++i) v[i >> 3] |= (u64)(i < len[j] ? lst[j][i] : 63) << (8 * (i & 7));
      p.task[t][j][0] = v[0]; p.task[t][j][1] = v[1];
    }
  }
}

}

extern "C" void kernel_launch(int rank, int nranks, ncclComm_t comm,
                              hipStream_t stream,
                              void* const* d_in, const int* in_sizes, int n_in,
                              void* d_out, int out_size,
                              const int* in_dtypes, int out_dtype,
                              void* const* d_out_all,
                              void* const* d_ws_all, size_t ws_bytes) {
  (void)comm; (void)in_dtypes; (void)out_dtype;
  if (nranks != NR || (unsigned)rank >= (unsigned)NR || n_in < 4 || in_sizes[0] != MLOC * KD || in_sizes[1] != KD * NLOC ||
      out_size != NR * MLOC * NLOC || ws_bytes < CTRL_OFF + CTRL_BYTES) {
    fprintf(stderr, "kernel_launch: unexpected problem shape\n");
    return;
  }
  static bool attr_done[NR];
  if (!attr_done[rank]) {
    HIP_CHECK(hipFuncSetAttribute((const void*)ag_gemm_kernel, hipFuncAttributeMaxDynamicSharedMemorySize, LDS_BYTES));
    attr_done[rank] = true;
  }
  Params p;
  p.x = (const uint8_t*)d_in[0];
  p.w = (const uint8_t*)d_in[1];
  p.sx = (const float*)d_in[2];
  p.sw = (const float*)d_in[3];
  p.myws = (char*)d_ws_all[rank];
  const int quad = 0x4d;
  auto qidx = [&](int r) { int n = 0; for (int q = 0; q < r; ++q) n += ((quad >> q) & 1) == ((quad >> r) & 1); return n; };
  int role[NR], na = 1, nb = 4;
  role[0] = rank;
  for (int j = 1; j < NR; ++j) {
    const int q = (rank + j) & 7;
    if (((quad >> q) & 1) == ((quad >> rank) & 1)) role[na++] = q; else role[nb++] = q;
  }
  p.rk = 0; p.wq = 0;
  for (int i = 0; i < NR; ++i) {
    p.ws[i] = (char*)d_ws_all[role[i]];
    p.out[i] = i == 0 ? (float*)d_out : (float*)d_out_all[role[i]] + (i >= 4 ? qidx(rank) * 128 : 0);
    p.rk |= (unsigned)role[i] << (4 * i);
    if (i >= 4) p.wq |= (unsigned)qidx(role[i]) << (4 * (i - 4));
  }
  build_tasks(p);
  hipLaunchKernelGGL(ag_gemm_kernel, dim3(NBLK), dim3(NTHR), LDS_BYTES, stream, p);
  HIP_CHECK(hipGetLastError());
}
```

```cpp
#include <hip/hip_runtime.h>
#include <rccl/rccl.h>
#include <cstdio>
#include <cstdint>

#define HIP_CHECK(expr) do { hipError_t _e = (expr); if (_e != hipSuccess) { \
    fprintf(stderr, "HIP error %s at %s:%d\n", hipGetErrorString(_e), __FILE__, __LINE__); \
    return; } } while (0)

namespace {

typedef unsigned u32x4 __attribute__((ext_vector_type(4)));
typedef unsigned u32x2 __attribute__((ext_vector_type(2)));
typedef int i32x8 __attribute__((ext_vector_type(8)));
typedef float f32x4 __attribute__((ext_vector_type(4)));
typedef float f32x2 __attribute__((ext_vector_type(2)));
typedef unsigned long long u64;

constexpr int NR = 8;
constexpr int KD = 4096;
constexpr int MLOC = 512;
constexpr int NLOC = 512;
constexpr unsigned SHARD = MLOC * KD;
constexpr unsigned GROUP = 16 * KD;
constexpr unsigned SLOT = NR * SHARD;
constexpr int NSLOT = 3;
constexpr size_t CTRL_OFF = (size_t)NSLOT * SLOT;
constexpr unsigned CTRL_BYTES = 64 * 1024;
constexpr unsigned C_CNT = 256, C_FLAG = 16384, C_TICK = 32768;
constexpr int NBLK = 256;
constexpr int NWAVE = 8;
constexpr int NTHR = NWAVE * 64;
constexpr unsigned LDS_BYTES = 96 * 1024;
constexpr u64 DEADLINE = 50000000ull;
constexpr int AUX_SYS = 17;
constexpr int AUX_AGENT = 16;

#define PHI 4
#define PSI 2
constexpr int NNO = 32 - PHI - PSI;
constexpr int NI = NNO + PSI;
constexpr int NX = PHI + PSI;
constexpr int NIP = 16;
constexpr int NUNIT = NI * 32;
#define NSTAT 6
#define NDYNP 10
constexpr int NDYN = NDYNP * NIP;
constexpr int NXT = (NUNIT - NSTAT * NIP - NDYN) / 4;
constexpr int NMAX = 120;
#define AHEAD 6
#define AHEAD1 8
#define EARLY 4
#define EARLY0 ((NUNIT - NDYN) * 2 + 128)
#define HSLEEP 2
#define T_S 1050
#define T_P 1200
#define WWIN 1
#define W1WIN 1
#define SWIN 1
#define T_W2 600
#define T_L 600
#define T_X0 150
#define T_X1 450
#define SPECM 0
#define T_SA 1800
#define T_W0 300
#define T_W1 700
#define WSLEEP 4
#define STR2(x) #x
#define STR(x) STR2(x)
static_assert(NNO >= 8 && PHI >= 1, "W quarters live in the unused groups of a cross source");

#define STAMP(i) do {} while (0)
#define STAMP_FLUSH() do {} while (0)
#define STAMP_DRAIN() do {} while (0)

struct Params {
  const uint8_t* x;
  char* myws;
  const uint8_t* w;
  unsigned rk;
  unsigned wq;
  char* ws[NR];
  const float* sx;
  const float* sw;
  float* out[NR];
  u64 task[2][14][2];
};

__device__ __forceinline__ __amdgpu_buffer_rsrc_t mk_rsrc(const void* p, unsigned bytes) {
  return __builtin_amdgcn_make_buffer_rsrc(const_cast<void*>(p), 0, bytes, 0x00020000);
}
__device__ __forceinline__ unsigned min3u(unsigned a, unsigned b, unsigned c) { return min(min(a, b), c); }
__device__ __forceinline__ unsigned min4(unsigned m, u32x4 v) { return min3u(min3u(m, v[0], v[1]), v[2], v[3]); }
__device__ __forceinline__ u32x4 canon(u32x4 v) {
#pragma unroll
  for (int d = 0; d < 4; ++d) v[d] = v[d] ? v[d] : 0x80808080u;
  return v;
}
__device__ __forceinline__ void st128(u32x4 v, __amdgpu_buffer_rsrc_t r, unsigned voff, unsigned soff) {
  __builtin_amdgcn_raw_buffer_store_b128(v, r, voff, soff, AUX_SYS);
  asm volatile("s_nop 1" :: "v"(v) : "memory");
}
typedef __attribute__((address_space(3))) int lds_int;
__device__ __forceinline__ void lds_st(lds_int* a, int v) { __hip_atomic_store(a, v, __ATOMIC_RELAXED, __HIP_MEMORY_SCOPE_WORKGROUP); }
__device__ __forceinline__ int lds_ld(lds_int* a) { return __hip_atomic_load(a, __ATOMIC_RELAXED, __HIP_MEMORY_SCOPE_WORKGROUP); }
__device__ __forceinline__ unsigned src_off(int s) { return (unsigned)(s >> 6) * GROUP + (unsigned)(s & 63) * 64; }
__device__ __forceinline__ void tr16(const unsigned (&r)[16], u32x4 (&o)[4]) {
#pragma unroll
  for (int d = 0; d < 4; ++d) {
    const unsigned a0 = r[4 * d], a1 = r[4 * d + 1], a2 = r[4 * d + 2], a3 = r[4 * d + 3];
    const unsigned t0 = __builtin_amdgcn_perm(a1, a0, 0x05010400u), t1 = __builtin_amdgcn_perm(a1, a0, 0x07030602u);
    const unsigned t2 = __builtin_amdgcn_perm(a3, a2, 0x05010400u), t3 = __builtin_amdgcn_perm(a3, a2, 0x07030602u);
    o[0][d] = __builtin_amdgcn_perm(t2, t0, 0x05040100u);
    o[1][d] = __builtin_amdgcn_perm(t2, t0, 0x07060302u);
    o[2][d] = __builtin_amdgcn_perm(t3, t1, 0x05040100u);
    o[3][d] = __builtin_amdgcn_perm(t3, t1, 0x07060302u);
  }
}

__global__ void __launch_bounds__(NTHR) ag_gemm_kernel(Params p) {
  extern __shared__ __attribute__((aligned(16))) char smem[];
  const int wid = __builtin_amdgcn_readfirstlane(threadIdx.x >> 6);
  const int lane = threadIdx.x & 63;
  const int bid = blockIdx.x;
  const int xcd = bid & 7, cu = bid >> 3;
  const int drole = xcd < 4 ? 1 + (wid >= 3 ? wid - 3 : wid) % 3 : xcd;
  const u64 t0 = __builtin_amdgcn_s_memrealtime();
  char* myws = p.myws;
  const uint8_t* px = p.x;
  char* pws = p.ws[drole];
  const uint8_t* pw = p.w;
  unsigned rk = p.rk, wq = p.wq;
  asm volatile("" : "+s"(myws), "+s"(px), "+s"(pws), "+s"(rk), "+s"(pw), "+s"(wq));
  const unsigned rank = rk & 7u;
  const __amdgpu_buffer_rsrc_t rc = mk_rsrc(myws + CTRL_OFF, CTRL_BYTES);
  const unsigned cnt = C_CNT + (unsigned)(bid * NWAVE + wid) * 4;
  const __amdgpu_buffer_rsrc_t rxs = mk_rsrc(px, SHARD);
  const int li = lane & 15, lg = lane >> 4;
  const unsigned vsrc = (unsigned)li * KD + (unsigned)lg * 16;
  const unsigned vlin = (unsigned)lane * 16;
  STAMP(0);
  if (__builtin_expect(cu < 4 && xcd < 4, 1)) {
    lds_int* pos = (lds_int*)smem;
    lds_int* done = pos + 192;
    const int pidx = cu * 4 + xcd;
    if (__builtin_expect(wid >= 6, 0)) {
      if (wid == 6) { __builtin_amdgcn_s_barrier(); return; }
      const unsigned craw = __builtin_amdgcn_raw_buffer_load_b32(rc, 0, cnt, AUX_AGENT);
      lds_st(pos + lane, -1); lds_st(pos + lane + 64, -1); lds_st(pos + lane + 128, -1); lds_st(done + lane, lane >= 6 ? 0x7fffffff : 0);
      asm volatile("s_waitcnt lgkmcnt(0)" ::: "memory");
      __builtin_amdgcn_s_barrier();
      const unsigned call = __builtin_amdgcn_readfirstlane(craw) + 1;
      __builtin_amdgcn_raw_buffer_store_b32(call, rc, 0, cnt, AUX_AGENT);
      if (bid == 0) __builtin_amdgcn_raw_buffer_store_b32(0u, rc, (unsigned)(lane & 7) * 128, C_TICK + ((call + 1) & 1) * 1024, AUX_AGENT);
      typedef __attribute__((address_space(1))) unsigned gu32;
      gu32* tick = (gu32*)(myws + CTRL_OFF + C_TICK + (call & 1) * 1024);
      int n = NSTAT;
      bool mine = true;
      for (;;) {
        int d = lds_ld(done + (lane & 7));
        d = min(d, __builtin_amdgcn_update_dpp(0x7fffffff, d, 0x111, 0xf, 0xf, false));
        d = min(d, __builtin_amdgcn_update_dpp(0x7fffffff, d, 0x112, 0xf, 0xf, false));
        d = min(d, __builtin_amdgcn_update_dpp(0x7fffffff, d, 0x114, 0xf, 0xf, false));
        const int lim = min(__builtin_amdgcn_readlane(d, 7) + (mine ? AHEAD1 : AHEAD), NMAX);
        const int k = lim - n;
        if (k <= 0) {
          if (n >= NMAX) { lds_st(pos + n + (lane & 3), NUNIT); break; }
          __builtin_amdgcn_s_sleep(2);
          continue;
        }
        unsigned t = 0;
        if (lane == 0) t = __hip_atomic_fetch_add(tick + (mine ? xcd * 32 : 128), (unsigned)k, __ATOMIC_RELAXED, __HIP_MEMORY_SCOPE_AGENT);
        if (mine) {
          const int t0 = (int)min(__builtin_amdgcn_readfirstlane(t), (unsigned)NXT);
          const int kk = min(k, NXT - t0);
          if (lane < kk) lds_st(pos + n + lane, NSTAT * NIP + (t0 + lane) * 4 + xcd);
          n += kk;
          mine = t0 + k < NXT;
        } else {
          const int t0 = (int)min(__builtin_amdgcn_readfirstlane(t), (unsigned)NDYN);
          const bool last = t0 + k >= NDYN;
          if (lane < (last ? k + 4 : k)) lds_st(pos + n + lane, min(NUNIT - NDYN + t0 + lane, NUNIT));
          if (last) break;
          n += k;
        }
      }
      STAMP(6);
      STAMP_FLUSH();
      return;
    }
    const int h = wid >= 3;
    u32x4 r[2];
    r[0] = __builtin_amdgcn_raw_buffer_load_b128(rxs, vsrc, src_off(2 * pidx + h), 0);
    __builtin_amdgcn_sched_barrier(0);
    const unsigned craw = __builtin_amdgcn_raw_buffer_load_b32(rc, 0, cnt, AUX_AGENT);
    __builtin_amdgcn_sched_barrier(0);
    r[1] = __builtin_amdgcn_raw_buffer_load_b128(rxs, vsrc, src_off(2 * (pidx + NIP) + h), 0);
    __builtin_amdgcn_sched_barrier(0);
    __builtin_amdgcn_s_barrier();
    __builtin_amdgcn_sched_barrier(0);
    const unsigned call = __builtin_amdgcn_readfirstlane(craw) + 1;
    const __amdgpu_buffer_rsrc_t rd = mk_rsrc(pws + (size_t)(call % NSLOT) * SLOT + (size_t)rank * SHARD, SHARD);
    int up[2] = {pidx, pidx + NIP};
#pragma unroll 1
    for (int n0 = 0; up[0] < NUNIT; n0 += 2) {
#pragma unroll
      for (int i = 0; i < 2; ++i) {
        const int n = n0 + i;
        const int u = up[i];
        st128(canon(r[i]), rd, u < NUNIT ? vlin : SHARD, (unsigned)(2 * u + h) * 1024);
        __builtin_amdgcn_sched_barrier(0);
        lds_st(done + wid, n + 1);
        int un = pidx + (n + 2) * NIP;
        if (n + 2 >= NSTAT) {
          for (;;) {
            un = __builtin_amdgcn_readfirstlane(lds_ld(pos + n + 2));
            if (un >= 0) break;
            __builtin_amdgcn_s_sleep(1);
          }
        }
        r[i] = __builtin_amdgcn_raw_buffer_load_b128(rxs, vsrc, src_off(2 * un + h), 0);
        up[i] = un;
        __builtin_amdgcn_sched_barrier(0);
      }
    }
    STAMP(5);
    STAMP_DRAIN();
    STAMP(6);
    __builtin_amdgcn_raw_buffer_store_b32(call, rc, 0, cnt, AUX_AGENT);
    STAMP_FLUSH();
    return;
  }

  const bool cross = xcd >= 4;
  const __amdgpu_buffer_rsrc_t rw = mk_rsrc(pw, KD * NLOC);
  const unsigned vw = (unsigned)lg * 16 * NLOC + (unsigned)li * 4;
  const unsigned craw = __builtin_amdgcn_raw_buffer_load_b32(rc, 0, cnt, AUX_AGENT);
  if (cross && cu >= 4 && cu < 12) {
    const int kb = (cu - 4) * 8 + wid;
    const unsigned so = (unsigned)kb * 64 * NLOC + ((wq >> (4 * (xcd - 4))) & 3u) * 128;
    unsigned raw[2][16];
    u32x4 o[2][4];
#pragma unroll
    for (int h = 0; h < 2; ++h)
#pragma unroll
      for (int i = 0; i < 16; ++i) raw[h][i] = __builtin_amdgcn_raw_buffer_load_b32(rw, vw + (unsigned)(i * NLOC + h * 64), so, 0);
    __builtin_amdgcn_sched_barrier(0);
    __builtin_amdgcn_s_barrier();
    __builtin_amdgcn_sched_barrier(0);
    tr16(raw[0], o[0]);
    tr16(raw[1], o[1]);
    const unsigned call = __builtin_amdgcn_readfirstlane(craw) + 1;
    const __amdgpu_buffer_rsrc_t rd = mk_rsrc(pws + (size_t)(call % NSLOT) * SLOT + (size_t)rank * SHARD, SHARD);
#pragma unroll
    for (int e = 0; e < 8; ++e) {
      if (e > WWIN) asm volatile("s_waitcnt vmcnt(" STR(WWIN) ")" ::: "memory");
      st128(canon(o[e & 1][e >> 1]), rd, vlin + (unsigned)(e >> 1) * 1024, (unsigned)((e & 1) * 64 + kb) * 4096);
      __builtin_amdgcn_sched_barrier(0);
      if (e == 0) STAMP(2);
    }
    STAMP(7);
    __builtin_amdgcn_s_barrier();
  }
  __builtin_amdgcn_sched_barrier(0);

  if (cu < 4) {
    constexpr int NPW = NX * 2;
    const int s5 = cu * 2 + (wid & 1) + (wid >> 1) * 8;
    auto spos = [&](int n) -> int { const int g = n >> 1; return (g < PHI ? NI + g : NNO + g - PHI) * 64 + (n & 1) * 32 + s5; };
    u32x4 r[NPW];
    r[0] = __builtin_amdgcn_raw_buffer_load_b128(rxs, vsrc, src_off(spos(0)), 0);
    r[1] = __builtin_amdgcn_raw_buffer_load_b128(rxs, vsrc, src_off(spos(1)), 0);
    __builtin_amdgcn_sched_barrier(0);
    const unsigned call = __builtin_amdgcn_readfirstlane(craw) + 1;
    const __amdgpu_buffer_rsrc_t rd = mk_rsrc(pws + (size_t)(call % NSLOT) * SLOT + (size_t)rank * SHARD, SHARD);
    st128(canon(r[0]), rd, vlin, (unsigned)spos(0) * 1024);
    STAMP(2);
    __builtin_amdgcn_sched_barrier(0);
    st128(canon(r[1]), rd, vlin, (unsigned)spos(1) * 1024);
    __builtin_amdgcn_sched_barrier(0);
    __builtin_amdgcn_s_barrier();
    __builtin_amdgcn_sched_barrier(0);
#pragma unroll
    for (int n = 2; n < NPW; ++n) r[n] = __builtin_amdgcn_raw_buffer_load_b128(rxs, vsrc, src_off(spos(n)), 0);
#pragma unroll
    for (int n = 2; n < NPW; ++n) r[n] = canon(r[n]);
    STAMP(1);
#pragma unroll
    for (int n = 2; n < NPW; ++n) {
      if (n == 2 || n == 2 * PHI) {
        while (__builtin_amdgcn_s_memrealtime() - t0 < (n == 2 * PHI ? T_P : T_S)) __builtin_amdgcn_s_sleep(2);
        asm volatile("" ::: "memory");
      }
      st128(r[n], rd, vlin, (unsigned)spos(n) * 1024);
      asm volatile("s_waitcnt vmcnt(" STR(SWIN) ")" ::: "memory");
    }
    __builtin_amdgcn_raw_buffer_store_b32(call, rc, 0, cnt, AUX_AGENT);
    STAMP_DRAIN();
    STAMP(5);
    if (cu == 0 && wid == 0) {
      const unsigned fo = C_FLAG + ((rk >> (4 * xcd)) & 7u) * 256;
      for (unsigned n = 1;; ++n) {
        const unsigned f = __builtin_amdgcn_raw_buffer_load_b32(rc, (unsigned)min(lane, 27) * 4, fo, AUX_SYS);
        if (__builtin_amdgcn_ballot_w64(f != call) == 0ull) break;
        if ((n & 63) == 0 && __builtin_amdgcn_s_memrealtime() - t0 > DEADLINE) {
          __builtin_amdgcn_raw_buffer_store_b32(call, rc, 0, 0, AUX_AGENT);
          p.out[0][(size_t)rank * MLOC * NLOC + lane] = __builtin_nanf("");
          break;
        }
        __builtin_amdgcn_s_sleep(1);
        asm volatile("" ::: "memory");
      }
    }
    STAMP(6);
    STAMP_FLUSH();
    return;
  }

  const int c28 = cu - 4;
  const int ci = xcd * 28 + c28;
  const int strm = cross ? c28 >> 1 : ci >> 3;
  const int slab = cross ? c28 & 1 : ci & 7;
  u64 tl = p.task[cross][strm][0], th = p.task[cross][strm][1];
  typedef const __attribute__((address_space(4))) float cf32;
  float sx = *(cf32*)p.sx, sw = *(cf32*)p.sw;
  float* po = p.out[cross ? xcd : 0];
  bool dead = false;
  unsigned call;
  auto expired = [&]() -> bool {
    return __builtin_amdgcn_readfirstlane(__builtin_amdgcn_raw_buffer_load_b32(rc, 0, 0, AUX_AGENT)) == call ||
           __builtin_amdgcn_s_memrealtime() - t0 > DEADLINE;
  };
  auto next = [&]() -> int {
    const int c = (int)(tl & 255u);
    tl = (tl >> 8) | (th << 56); th = (th >> 8) | (63ull << 56);
    if (c == 63) return -1;
    int role = c >> 5;
    if (cross && role >= 4) role = 4 + ((role + xcd) & 3);
    return (int)(((rk >> (4 * role)) & 7u) << 8) | c;
  };
  u32x4 xv[8];
  auto issue_own = [&](int code) {
    const __amdgpu_buffer_rsrc_t r = rxs;
    const unsigned so = (unsigned)(code & 31) * GROUP + (unsigned)wid * 512;
#pragma unroll
    for (int q = 0; q < 8; ++q) xv[q] = __builtin_amdgcn_raw_buffer_load_b128(r, vsrc + (unsigned)q * 64, so, 0);
  };
  int code = next();
  u32x4 wf[8][4];
  if (!cross) {
    unsigned warm = 0;
    {
      const int wi = wid * 28 + c28;
      if (wi < NI * 4)
        warm = __builtin_amdgcn_raw_buffer_load_b32(rxs, (unsigned)(lane >> 4) * KD + (unsigned)((lane >> 1) & 7) * 512 + (unsigned)(lane & 1) * 64,
                                                    (unsigned)wi * 4 * KD + (unsigned)xcd * 128, 0);
    }
    __builtin_amdgcn_sched_barrier(0);
    const unsigned vo = vw + (unsigned)wid * 512 * NLOC + (unsigned)slab * 64;
    unsigned raw[3][16];
#pragma unroll
    for (int u = 0; u < 10; ++u) {
      if (u < 8) {
#pragma unroll
        for (int i = 0; i < 16; ++i) raw[u % 3][i] = __builtin_amdgcn_raw_buffer_load_b32(rw, vo, (unsigned)(u * 64 + i) * NLOC, 0);
      }
      if (u == 8) issue_own(code);
      __builtin_amdgcn_sched_barrier(0);
      if (u >= 2) tr16(raw[(u - 2) % 3], wf[u - 2]);
      __builtin_amdgcn_sched_barrier(0);
    }
    asm volatile("" : "+s"(sx), "+s"(sw), "+s"(po) : "v"(warm));
    call = __builtin_amdgcn_readfirstlane(craw) + 1;
  } else {
    call = __builtin_amdgcn_readfirstlane(craw) + 1;
    const __amdgpu_buffer_rsrc_t rq = mk_rsrc(myws + (size_t)(call % NSLOT) * SLOT + (size_t)((rk >> (4 * xcd)) & 7u) * SHARD +
                                              (size_t)(slab * 64 + wid * 8) * 4096, 32768);
    const unsigned vq = (unsigned)(lane & 7) * 4096 + (unsigned)((lane >> 3) & 3) * 1024 + 1020;
    unsigned got = 0;
    asm volatile("" : "+s"(sx), "+s"(sw), "+s"(po));
    while (__builtin_amdgcn_s_memrealtime() - t0 < T_W0) __builtin_amdgcn_s_sleep(8);
    asm volatile("" ::: "memory");
    issue_own(code);
#pragma unroll 1
    for (unsigned n = 1; got != 15 && !dead; ++n) {
      __builtin_amdgcn_s_sleep(WSLEEP);
      const unsigned hh = __builtin_amdgcn_raw_buffer_load_b32(rq, vq, 0, AUX_SYS);
      const unsigned miss = (unsigned)__builtin_amdgcn_ballot_w64(hh == 0u);
#pragma unroll
      for (int e = 0; e < 4; ++e) {
        if (!((got >> e) & 1) && !((miss >> (8 * e)) & 255u)) {
          got |= 1u << e;
#pragma unroll
          for (int q = 0; q < 8; ++q) wf[q][e] = __builtin_amdgcn_raw_buffer_load_b128(rq, vlin + (unsigned)e * 1024, (unsigned)q * 4096, AUX_SYS);
        }
      }
      if ((n & 63) == 0 && expired()) dead = true;
      asm volatile("" ::: "memory");
    }
    STAMP(3);
    for (unsigned n = 1;; ++n) {
      unsigned m = 0xffffffffu;
#pragma unroll
      for (int q = 0; q < 8; ++q)
#pragma unroll
        for (int e = 0; e < 4; ++e) m = min4(m, wf[q][e]);
      if (__builtin_expect(__builtin_amdgcn_ballot_w64(m == 0u) == 0ull || dead, 1)) break;
      if ((n & 63) == 0 && expired()) dead = true;
#pragma unroll
      for (int q = 0; q < 8; ++q)
#pragma unroll
        for (int e = 0; e < 4; ++e) wf[q][e] = __builtin_amdgcn_raw_buffer_load_b128(rq, vlin + (unsigned)e * 1024, (unsigned)q * 4096, AUX_SYS);
      asm volatile("" ::: "memory");
    }
  }
  STAMP(1);
  const float scale = sx * sw;
  const __amdgpu_buffer_rsrc_t rs = mk_rsrc(myws + (size_t)(call % NSLOT) * SLOT, SLOT);
  const __amdgpu_buffer_rsrc_t rz = mk_rsrc(myws + (size_t)((call + 2) % NSLOT) * SLOT, SLOT);
  const __amdgpu_buffer_rsrc_t ro = mk_rsrc(po, NR * MLOC * NLOC * 4 - (cross ? 3 * 512 : 0));
  const unsigned vh = (unsigned)(lane & 7) * 1024 + 1020;
  const int orow = 2 * wid + (lg & 1);
  const unsigned vout = lane < 32 ? (unsigned)orow * (NLOC * 4) + (unsigned)slab * 256 + (unsigned)li * 16 : 0x80000000u;
  char* const lw = smem + wid * 4096 + li * 256;
  const char* const lr = smem + orow * 256 + ((li ^ orow) * 16) + (lane >> 5) * 16384;
  const char* const xslot = myws + (size_t)(call % NSLOT) * SLOT;
  auto expected = [&](int code) -> bool {
    const int g = code & 31;
    const int when = (code & 0xe0) == 0 ? 0 : (code & 0x80) == 0 ? 213 - SPECM + 113 * g : g == NI ? 400 : g > NI ? T_SA : T_P + 300;
    return code >= 0 && (int)(__builtin_amdgcn_s_memrealtime() - t0) >= when;
  };
  __builtin_amdgcn_raw_buffer_store_b32(call, rc, 0, cnt, AUX_AGENT);
  __builtin_amdgcn_sched_barrier(0);
  int buf = 0;
#pragma unroll 1
  do {
    const int ncode = next();
    if (code & 0xe0) {
      unsigned m = 0xffffffffu;
#pragma unroll
      for (int q = 0; q < 8; ++q) m = min4(m, xv[q]);
      if (__builtin_expect(__builtin_amdgcn_ballot_w64(m == 0u) != 0ull && !dead, 0)) {
        const int s = (code & 31) * 64 + wid * 8;
        const unsigned so = (unsigned)(code >> 8) * SHARD + (unsigned)s * 1024;
        const unsigned ho = (code & 0x80) == 0 && s >= EARLY0 ? so - EARLY * 8192 : so;
        bool poll = true;
        for (unsigned n = 1;; ++n) {
          if (poll) {
            const unsigned hh = __builtin_amdgcn_raw_buffer_load_b32(rs, vh, ho, AUX_SYS);
            poll = __builtin_amdgcn_ballot_w64(hh == 0u) != 0ull;
            if (poll) __builtin_amdgcn_s_sleep(HSLEEP);
          }
          if (!poll) {
#pragma unroll
            for (int q = 0; q < 8; ++q) xv[q] = __builtin_amdgcn_raw_buffer_load_b128(rs, vlin + (unsigned)(q & 3) * 1024, so + (unsigned)(q & 4) * 1024, AUX_SYS);
            m = 0xffffffffu;
#pragma unroll
            for (int q = 0; q < 8; ++q) m = min4(m, xv[q]);
            if (__builtin_amdgcn_ballot_w64(m == 0u) == 0ull) break;
            if (n > 64) __builtin_amdgcn_s_sleep(16);
          }
          if ((n & 63) == 0 && expired()) { dead = true; break; }
          asm volatile("" ::: "memory");
        }
        if (dead) __builtin_amdgcn_raw_buffer_store_b32(call, rc, 0, 0, AUX_AGENT);
      }
    }
    const bool nown = (ncode & 0xe0) == 0;
    __amdgpu_buffer_rsrc_t rn;
    const unsigned nstep = nown ? 64 : 1024;
    const unsigned nso = (unsigned)(ncode & 31) * GROUP + (unsigned)wid * 8 * nstep;
    const unsigned nvo = nown ? vsrc : vlin;
    f32x4 acc[4];
#pragma unroll
    for (int e = 0; e < 4; ++e) acc[e] = f32x4{0.f, 0.f, 0.f, 0.f};
#pragma unroll
    for (int ks = 0; ks < 4; ++ks) {
      const i32x8 b = {(int)xv[2 * ks][0], (int)xv[2 * ks][1], (int)xv[2 * ks][2], (int)xv[2 * ks][3],
                       (int)xv[2 * ks + 1][0], (int)xv[2 * ks + 1][1], (int)xv[2 * ks + 1][2], (int)xv[2 * ks + 1][3]};
#pragma unroll
      for (int e = 0; e < 4; ++e) {
        const u32x4 a0 = wf[2 * ks][e], a1 = wf[2 * ks + 1][e];
        const i32x8 a = {(int)a0[0], (int)a0[1], (int)a0[2], (int)a0[3], (int)a1[0], (int)a1[1], (int)a1[2], (int)a1[3]};
        acc[e] = __builtin_amdgcn_mfma_scale_f32_16x16x128_f8f6f4(a, b, acc[e],   1,   0, 0, 0, 0, 0);
      }
      __builtin_amdgcn_sched_barrier(0);
      if (ks == 0 && !cross) {
        const unsigned zo = (code & 0xe0) ? (unsigned)(code >> 8) * SHARD + (unsigned)(code & 31) * GROUP
                                          : ((rk >> (16 + ((code >> 1) & 12))) & 7u) * SHARD + (unsigned)(code & 7) * GROUP;
        st128(u32x4{0u, 0u, 0u, 0u}, rz, vlin, zo + (unsigned)(slab * 8 + wid) * 1024);
      }
      if (ks == 0) rn = mk_rsrc(nown ? (const char*)px : xslot + (size_t)((ncode >> 8) & 7) * SHARD, expected(ncode) ? SHARD : 0);
      xv[2 * ks] = __builtin_amdgcn_raw_buffer_load_b128(rn, nvo, nso + (unsigned)(2 * ks) * nstep, AUX_SYS);
      xv[2 * ks + 1] = __builtin_amdgcn_raw_buffer_load_b128(rn, nvo, nso + (unsigned)(2 * ks + 1) * nstep, AUX_SYS);
      __builtin_amdgcn_sched_barrier(0);
    }
#pragma unroll
    for (int r = 0; r < 4; ++r) {
      f32x4 v = {acc[0][r], acc[1][r], acc[2][r], acc[3][r]};
      if (dead) v[0] = __builtin_nanf("");
      *(f32x4*)(lw + buf + (((lg * 4 + r) ^ li) * 16)) = v;
    }
    asm volatile("s_waitcnt lgkmcnt(0)" ::: "memory");
    __builtin_amdgcn_s_barrier();
    asm volatile("" ::: "memory");
    f32x4 pr[4], o;
#pragma unroll
    for (int v = 0; v < 4; ++v) pr[v] = *(const f32x4*)(lr + buf + v * 4096);
    const f32x4 sum = (pr[0] + pr[1]) + (pr[2] + pr[3]);
#pragma unroll
    for (int r = 0; r < 4; ++r) {
      const unsigned su = __float_as_uint(sum[r]);
      const u32x2 sw2 = __builtin_amdgcn_permlane32_swap(su, su, false, false);
      const unsigned s0 = sw2[0], s1 = sw2[1];
      const float y = (__uint_as_float(s0) + __uint_as_float(s1)) * scale;
      o[r] = y * __builtin_amdgcn_rcpf(1.f + __expf(-__builtin_amdgcn_fmed3f(y, -60.f, 60.f)));
    }
    st128(__builtin_bit_cast(u32x4, o), ro, vout, ((unsigned)(code >> 8) * MLOC + (unsigned)(code & 31) * 16) * (NLOC * 4));
    __builtin_amdgcn_sched_barrier(0);
    code = ncode;
    buf ^= 32768;
  } while (code >= 0);
  STAMP(5);
  if (cross) {
    asm volatile("s_waitcnt vmcnt(0)" ::: "memory");
    __builtin_amdgcn_s_barrier();
    if (wid == 0) __builtin_amdgcn_raw_buffer_store_b32(call, mk_rsrc(pws + CTRL_OFF, CTRL_BYTES), 0, C_FLAG + rank * 256 + (unsigned)c28 * 4, AUX_SYS);
  }
  STAMP_DRAIN();
  STAMP(6);
  STAMP_FLUSH();
}

#define DUR0 1.2f
#define DUR1 2.0f
#define BEG0 8.f
#define BEG2 13.9f
#define BEG1 12.f
void build_tasks(Params& p) {
  for (int t = 0; t < 2; ++t) {
    float when[168];
    uint8_t what[168];
    int n = 0;
    auto add = [&](float w, int role, int g) {
      int i = n++;
      for (; i > 0 && when[i - 1] > w; --i) { when[i] = when[i - 1]; what[i] = what[i - 1]; }
      when[i] = w; what[i] = (uint8_t)(role << 5 | g);
    };
    for (int g = 0; g < (t ? NNO : 32); ++g) add(0.f, 0, g);
    for (int g = 0; g < (t ? NNO : NI); ++g) for (int r = 1; r < 4; ++r) add(1.f + 1.13f * (g + 1), r, g);
    for (int i = 0; i < (t ? PHI : NX); ++i)
      for (int r = t ? 5 : 4; r < 8; ++r)
        add(i == 0 ? 3.f : i < PHI ? T_SA * 0.01f : T_P * 0.01f + 1.5f * (i - PHI + 1), r, i < PHI ? NI + i : NNO + i - PHI);
    float busy[14];
    int len[14];
    uint8_t lst[14][16];
    for (int j = 0; j < 14; ++j) { busy[j] = t ? (j < 4 ? BEG2 : BEG1) : BEG0; len[j] = 0; }
    for (int i = 0; i < n; ++i) {
      int b = -1;
      for (int j = 0; j < 14; ++j) if (len[j] < 15 && (b < 0 || busy[j] < busy[b])) b = j;
      busy[b] = (busy[b] > when[i] ? busy[b] : when[i]) + (t ? DUR1 : DUR0);
      lst[b][len[b]++] = what[i];
    }
    for (int j = 0; j < 14; ++j) {
      u64 v[2] = {0, 0};
      const int o = 0;
      for (int i = 0; i < 16; ++i) v[i >> 3] |= (u64)(i >= o && i - o < len[j] ? lst[j][i - o] : 63) << (8 * (i & 7));
      p.task[t][j][0] = v[0]; p.task[t][j][1] = v[1];
    }
  }
}

}

extern "C" void kernel_launch(int rank, int nranks, ncclComm_t comm,
                              hipStream_t stream,
                              void* const* d_in, const int* in_sizes, int n_in,
                              void* d_out, int out_size,
                              const int* in_dtypes, int out_dtype,
                              void* const* d_out_all,
                              void* const* d_ws_all, size_t ws_bytes) {
  (void)comm; (void)in_dtypes; (void)out_dtype;
  if (nranks != NR || (unsigned)rank >= (unsigned)NR || n_in < 4 || in_sizes[0] != MLOC * KD || in_sizes[1] != KD * NLOC ||
      out_size != NR * MLOC * NLOC || ws_bytes < CTRL_OFF + CTRL_BYTES) {
    fprintf(stderr, "kernel_launch: unexpected problem shape\n");
    return;
  }
  static bool attr_done[NR];
  if (!attr_done[rank]) {
    HIP_CHECK(hipFuncSetAttribute((const void*)ag_gemm_kernel, hipFuncAttributeMaxDynamicSharedMemorySize, LDS_BYTES));
    attr_done[rank] = true;
  }
  Params p;
  p.x = (const uint8_t*)d_in[0];
  p.w = (const uint8_t*)d_in[1];
  p.sx = (const float*)d_in[2];
  p.sw = (const float*)d_in[3];
  p.myws = (char*)d_ws_all[rank];
  const int quad = 0x4d;
  auto qidx = [&](int r) { int n = 0; for (int q = 0; q < r; ++q) n += ((quad >> q) & 1) == ((quad >> r) & 1); return n; };
  int role[NR], na = 1, nb = 4;
  role[0] = rank;
  for (int j = 1; j < NR; ++j) {
    const int q = (rank + j) & 7;
    if (((quad >> q) & 1) == ((quad >> rank) & 1)) role[na++] = q; else role[nb++] = q;
  }
  p.rk = 0; p.wq = 0;
  for (int i = 0; i < NR; ++i) {
    p.ws[i] = (char*)d_ws_all[role[i]];
    p.out[i] = i == 0 ? (float*)d_out : (float*)d_out_all[role[i]] + (i >= 4 ? qidx(rank) * 128 : 0);
    p.rk |= (unsigned)role[i] << (4 * i);
    if (i >= 4) p.wq |= (unsigned)qidx(role[i]) << (4 * (i - 4));
  }
  build_tasks(p);
  hipLaunchKernelGGL(ag_gemm_kernel, dim3(NBLK), dim3(NTHR), LDS_BYTES, stream, p);
  HIP_CHECK(hipGetLastError());
}
```

```cpp
#include <hip/hip_runtime.h>
#include <rccl/rccl.h>
#include <cstdio>
#include <cstdint>

#define HIP_CHECK(expr) do { hipError_t _e = (expr); if (_e != hipSuccess) { \
    fprintf(stderr, "HIP error %s at %s:%d\n", hipGetErrorString(_e), __FILE__, __LINE__); \
    return; } } while (0)

namespace {

typedef unsigned u32x4 __attribute__((ext_vector_type(4)));
typedef unsigned u32x2 __attribute__((ext_vector_type(2)));
typedef int i32x8 __attribute__((ext_vector_type(8)));
typedef float f32x4 __attribute__((ext_vector_type(4)));
typedef float f32x2 __attribute__((ext_vector_type(2)));
typedef unsigned long long u64;

constexpr int NR = 8;
constexpr int KD = 4096;
constexpr int MLOC = 512;
constexpr int NLOC = 512;
constexpr unsigned SHARD = MLOC * KD;
constexpr unsigned GROUP = 16 * KD;
constexpr unsigned SLOT = NR * SHARD;
constexpr int NSLOT = 3;
constexpr size_t CTRL_OFF = (size_t)NSLOT * SLOT;
constexpr unsigned CTRL_BYTES = 64 * 1024;
constexpr unsigned C_CNT = 256, C_ROLE = 4096, C_FLAG = 16384, C_TICK = 32768;
constexpr int NBLK = 256;
constexpr int NWAVE = 8;
constexpr int NTHR = NWAVE * 64;
constexpr unsigned LDS_BYTES = 96 * 1024;
constexpr u64 DEADLINE = 50000000ull;
constexpr int AUX_SYS = 17;
constexpr int AUX_AGENT = 16;

#define PHI 4
#define PSI 2
constexpr int NNO = 32 - PHI - PSI;
constexpr int NI = NNO + PSI;
constexpr int NX = PHI + PSI;
constexpr int NIP = 16;
constexpr int NUNIT = NI * 32;
#define NSTAT 6
#define NDYNP 10
constexpr int NDYN = NDYNP * NIP;
constexpr int NXT = (NUNIT - NSTAT * NIP - NDYN) / 4;
constexpr int NMAX = 120;
#define AHEAD 6
#define AHEAD1 8
#define EARLY 4
#define EARLY0 ((NUNIT - NDYN) * 2 + 128)
#define HSLEEP 2
#define T_S 1050
#define T_P 1200
#define WWIN 1
#define W1WIN 1
#define SWIN 1
#define T_W2 600
#define T_L 600
#define T_X0 150
#define T_X1 450
#define SPECM 0
#define T_SA 1800
#define T_W0 300
#define T_W1 700
#define WSLEEP 4
#define STR2(x) #x
#define STR(x) STR2(x)
static_assert(NNO >= 8 && PHI >= 1, "W quarters live in the unused groups of a cross source");

#define STAMP(i) do {} while (0)
#define STAMP_FLUSH() do {} while (0)
#define STAMP_DRAIN() do {} while (0)

struct Params {
  const uint8_t* x;
  char* myws;
  const uint8_t* w;
  unsigned rk;
  unsigned wq;
  char* ws[NR];
  const float* sx;
  const float* sw;
  float* out[NR];
  u64 task[2][14][2];
};

__device__ __forceinline__ __amdgpu_buffer_rsrc_t mk_rsrc(const void* p, unsigned bytes) {
  return __builtin_amdgcn_make_buffer_rsrc(const_cast<void*>(p), 0, bytes, 0x00020000);
}
__device__ __forceinline__ unsigned min3u(unsigned a, unsigned b, unsigned c) { return min(min(a, b), c); }
__device__ __forceinline__ unsigned min4(unsigned m, u32x4 v) { return min3u(min3u(m, v[0], v[1]), v[2], v[3]); }
__device__ __forceinline__ u32x4 canon(u32x4 v) {
#pragma unroll
  for (int d = 0; d < 4; ++d) v[d] = v[d] ? v[d] : 0x80808080u;
  return v;
}
__device__ __forceinline__ void st128(u32x4 v, __amdgpu_buffer_rsrc_t r, unsigned voff, unsigned soff) {
  __builtin_amdgcn_raw_buffer_store_b128(v, r, voff, soff, AUX_SYS);
  asm volatile("s_nop 1" :: "v"(v) : "memory");
}
typedef __attribute__((address_space(3))) int lds_int;
typedef __attribute__((address_space(1))) unsigned gu32;
__device__ __forceinline__ void lds_st(lds_int* a, int v) { __hip_atomic_store(a, v, __ATOMIC_RELAXED, __HIP_MEMORY_SCOPE_WORKGROUP); }
__device__ __forceinline__ int lds_ld(lds_int* a) { return __hip_atomic_load(a, __ATOMIC_RELAXED, __HIP_MEMORY_SCOPE_WORKGROUP); }
__device__ __forceinline__ unsigned src_off(int s) { return (unsigned)(s >> 6) * GROUP + (unsigned)(s & 63) * 64; }
__device__ __forceinline__ void tr16(const unsigned (&r)[16], u32x4 (&o)[4]) {
#pragma unroll
  for (int d = 0; d < 4; ++d) {
    const unsigned a0 = r[4 * d], a1 = r[4 * d + 1], a2 = r[4 * d + 2], a3 = r[4 * d + 3];
    const unsigned t0 = __builtin_amdgcn_perm(a1, a0, 0x05010400u), t1 = __builtin_amdgcn_perm(a1, a0, 0x07030602u);
    const unsigned t2 = __builtin_amdgcn_perm(a3, a2, 0x05010400u), t3 = __builtin_amdgcn_perm(a3, a2, 0x07030602u);
    o[0][d] = __builtin_amdgcn_perm(t2, t0, 0x05040100u);
    o[1][d] = __builtin_amdgcn_perm(t2, t0, 0x07060302u);
    o[2][d] = __builtin_amdgcn_perm(t3, t1, 0x05040100u);
    o[3][d] = __builtin_amdgcn_perm(t3, t1, 0x07060302u);
  }
}

__global__ void __launch_bounds__(NTHR) ag_gemm_kernel(Params p) {
  extern __shared__ __attribute__((aligned(16))) char smem[];
  const int wid = __builtin_amdgcn_readfirstlane(threadIdx.x >> 6);
  const int lane = threadIdx.x & 63;
  const int bid = blockIdx.x;
  const int cu = bid >> 3;
  const u64 t0 = __builtin_amdgcn_s_memrealtime();
  int xcd = __builtin_amdgcn_s_getreg(20 | (3 << 11)) & 7;
  auto peer = [&]() -> int { return xcd < 4 ? 1 + (wid >= 3 ? wid - 3 : wid) % 3 : xcd; };
  char* myws = p.myws;
  const uint8_t* px = p.x;
  char* pws = p.ws[peer()];
  const uint8_t* pw = p.w;
  unsigned rk = p.rk, wq = p.wq;
  asm volatile("" : "+s"(myws), "+s"(px), "+s"(pws), "+s"(rk), "+s"(pw), "+s"(wq));
  const unsigned rank = rk & 7u;
  const __amdgpu_buffer_rsrc_t rc = mk_rsrc(myws + CTRL_OFF, CTRL_BYTES);
  const __amdgpu_buffer_rsrc_t rxs = mk_rsrc(px, SHARD);
  const __amdgpu_buffer_rsrc_t rw = mk_rsrc(pw, KD * NLOC);
  const int li = lane & 15, lg = lane >> 4;
  const unsigned vsrc = (unsigned)li * KD + (unsigned)lg * 16;
  const unsigned vlin = (unsigned)lane * 16;
  const unsigned vw = (unsigned)lg * 16 * NLOC + (unsigned)li * 4;
  STAMP(0);
  gu32* const roles = (gu32*)(myws + CTRL_OFF + C_ROLE) + cu * 32 + (lane & 7);
  lds_int* const verdict = (lds_int*)(smem + 65536);
  unsigned took = 0;
  auto ask = [&]() { if (wid == 7 && lane < 8) took = __hip_atomic_fetch_add(roles, lane == xcd ? 1u : 0u, __ATOMIC_RELAXED, __HIP_MEMORY_SCOPE_AGENT); };
  auto resolve = [&]() {
    if (wid != 7) return;
    int r = xcd;
    unsigned c;
    for (;;) {
      c = __builtin_amdgcn_readlane(took, r);
      const unsigned behind = (unsigned)__builtin_amdgcn_ballot_w64((int)(took - c) < 0) & 255u;
      if (__builtin_expect(behind == 0, 1)) break;
      const int rn = __builtin_ctz(behind);
      if (lane < 8) took = __hip_atomic_fetch_add(roles, lane == rn ? 1u : lane == r ? ~0u : 0u, __ATOMIC_RELAXED, __HIP_MEMORY_SCOPE_AGENT);
      r = rn;
    }
    lds_st(verdict + (lane & 1), lane & 1 ? r : (int)c + 1);
  };
  auto meet = [&]() -> int {
    asm volatile("s_waitcnt lgkmcnt(0)" ::: "memory");
    __builtin_amdgcn_s_barrier();
    asm volatile("" ::: "memory");
    return lds_ld(verdict + (lane & 1));
  };
  if (__builtin_expect(cu < 4, 1)) {
    const unsigned cnt = C_CNT + (unsigned)(bid * NWAVE + wid) * 4;
    const int h = wid >= 3;
    const int s5 = cu * 2 + (wid & 1) + (wid >> 1) * 8;
    lds_int* const pos = (lds_int*)smem;
    lds_int* const done = pos + 192;
    static_assert(NIP == 16, "the first two pieces of a wave are 32 apart");
    auto spiece = [&]() -> int { return xcd < 4 ? 2 * (cu * 4 + xcd) + h : NI * 64 + s5; };
    u32x4 r0[2];
    r0[0] = __builtin_amdgcn_raw_buffer_load_b128(rxs, vsrc, src_off(spiece()), 0);
    __builtin_amdgcn_sched_barrier(0);
    const unsigned craw = __builtin_amdgcn_raw_buffer_load_b32(rc, 0, cnt, AUX_AGENT);
    __builtin_amdgcn_sched_barrier(0);
    r0[1] = __builtin_amdgcn_raw_buffer_load_b128(rxs, vsrc, src_off(spiece() + 32), 0);
    __builtin_amdgcn_sched_barrier(0);
    unsigned call;
#pragma unroll 1
    for (bool again = false;; again = true) {
      if (xcd < 4) {
        if (wid == 6) { lds_st(pos + lane, -1); lds_st(pos + lane + 64, -1); lds_st(pos + lane + 128, -1); lds_st(done + lane, lane >= 6 ? 0x7fffffff : 0); }
        asm volatile("s_waitcnt lgkmcnt(0)" ::: "memory");
        __builtin_amdgcn_s_barrier();
        __builtin_amdgcn_sched_barrier(0);
      }
      call = __builtin_amdgcn_readfirstlane(craw) + 1;
      const __amdgpu_buffer_rsrc_t rd = mk_rsrc(pws + (size_t)(call % NSLOT) * SLOT + (size_t)rank * SHARD, SHARD);
      if (__builtin_expect(xcd < 4, 1)) {
        const int pidx = cu * 4 + xcd;
        if (__builtin_expect(wid < 6, 1)) {
          u32x4 r[2] = {r0[0], r0[1]};
          int up[2] = {pidx, pidx + NIP};
#pragma unroll 1
          for (int n0 = 0; up[0] < NUNIT; n0 += 2) {
#pragma unroll
            for (int i = 0; i < 2; ++i) {
              const int n = n0 + i;
              const int u = up[i];
              st128(canon(r[i]), rd, u < NUNIT ? vlin : SHARD, (unsigned)(2 * u + h) * 1024);
              __builtin_amdgcn_sched_barrier(0);
              lds_st(done + wid, n + 1);
              int un = pidx + (n + 2) * NIP;
              if (n + 2 >= NSTAT) {
                for (;;) {
                  un = __builtin_amdgcn_readfirstlane(lds_ld(pos + n + 2));
                  if (un >= 0) break;
                  __builtin_amdgcn_s_sleep(1);
                }
              }
              r[i] = __builtin_amdgcn_raw_buffer_load_b128(rxs, vsrc, src_off(2 * un + h), 0);
              up[i] = un;
              __builtin_amdgcn_sched_barrier(0);
            }
          }
          STAMP(5);
          STAMP_DRAIN();
        } else if (wid == 7) {
          if (!again) ask();
          if (pidx == 0) __builtin_amdgcn_raw_buffer_store_b32(0u, rc, (unsigned)(lane & 7) * 128, C_TICK + ((call + 1) & 1) * 1024, AUX_AGENT);
          gu32* tick = (gu32*)(myws + CTRL_OFF + C_TICK + (call & 1) * 1024);
          int n = NSTAT;
          bool mine = true;
          for (;;) {
            int d = lds_ld(done + (lane & 7));
            d = min(d, __builtin_amdgcn_update_dpp(0x7fffffff, d, 0x111, 0xf, 0xf, false));
            d = min(d, __builtin_amdgcn_update_dpp(0x7fffffff, d, 0x112, 0xf, 0xf, false));
            d = min(d, __builtin_amdgcn_update_dpp(0x7fffffff, d, 0x114, 0xf, 0xf, false));
            const int lim = min(__builtin_amdgcn_readlane(d, 7) + (mine ? AHEAD1 : AHEAD), NMAX);
            const int k = lim - n;
            if (k <= 0) {
              if (n >= NMAX) { lds_st(pos + n + (lane & 3), NUNIT); break; }
              __builtin_amdgcn_s_sleep(2);
              continue;
            }
            unsigned t = 0;
            if (lane == 0) t = __hip_atomic_fetch_add(tick + (mine ? xcd * 32 : 128), (unsigned)k, __ATOMIC_RELAXED, __HIP_MEMORY_SCOPE_AGENT);
            if (mine) {
              const int t0 = (int)min(__builtin_amdgcn_readfirstlane(t), (unsigned)NXT);
              const int kk = min(k, NXT - t0);
              if (lane < kk) lds_st(pos + n + lane, NSTAT * NIP + (t0 + lane) * 4 + xcd);
              n += kk;
              mine = t0 + k < NXT;
            } else {
              const int t0 = (int)min(__builtin_amdgcn_readfirstlane(t), (unsigned)NDYN);
              const bool last = t0 + k >= NDYN;
              if (lane < (last ? k + 4 : k)) lds_st(pos + n + lane, min(NUNIT - NDYN + t0 + lane, NUNIT));
              if (last) break;
              n += k;
            }
          }
        }
        STAMP(6);
      } else {
        constexpr int NPW = NX * 2;
        auto spos = [&](int n) -> int { const int g = n >> 1; return (g < PHI ? NI + g : NNO + g - PHI) * 64 + (n & 1) * 32 + s5; };
        u32x4 r[NPW];
        st128(canon(r0[0]), rd, vlin, (unsigned)spos(0) * 1024);
        STAMP(2);
        __builtin_amdgcn_sched_barrier(0);
        st128(canon(r0[1]), rd, vlin, (unsigned)spos(1) * 1024);
        __builtin_amdgcn_sched_barrier(0);
        __builtin_amdgcn_s_barrier();
        __builtin_amdgcn_sched_barrier(0);
        if (!again) ask();
#pragma unroll
        for (int n = 2; n < NPW; ++n) r[n] = __builtin_amdgcn_raw_buffer_load_b128(rxs, vsrc, src_off(spos(n)), 0);
#pragma unroll
        for (int n = 2; n < NPW; ++n) r[n] = canon(r[n]);
        STAMP(1);
#pragma unroll
        for (int n = 2; n < NPW; ++n) {
          if (n == 2 || n == 2 * PHI) {
            while (__builtin_amdgcn_s_memrealtime() - t0 < (n == 2 * PHI ? T_P : T_S)) __builtin_amdgcn_s_sleep(2);
            asm volatile("" ::: "memory");
          }
          st128(r[n], rd, vlin, (unsigned)spos(n) * 1024);
          asm volatile("s_waitcnt vmcnt(" STR(SWIN) ")" ::: "memory");
        }
        STAMP_DRAIN();
        STAMP(5);
      }
      if (again) break;
      resolve();
      const int role = __builtin_amdgcn_readlane(meet(), 1);
      if (__builtin_expect(role == xcd, 1)) break;
      xcd = role;
      pws = p.ws[peer()];
      r0[0] = __builtin_amdgcn_raw_buffer_load_b128(rxs, vsrc, src_off(spiece()), 0);
      r0[1] = __builtin_amdgcn_raw_buffer_load_b128(rxs, vsrc, src_off(spiece() + 32), 0);
    }
    __builtin_amdgcn_raw_buffer_store_b32(call, rc, 0, cnt, AUX_AGENT);
    if (cu == 0 && wid == 0 && xcd >= 4) {
      const unsigned fo = C_FLAG + ((rk >> (4 * xcd)) & 7u) * 256;
      for (unsigned n = 1;; ++n) {
        const unsigned f = __builtin_amdgcn_raw_buffer_load_b32(rc, (unsigned)min(lane, 27) * 4, fo, AUX_SYS);
        if (__builtin_amdgcn_ballot_w64(f != call) == 0ull) break;
        if ((n & 63) == 0 && __builtin_amdgcn_s_memrealtime() - t0 > DEADLINE) {
          __builtin_amdgcn_raw_buffer_store_b32(call, rc, 0, 0, AUX_AGENT);
          p.out[0][(size_t)rank * MLOC * NLOC + lane] = __builtin_nanf("");
          break;
        }
        __builtin_amdgcn_s_sleep(1);
        asm volatile("" ::: "memory");
      }
    }
    STAMP_FLUSH();
    return;
  }

  ask();
  __builtin_amdgcn_sched_barrier(0);
  unsigned warm = 0;
  u32x4 o[2][4];
  auto first_loads = [&](bool first) {
    if (xcd < 4) {
      const int wi = wid * 28 + cu - 4;
      if (wi < NI * 4)
        warm = __builtin_amdgcn_raw_buffer_load_b32(rxs, (unsigned)(lane >> 4) * KD + (unsigned)((lane >> 1) & 7) * 512 + (unsigned)(lane & 1) * 64,
                                                    (unsigned)wi * 4 * KD + (unsigned)xcd * 128, 0);
      if (first) resolve();
    } else if (cu < 12) {
      const unsigned so = (unsigned)((cu - 4) * 8 + wid) * 64 * NLOC + ((wq >> (4 * (xcd - 4))) & 3u) * 128;
      unsigned raw[2][16];
#pragma unroll
      for (int h = 0; h < 2; ++h)
#pragma unroll
        for (int i = 0; i < 16; ++i) raw[h][i] = __builtin_amdgcn_raw_buffer_load_b32(rw, vw + (unsigned)(i * NLOC + h * 64), so, 0);
      __builtin_amdgcn_sched_barrier(0);
      if (first) resolve();
      tr16(raw[0], o[0]);
      tr16(raw[1], o[1]);
#pragma unroll
      for (int e = 0; e < 8; ++e) o[e & 1][e >> 1] = canon(o[e & 1][e >> 1]);
    } else if (first) {
      resolve();
    }
    __builtin_amdgcn_sched_barrier(0);
  };
  first_loads(true);
  const int vd = meet();
  const unsigned call = __builtin_amdgcn_readlane(vd, 0);
  if (__builtin_expect(__builtin_amdgcn_readlane(vd, 1) != xcd, 0)) {
    xcd = __builtin_amdgcn_readlane(vd, 1);
    pws = p.ws[peer()];
    first_loads(false);
  }
  const bool cross = xcd >= 4;
  if (cross && cu < 12) {
    const __amdgpu_buffer_rsrc_t rd = mk_rsrc(pws + (size_t)(call % NSLOT) * SLOT + (size_t)rank * SHARD, SHARD);
    const int kb = (cu - 4) * 8 + wid;
#pragma unroll
    for (int e = 0; e < 8; ++e) {
      if (e > WWIN) asm volatile("s_waitcnt vmcnt(" STR(WWIN) ")" ::: "memory");
      st128(o[e & 1][e >> 1], rd, vlin + (unsigned)(e >> 1) * 1024, (unsigned)((e & 1) * 64 + kb) * 4096);
      __builtin_amdgcn_sched_barrier(0);
      if (e == 0) STAMP(2);
    }
    STAMP(7);
    __builtin_amdgcn_s_barrier();
  }
  __builtin_amdgcn_sched_barrier(0);

  const int c28 = cu - 4;
  const int ci = xcd * 28 + c28;
  const int strm = cross ? c28 >> 1 : ci >> 3;
  const int slab = cross ? c28 & 1 : ci & 7;
  u64 tl = p.task[cross][strm][0], th = p.task[cross][strm][1];
  typedef const __attribute__((address_space(4))) float cf32;
  float sx = *(cf32*)p.sx, sw = *(cf32*)p.sw;
  float* po = p.out[cross ? xcd : 0];
  bool dead = false;
  auto expired = [&]() -> bool {
    return __builtin_amdgcn_readfirstlane(__builtin_amdgcn_raw_buffer_load_b32(rc, 0, 0, AUX_AGENT)) == call ||
           __builtin_amdgcn_s_memrealtime() - t0 > DEADLINE;
  };
  auto next = [&]() -> int {
    const int c = (int)(tl & 255u);
    tl = (tl >> 8) | (th << 56); th = (th >> 8) | (63ull << 56);
    if (c == 63) return -1;
    int role = c >> 5;
    if (cross && role >= 4) role = 4 + ((role + xcd) & 3);
    return (int)(((rk >> (4 * role)) & 7u) << 8) | c;
  };
  u32x4 xv[8];
  auto issue_own = [&](int code) {
    const __amdgpu_buffer_rsrc_t r = rxs;
    const unsigned so = (unsigned)(code & 31) * GROUP + (unsigned)wid * 512;
#pragma unroll
    for (int q = 0; q < 8; ++q) xv[q] = __builtin_amdgcn_raw_buffer_load_b128(r, vsrc + (unsigned)q * 64, so, 0);
  };
  int code = next();
  u32x4 wf[8][4];
  if (!cross) {
    const unsigned vo = vw + (unsigned)wid * 512 * NLOC + (unsigned)slab * 64;
    unsigned raw[3][16];
#pragma unroll
    for (int u = 0; u < 10; ++u) {
      if (u < 8) {
#pragma unroll
        for (int i = 0; i < 16; ++i) raw[u % 3][i] = __builtin_amdgcn_raw_buffer_load_b32(rw, vo, (unsigned)(u * 64 + i) * NLOC, 0);
      }
      if (u == 8) issue_own(code);
      __builtin_amdgcn_sched_barrier(0);
      if (u >= 2) tr16(raw[(u - 2) % 3], wf[u - 2]);
      __builtin_amdgcn_sched_barrier(0);
    }
    asm volatile("" : "+s"(sx), "+s"(sw), "+s"(po) : "v"(warm));
  } else {
    const __amdgpu_buffer_rsrc_t rq = mk_rsrc(myws + (size_t)(call % NSLOT) * SLOT + (size_t)((rk >> (4 * xcd)) & 7u) * SHARD +
                                              (size_t)(slab * 64 + wid * 8) * 4096, 32768);
    const unsigned vq = (unsigned)(lane & 7) * 4096 + (unsigned)((lane >> 3) & 3) * 1024 + 1020;
    unsigned got = 0;
    asm volatile("" : "+s"(sx), "+s"(sw), "+s"(po));
    while (__builtin_amdgcn_s_memrealtime() - t0 < T_W0) __builtin_amdgcn_s_sleep(8);
    asm volatile("" ::: "memory");
    issue_own(code);
#pragma unroll 1
    for (unsigned n = 1; got != 15 && !dead; ++n) {
      __builtin_amdgcn_s_sleep(WSLEEP);
      const unsigned hh = __builtin_amdgcn_raw_buffer_load_b32(rq, vq, 0, AUX_SYS);
      const unsigned miss = (unsigned)__builtin_amdgcn_ballot_w64(hh == 0u);
#pragma unroll
      for (int e = 0; e < 4; ++e) {
        if (!((got >> e) & 1) && !((miss >> (8 * e)) & 255u)) {
          got |= 1u << e;
#pragma unroll
          for (int q = 0; q < 8; ++q) wf[q][e] = __builtin_amdgcn_raw_buffer_load_b128(rq, vlin + (unsigned)e * 1024, (unsigned)q * 4096, AUX_SYS);
        }
      }
      if ((n & 63) == 0 && expired()) dead = true;
      asm volatile("" ::: "memory");
    }
    STAMP(3);
    for (unsigned n = 1;; ++n) {
      unsigned m = 0xffffffffu;
#pragma unroll
      for (int q = 0; q < 8; ++q)
#pragma unroll
        for (int e = 0; e < 4; ++e) m = min4(m, wf[q][e]);
      if (__builtin_expect(__builtin_amdgcn_ballot_w64(m == 0u) == 0ull || dead, 1)) break;
      if ((n & 63) == 0 && expired()) dead = true;
#pragma unroll
      for (int q = 0; q < 8; ++q)
#pragma unroll
        for (int e = 0; e < 4; ++e) wf[q][e] = __builtin_amdgcn_raw_buffer_load_b128(rq, vlin + (unsigned)e * 1024, (unsigned)q * 4096, AUX_SYS);
      asm volatile("" ::: "memory");
    }
  }
  STAMP(1);
  const float scale = sx * sw;
  const __amdgpu_buffer_rsrc_t rs = mk_rsrc(myws + (size_t)(call % NSLOT) * SLOT, SLOT);
  const __amdgpu_buffer_rsrc_t rz = mk_rsrc(myws + (size_t)((call + 2) % NSLOT) * SLOT, SLOT);
  const __amdgpu_buffer_rsrc_t ro = mk_rsrc(po, NR * MLOC * NLOC * 4 - (cross ? 3 * 512 : 0));
  const unsigned vh = (unsigned)(lane & 7) * 1024 + 1020;
  const int orow = 2 * wid + (lg & 1);
  const unsigned vout = lane < 32 ? (unsigned)orow * (NLOC * 4) + (unsigned)slab * 256 + (unsigned)li * 16 : 0x80000000u;
  char* const lw = smem + wid * 4096 + li * 256;
  const char* const lr = smem + orow * 256 + ((li ^ orow) * 16) + (lane >> 5) * 16384;
  const char* const xslot = myws + (size_t)(call % NSLOT) * SLOT;
  auto expected = [&](int code) -> bool {
    const int g = code & 31;
    const int when = (code & 0xe0) == 0 ? 0 : (code & 0x80) == 0 ? 213 - SPECM + 113 * g : g == NI ? 400 : g > NI ? T_SA : T_P + 300;
    return code >= 0 && (int)(__builtin_amdgcn_s_memrealtime() - t0) >= when;
  };
  __builtin_amdgcn_sched_barrier(0);
  int buf = 0;
#pragma unroll 1
  do {
    const int ncode = next();
    if (code & 0xe0) {
      unsigned m = 0xffffffffu;
#pragma unroll
      for (int q = 0; q < 8; ++q) m = min4(m, xv[q]);
      if (__builtin_expect(__builtin_amdgcn_ballot_w64(m == 0u) != 0ull && !dead, 0)) {
        const int s = (code & 31) * 64 + wid * 8;
        const unsigned so = (unsigned)(code >> 8) * SHARD + (unsigned)s * 1024;
        const unsigned ho = (code & 0x80) == 0 && s >= EARLY0 ? so - EARLY * 8192 : so;
        bool poll = true;
        for (unsigned n = 1;; ++n) {
          if (poll) {
            const unsigned hh = __builtin_amdgcn_raw_buffer_load_b32(rs, vh, ho, AUX_SYS);
            poll = __builtin_amdgcn_ballot_w64(hh == 0u) != 0ull;
            if (poll) __builtin_amdgcn_s_sleep(HSLEEP);
          }
          if (!poll) {
#pragma unroll
            for (int q = 0; q < 8; ++q) xv[q] = __builtin_amdgcn_raw_buffer_load_b128(rs, vlin + (unsigned)(q & 3) * 1024, so + (unsigned)(q & 4) * 1024, AUX_SYS);
            m = 0xffffffffu;
#pragma unroll
            for (int q = 0; q < 8; ++q) m = min4(m, xv[q]);
            if (__builtin_amdgcn_ballot_w64(m == 0u) == 0ull) break;
            if (n > 64) __builtin_amdgcn_s_sleep(16);
          }
          if ((n & 63) == 0 && expired()) { dead = true; break; }
          asm volatile("" ::: "memory");
        }
        if (dead) __builtin_amdgcn_raw_buffer_store_b32(call, rc, 0, 0, AUX_AGENT);
      }
    }
    const bool nown = (ncode & 0xe0) == 0;
    __amdgpu_buffer_rsrc_t rn;
    const unsigned nstep = nown ? 64 : 1024;
    const unsigned nso = (unsigned)(ncode & 31) * GROUP + (unsigned)wid * 8 * nstep;
    const unsigned nvo = nown ? vsrc : vlin;
    f32x4 acc[4];
#pragma unroll
    for (int e = 0; e < 4; ++e) acc[e] = f32x4{0.f, 0.f, 0.f, 0.f};
#pragma unroll
    for (int ks = 0; ks < 4; ++ks) {
      const i32x8 b = {(int)xv[2 * ks][0], (int)xv[2 * ks][1], (int)xv[2 * ks][2], (int)xv[2 * ks][3],
                       (int)xv[2 * ks + 1][0], (int)xv[2 * ks + 1][1], (int)xv[2 * ks + 1][2], (int)xv[2 * ks + 1][3]};
#pragma unroll
      for (int e = 0; e < 4; ++e) {
        const u32x4 a0 = wf[2 * ks][e], a1 = wf[2 * ks + 1][e];
        const i32x8 a = {(int)a0[0], (int)a0[1], (int)a0[2], (int)a0[3], (int)a1[0], (int)a1[1], (int)a1[2], (int)a1[3]};
        acc[e] = __builtin_amdgcn_mfma_scale_f32_16x16x128_f8f6f4(a, b, acc[e],   1,   0, 0, 0, 0, 0);
      }
      __builtin_amdgcn_sched_barrier(0);
      if (ks == 0 && !cross) {
        const unsigned zo = (code & 0xe0) ? (unsigned)(code >> 8) * SHARD + (unsigned)(code & 31) * GROUP
                                          : ((rk >> (16 + ((code >> 1) & 12))) & 7u) * SHARD + (unsigned)(code & 7) * GROUP;
        st128(u32x4{0u, 0u, 0u, 0u}, rz, vlin, zo + (unsigned)(slab * 8 + wid) * 1024);
      }
      if (ks == 0) rn = mk_rsrc(nown ? (const char*)px : xslot + (size_t)((ncode >> 8) & 7) * SHARD, expected(ncode) ? SHARD : 0);
      xv[2 * ks] = __builtin_amdgcn_raw_buffer_load_b128(rn, nvo, nso + (unsigned)(2 * ks) * nstep, AUX_SYS);
      xv[2 * ks + 1] = __builtin_amdgcn_raw_buffer_load_b128(rn, nvo, nso + (unsigned)(2 * ks + 1) * nstep, AUX_SYS);
      __builtin_amdgcn_sched_barrier(0);
    }
#pragma unroll
    for (int r = 0; r < 4; ++r) {
      f32x4 v = {acc[0][r], acc[1][r], acc[2][r], acc[3][r]};
      if (dead) v[0] = __builtin_nanf("");
      *(f32x4*)(lw + buf + (((lg * 4 + r) ^ li) * 16)) = v;
    }
    asm volatile("s_waitcnt lgkmcnt(0)" ::: "memory");
    __builtin_amdgcn_s_barrier();
    asm volatile("" ::: "memory");
    f32x4 pr[4], ov;
#pragma unroll
    for (int v = 0; v < 4; ++v) pr[v] = *(const f32x4*)(lr + buf + v * 4096);
    const f32x4 sum = (pr[0] + pr[1]) + (pr[2] + pr[3]);
#pragma unroll
    for (int r = 0; r < 4; ++r) {
      const unsigned su = __float_as_uint(sum[r]);
      const u32x2 sw2 = __builtin_amdgcn_permlane32_swap(su, su, false, false);
      const unsigned s0 = sw2[0], s1 = sw2[1];
      const float y = (__uint_as_float(s0) + __uint_as_float(s1)) * scale;
      ov[r] = y * __builtin_amdgcn_rcpf(1.f + __expf(-__builtin_amdgcn_fmed3f(y, -60.f, 60.f)));
    }
    st128(__builtin_bit_cast(u32x4, ov), ro, vout, ((unsigned)(code >> 8) * MLOC + (unsigned)(code & 31) * 16) * (NLOC * 4));
    __builtin_amdgcn_sched_barrier(0);
    code = ncode;
    buf ^= 32768;
  } while (code >= 0);
  STAMP(5);
  if (cross) {
    asm volatile("s_waitcnt vmcnt(0)" ::: "memory");
    __builtin_amdgcn_s_barrier();
    if (wid == 0) __builtin_amdgcn_raw_buffer_store_b32(call, mk_rsrc(pws + CTRL_OFF, CTRL_BYTES), 0, C_FLAG + rank * 256 + (unsigned)c28 * 4, AUX_SYS);
  }
  STAMP_DRAIN();
  STAMP(6);
  STAMP_FLUSH();
}

#define DUR0 1.2f
#define DUR1 2.0f
#define BEG0 8.f
#define BEG2 13.9f
#define BEG1 12.f
void build_tasks(Params& p) {
  for (int t = 0; t < 2; ++t) {
    float when[168];
    uint8_t what[168];
    int n = 0;
    auto add = [&](float w, int role, int g) {
      int i = n++;
      for (; i > 0 && when[i - 1] > w; --i) { when[i] = when[i - 1]; what[i] = what[i - 1]; }
      when[i] = w; what[i] = (uint8_t)(role << 5 | g);
    };
    for (int g = 0; g < (t ? NNO : 32); ++g) add(0.f, 0, g);
    for (int g = 0; g < (t ? NNO : NI); ++g) for (int r = 1; r < 4; ++r) add(1.f + 1.13f * (g + 1), r, g);
    for (int i = 0; i < (t ? PHI : NX); ++i)
      for (int r = t ? 5 : 4; r < 8; ++r)
        add(i == 0 ? 3.f : i < PHI ? T_SA * 0.01f : T_P * 0.01f + 1.5f * (i - PHI + 1), r, i < PHI ? NI + i : NNO + i - PHI);
    float busy[14];
    int len[14];
    uint8_t lst[14][16];
    for (int j = 0; j < 14; ++j) { busy[j] = t ? (j < 4 ? BEG2 : BEG1) : BEG0; len[j] = 0; }
    for (int i = 0; i < n; ++i) {
      int b = -1;
      for (int j = 0; j < 14; ++j) if (len[j] < 15 && (b < 0 || busy[j] < busy[b])) b = j;
      busy[b] = (busy[b] > when[i] ? busy[b] : when[i]) + (t ? DUR1 : DUR0);
      lst[b][len[b]++] = what[i];
    }
    for (int j = 0; j < 14; ++j) {
      u64 v[2] = {0, 0};
      const int o = 0;
      for (int i = 0; i < 16; ++i) v[i >> 3] |= (u64)(i >= o && i - o < len[j] ? lst[j][i - o] : 63) << (8 * (i & 7));
      p.task[t][j][0] = v[0]; p.task[t][j][1] = v[1];
    }
  }
}

}

extern "C" void kernel_launch(int rank, int nranks, ncclComm_t comm,
                              hipStream_t stream,
                              void* const* d_in, const int* in_sizes, int n_in,
                              void* d_out, int out_size,
                              const int* in_dtypes, int out_dtype,
                              void* const* d_out_all,
                              void* const* d_ws_all, size_t ws_bytes) {
  (void)comm; (void)in_dtypes; (void)out_dtype;
  if (nranks != NR || (unsigned)rank >= (unsigned)NR || n_in < 4 || in_sizes[0] != MLOC * KD || in_sizes[1] != KD * NLOC ||
      out_size != NR * MLOC * NLOC || ws_bytes < CTRL_OFF + CTRL_BYTES) {
    fprintf(stderr, "kernel_launch: unexpected problem shape\n");
    return;
  }
  static bool attr_done[NR];
  if (!attr_done[rank]) {
    HIP_CHECK(hipFuncSetAttribute((const void*)ag_gemm_kernel, hipFuncAttributeMaxDynamicSharedMemorySize, LDS_BYTES));
    attr_done[rank] = true;
  }
  Params p;
  p.x = (const uint8_t*)d_in[0];
  p.w = (const uint8_t*)d_in[1];
  p.sx = (const float*)d_in[2];
  p.sw = (const float*)d_in[3];
  p.myws = (char*)d_ws_all[rank];
  const int quad = 0x4d;
  auto qidx = [&](int r) { int n = 0; for (int q = 0; q < r; ++q) n += ((quad >> q) & 1) == ((quad >> r) & 1); return n; };
  int role[NR], na = 1, nb = 4;
  role[0] = rank;
  for (int j = 1; j < NR; ++j) {
    const int q = (rank + j) & 7;
    if (((quad >> q) & 1) == ((quad >> rank) & 1)) role[na++] = q; else role[nb++] = q;
  }
  p.rk = 0; p.wq = 0;
  for (int i = 0; i < NR; ++i) {
    p.ws[i] = (char*)d_ws_all[role[i]];
    p.out[i] = i == 0 ? (float*)d_out : (float*)d_out_all[role[i]] + (i >= 4 ? qidx(rank) * 128 : 0);
    p.rk |= (unsigned)role[i] << (4 * i);
    if (i >= 4) p.wq |= (unsigned)qidx(role[i]) << (4 * (i - 4));
  }
  build_tasks(p);
  hipLaunchKernelGGL(ag_gemm_kernel, dim3(NBLK), dim3(NTHR), LDS_BYTES, stream, p);
  HIP_CHECK(hipGetLastError());
}
```

```cpp
#include <hip/hip_runtime.h>
#include <rccl/rccl.h>
#include <cstdio>
#include <cstdint>

#define HIP_CHECK(expr) do { hipError_t _e = (expr); if (_e != hipSuccess) { \
    fprintf(stderr, "HIP error %s at %s:%d\n", hipGetErrorString(_e), __FILE__, __LINE__); \
    return; } } while (0)

typedef unsigned u32x4 __attribute__((ext_vector_type(4)));
__device__ u32x4 raw_load128(u32x4 rsrc, int voff, int soff, int aux) __asm("llvm.amdgcn.raw.buffer.load.v4i32");
__device__ unsigned raw_load32(u32x4 rsrc, int voff, int soff, int aux) __asm("llvm.amdgcn.raw.buffer.load.i32");

namespace {

typedef unsigned u32x2 __attribute__((ext_vector_type(2)));
typedef int i32x8 __attribute__((ext_vector_type(8)));
typedef float f32x4 __attribute__((ext_vector_type(4)));
typedef float f32x2 __attribute__((ext_vector_type(2)));
typedef unsigned long long u64;

constexpr int NR = 8;
constexpr int KD = 4096;
constexpr int MLOC = 512;
constexpr int NLOC = 512;
constexpr unsigned SHARD = MLOC * KD;
constexpr unsigned GROUP = 16 * KD;
constexpr unsigned SLOT = NR * SHARD;
constexpr int NSLOT = 3;
constexpr size_t CTRL_OFF = (size_t)NSLOT * SLOT;
constexpr unsigned CTRL_BYTES = 64 * 1024;
constexpr unsigned C_CNT = 256, C_ROLE = 4096, C_FLAG = 16384, C_TICK = 32768;
constexpr int NBLK = 256;
constexpr int NWAVE = 8;
constexpr int NTHR = NWAVE * 64;
constexpr unsigned LDS_BYTES = 96 * 1024;
constexpr u64 DEADLINE = 50000000ull;
constexpr int AUX_SYS = 17;
constexpr int AUX_AGENT = 16;

#define PHI 4
#define PSI 2
constexpr int NNO = 32 - PHI - PSI;
constexpr int NI = NNO + PSI;
constexpr int NX = PHI + PSI;
constexpr int NIP = 16;
constexpr int NUNIT = NI * 32;
#define NSTAT 6
#define NDYNP 10
constexpr int NDYN = NDYNP * NIP;
constexpr int NXT = (NUNIT - NSTAT * NIP - NDYN) / 4;
constexpr int NMAX = 120;
#define AHEAD 6
#define AHEAD1 8
#define EARLY 4
#define EARLY0 ((NUNIT - NDYN) * 2 + 128)
#define HSLEEP 2
#define T_S 1050
#define T_P 1200
#define WWIN 1
#define W1WIN 1
#define SWIN 1
#define T_W2 600
#define T_L 600
#define T_X0 150
#define T_X1 450
#define SPECM 0
#define T_SA 1800
#define T_W0 300
#define T_W1 700
#define WSLEEP 4
#define STR2(x) #x
#define STR(x) STR2(x)
static_assert(NNO >= 8 && PHI >= 1, "W quarters live in the unused groups of a cross source");

#define STAMP(i) do {} while (0)
#define STAMP_FLUSH() do {} while (0)
#define STAMP_DRAIN() do {} while (0)

struct Params {
  u32x4 dx, dc;
  char* myws;
  const uint8_t* w;
  unsigned rk;
  unsigned wq;
  char* ws[NR];
  const float* sx;
  const float* sw;
  float* out[NR];
  u64 task[2][14][2];
};

__device__ __forceinline__ __amdgpu_buffer_rsrc_t mk_rsrc(const void* p, unsigned bytes) {
  return __builtin_amdgcn_make_buffer_rsrc(const_cast<void*>(p), 0, bytes, 0x00020000);
}
__device__ __forceinline__ unsigned min3u(unsigned a, unsigned b, unsigned c) { return min(min(a, b), c); }
__device__ __forceinline__ unsigned min4(unsigned m, u32x4 v) { return min3u(min3u(m, v[0], v[1]), v[2], v[3]); }
__device__ __forceinline__ u32x4 canon(u32x4 v) {
#pragma unroll
  for (int d = 0; d < 4; ++d) v[d] = v[d] ? v[d] : 0x80808080u;
  return v;
}
__device__ __forceinline__ void st128(u32x4 v, __amdgpu_buffer_rsrc_t r, unsigned voff, unsigned soff) {
  __builtin_amdgcn_raw_buffer_store_b128(v, r, voff, soff, AUX_SYS);
  asm volatile("s_nop 1" :: "v"(v) : "memory");
}
typedef __attribute__((address_space(3))) int lds_int;
typedef __attribute__((address_space(1))) unsigned gu32;
__device__ __forceinline__ void lds_st(lds_int* a, int v) { __hip_atomic_store(a, v, __ATOMIC_RELAXED, __HIP_MEMORY_SCOPE_WORKGROUP); }
__device__ __forceinline__ int lds_ld(lds_int* a) { return __hip_atomic_load(a, __ATOMIC_RELAXED, __HIP_MEMORY_SCOPE_WORKGROUP); }
__device__ __forceinline__ unsigned src_off(int s) { return (unsigned)(s >> 6) * GROUP + (unsigned)(s & 63) * 64; }
__device__ __forceinline__ void tr16(const unsigned (&r)[16], u32x4 (&o)[4]) {
#pragma unroll
  for (int d = 0; d < 4; ++d) {
    const unsigned a0 = r[4 * d], a1 = r[4 * d + 1], a2 = r[4 * d + 2], a3 = r[4 * d + 3];
    const unsigned t0 = __builtin_amdgcn_perm(a1, a0, 0x05010400u), t1 = __builtin_amdgcn_perm(a1, a0, 0x07030602u);
    const unsigned t2 = __builtin_amdgcn_perm(a3, a2, 0x05010400u), t3 = __builtin_amdgcn_perm(a3, a2, 0x07030602u);
    o[0][d] = __builtin_amdgcn_perm(t2, t0, 0x05040100u);
    o[1][d] = __builtin_amdgcn_perm(t2, t0, 0x07060302u);
    o[2][d] = __builtin_amdgcn_perm(t3, t1, 0x05040100u);
    o[3][d] = __builtin_amdgcn_perm(t3, t1, 0x07060302u);
  }
}

__global__ void __launch_bounds__(NTHR) ag_gemm_kernel(Params p) {
  extern __shared__ __attribute__((aligned(16))) char smem[];
  const u32x4 dx = p.dx, dc = p.dc;
  char* myws = p.myws;
  const uint8_t* pw = p.w;
  unsigned rk = p.rk, wq = p.wq;
  __builtin_amdgcn_sched_barrier(0);
  const int wid = __builtin_amdgcn_readfirstlane(threadIdx.x >> 6);
  const int lane = threadIdx.x & 63;
  const int bid = blockIdx.x;
  const int cu = bid >> 3;
  const u64 t0 = __builtin_amdgcn_s_memrealtime();
  int xcd = __builtin_amdgcn_s_getreg(20 | (3 << 11)) & 7;
  auto peer = [&]() -> int { return xcd < 4 ? (0x21321321 >> (4 * wid)) & 3 : xcd; };
  const int li = lane & 15, lg = lane >> 4;
  unsigned vsrc = (unsigned)li * KD + (unsigned)lg * 16;
  const unsigned vlin = (unsigned)lane * 16;
  unsigned vw = (unsigned)lg * 16 * NLOC + (unsigned)li * 4;
  const int h = (wid + 5) >> 3;
  const int s5 = cu * 2 + (wid & 1) + (wid >> 1) * 8;
  static_assert(NIP == 16, "the first two pieces of a wave are 32 apart, in one group");
  auto sfirst = [&]() -> unsigned {
    const unsigned m = 0u - (unsigned)(xcd >> 2);
    return ((unsigned)(2 * (cu * 4 + xcd) + h) * 64 & ~m) | ((NI * GROUP + (unsigned)s5 * 64) & m);
  };
  unsigned so0 = sfirst();
  unsigned cnt = C_CNT + (unsigned)(bid * NWAVE + wid) * 4;
  asm volatile("" : "+s"(so0), "+s"(cnt), "+v"(vsrc), "+v"(vw));
  __builtin_amdgcn_sched_barrier(0);
  u32x4 r0[2];
  unsigned craw;
  if (__builtin_expect(cu < 4, 1)) {
    r0[0] = raw_load128(dx, vsrc, so0, 0);
    __builtin_amdgcn_sched_barrier(0);
    craw = raw_load32(dc, 0, cnt, AUX_AGENT);
    __builtin_amdgcn_sched_barrier(0);
    r0[1] = raw_load128(dx, vsrc + 2048, so0, 0);
    __builtin_amdgcn_sched_barrier(0);
  }
  char* pws = p.ws[peer()];
  const uint8_t* px = (const uint8_t*)((u64)dx[1] << 32 | dx[0]);
  const __amdgpu_buffer_rsrc_t rc = mk_rsrc(myws + CTRL_OFF, CTRL_BYTES);
  const __amdgpu_buffer_rsrc_t rxs = mk_rsrc(px, SHARD);
  const unsigned rank = rk & 7u;
  const __amdgpu_buffer_rsrc_t rw = mk_rsrc(pw, KD * NLOC);
  gu32* const roles = (gu32*)(myws + CTRL_OFF + C_ROLE) + cu * 32 + (lane & 7);
  lds_int* const verdict = (lds_int*)(smem + 65536);
  unsigned took = 0;
  const bool asker = wid == (cu < 4 ? 7 : 0);
  auto ask = [&]() { if (asker && lane < 8) took = __hip_atomic_fetch_add(roles, lane == xcd ? 1u : 0u, __ATOMIC_RELAXED, __HIP_MEMORY_SCOPE_AGENT); };
  auto resolve = [&]() {
    if (!asker) return;
    int r = xcd;
    unsigned c = __builtin_amdgcn_readlane(took, r);
    unsigned behind = (unsigned)__builtin_amdgcn_ballot_w64((int)(took - c) < 0) & 255u;
    while (__builtin_expect(behind != 0, 0)) {
      const int rn = __builtin_ctz(behind);
      if (lane < 8) took = __hip_atomic_fetch_add(roles, lane == rn ? 1u : lane == r ? ~0u : 0u, __ATOMIC_RELAXED, __HIP_MEMORY_SCOPE_AGENT);
      r = rn;
      c = __builtin_amdgcn_readlane(took, r);
      behind = (unsigned)__builtin_amdgcn_ballot_w64((int)(took - c) < 0) & 255u;
    }
    lds_st(verdict + (lane & 1), lane & 1 ? r : (int)c + 1);
  };
  auto meet = [&]() -> int {
    asm volatile("s_waitcnt lgkmcnt(0)" ::: "memory");
    __builtin_amdgcn_s_barrier();
    asm volatile("" ::: "memory");
    return lds_ld(verdict + (lane & 1));
  };
  if (__builtin_expect(cu < 4, 1)) {
    lds_int* const pos = (lds_int*)smem;
    lds_int* const done = pos + 192;
    auto dfirst = [&](unsigned so) -> unsigned { return (so & ~(GROUP - 1)) | (so & (GROUP - 1)) * 16; };
    unsigned do0 = dfirst(so0), vd0 = xcd < 4 && wid >= 6 ? SHARD : vlin;
    char* pdst = pws + (size_t)rank * SHARD;
    asm volatile("" : "+s"(do0), "+v"(vd0), "+s"(pdst));
    __builtin_amdgcn_sched_barrier(0);
    const unsigned call = __builtin_amdgcn_readfirstlane(craw) + 1;
    __amdgpu_buffer_rsrc_t rd = mk_rsrc(pdst + (size_t)(call % NSLOT) * SLOT, SHARD);
    st128(canon(r0[0]), rd, vd0, do0);
    STAMP(2);
    __builtin_amdgcn_sched_barrier(0);
#pragma unroll 1
    for (bool again = false;; again = true) {
      if (__builtin_expect(xcd < 4, 1)) {
        const int pidx = cu * 4 + xcd;
        if (wid == 6) { lds_st(pos + lane, -1); lds_st(pos + lane + 64, -1); lds_st(pos + lane + 128, -1); lds_st(done + lane, lane >= 6 ? 0x7fffffff : 0); }
        asm volatile("s_waitcnt lgkmcnt(0)" ::: "memory");
        __builtin_amdgcn_s_barrier();
        __builtin_amdgcn_sched_barrier(0);
        if (__builtin_expect(wid < 6, 1)) {
          u32x4 r[2] = {r0[0], r0[1]};
          int up[2] = {pidx, pidx + NIP};
#pragma unroll 1
          for (int n0 = 0; up[0] < NUNIT; n0 += 2) {
#pragma unroll
            for (int i = 0; i < 2; ++i) {
              const int n = n0 + i;
              lds_st(done + wid, n + 1);
              int un = pidx + (n + 2) * NIP;
              if (n + 2 >= NSTAT) {
                for (;;) {
                  un = __builtin_amdgcn_readfirstlane(lds_ld(pos + n + 2));
                  if (un >= 0) break;
                  __builtin_amdgcn_s_sleep(1);
                }
              }
              r[i] = __builtin_amdgcn_raw_buffer_load_b128(rxs, vsrc, src_off(2 * un + h), 0);
              up[i] = un;
              __builtin_amdgcn_sched_barrier(0);
              const int u = up[i ^ 1];
              st128(canon(r[i ^ 1]), rd, u < NUNIT ? vlin : SHARD, (unsigned)(2 * u + h) * 1024);
              __builtin_amdgcn_sched_barrier(0);
            }
          }
          STAMP(5);
          STAMP_DRAIN();
        } else if (wid == 7) {
          if (!again) ask();
          if (pidx == 0) __builtin_amdgcn_raw_buffer_store_b32(0u, rc, (unsigned)(lane & 7) * 128, C_TICK + ((call + 1) & 1) * 1024, AUX_AGENT);
          gu32* tick = (gu32*)(myws + CTRL_OFF + C_TICK + (call & 1) * 1024);
          int n = NSTAT;
          bool mine = true;
          for (;;) {
            int d = lds_ld(done + (lane & 7));
            d = min(d, __builtin_amdgcn_update_dpp(0x7fffffff, d, 0x111, 0xf, 0xf, false));
            d = min(d, __builtin_amdgcn_update_dpp(0x7fffffff, d, 0x112, 0xf, 0xf, false));
            d = min(d, __builtin_amdgcn_update_dpp(0x7fffffff, d, 0x114, 0xf, 0xf, false));
            const int lim = min(__builtin_amdgcn_readlane(d, 7) + (mine ? AHEAD1 : AHEAD), NMAX);
            const int k = lim - n;
            if (k <= 0) {
              if (n >= NMAX) { lds_st(pos + n + (lane & 3), NUNIT); break; }
              __builtin_amdgcn_s_sleep(2);
              continue;
            }
            unsigned t = 0;
            if (lane == 0) t = __hip_atomic_fetch_add(tick + (mine ? xcd * 32 : 128), (unsigned)k, __ATOMIC_RELAXED, __HIP_MEMORY_SCOPE_AGENT);
            if (mine) {
              const int t0 = (int)min(__builtin_amdgcn_readfirstlane(t), (unsigned)NXT);
              const int kk = min(k, NXT - t0);
              if (lane < kk) lds_st(pos + n + lane, NSTAT * NIP + (t0 + lane) * 4 + xcd);
              n += kk;
              mine = t0 + k < NXT;
            } else {
              const int t0 = (int)min(__builtin_amdgcn_readfirstlane(t), (unsigned)NDYN);
              const bool last = t0 + k >= NDYN;
              if (lane < (last ? k + 4 : k)) lds_st(pos + n + lane, min(NUNIT - NDYN + t0 + lane, NUNIT));
              if (last) break;
              n += k;
            }
          }
        }
        STAMP(6);
      } else {
        constexpr int NPW = NX * 2;
        auto spos = [&](int n) -> int { const int g = n >> 1; return (g < PHI ? NI + g : NNO + g - PHI) * 64 + (n & 1) * 32 + s5; };
        u32x4 r[NPW];
        st128(canon(r0[1]), rd, vlin, (unsigned)spos(1) * 1024);
        __builtin_amdgcn_sched_barrier(0);
        __builtin_amdgcn_s_barrier();
        __builtin_amdgcn_sched_barrier(0);
        if (!again) ask();
#pragma unroll
        for (int n = 2; n < NPW; ++n) r[n] = __builtin_amdgcn_raw_buffer_load_b128(rxs, vsrc, src_off(spos(n)), 0);
#pragma unroll
        for (int n = 2; n < NPW; ++n) r[n] = canon(r[n]);
        STAMP(1);
#pragma unroll
        for (int n = 2; n < NPW; ++n) {
          if (n == 2 || n == 2 * PHI) {
            while (__builtin_amdgcn_s_memrealtime() - t0 < (n == 2 * PHI ? T_P : T_S)) __builtin_amdgcn_s_sleep(2);
            asm volatile("" ::: "memory");
          }
          st128(r[n], rd, vlin, (unsigned)spos(n) * 1024);
          asm volatile("s_waitcnt vmcnt(" STR(SWIN) ")" ::: "memory");
        }
        STAMP_DRAIN();
        STAMP(5);
      }
      if (again) break;
      resolve();
      const int role = __builtin_amdgcn_readlane(meet(), 1);
      if (__builtin_expect(role == xcd, 1)) break;
      xcd = role;
      rd = mk_rsrc(p.ws[peer()] + (size_t)(call % NSLOT) * SLOT + (size_t)rank * SHARD, SHARD);
      r0[0] = __builtin_amdgcn_raw_buffer_load_b128(rxs, vsrc, sfirst(), 0);
      r0[1] = __builtin_amdgcn_raw_buffer_load_b128(rxs, vsrc + 2048, sfirst(), 0);
      st128(canon(r0[0]), rd, xcd < 4 && wid >= 6 ? SHARD : vlin, dfirst(sfirst()));
    }
    __builtin_amdgcn_raw_buffer_store_b32(call, rc, 0, cnt, AUX_AGENT);
    if (cu == 0 && wid == 0 && xcd >= 4) {
      const unsigned fo = C_FLAG + ((rk >> (4 * xcd)) & 7u) * 256;
      for (unsigned n = 1;; ++n) {
        const unsigned f = __builtin_amdgcn_raw_buffer_load_b32(rc, (unsigned)min(lane, 27) * 4, fo, AUX_SYS);
        if (__builtin_amdgcn_ballot_w64(f != call) == 0ull) break;
        if ((n & 63) == 0 && __builtin_amdgcn_s_memrealtime() - t0 > DEADLINE) {
          __builtin_amdgcn_raw_buffer_store_b32(call, rc, 0, 0, AUX_AGENT);
          p.out[0][(size_t)rank * MLOC * NLOC + lane] = __builtin_nanf("");
          break;
        }
        __builtin_amdgcn_s_sleep(1);
        asm volatile("" ::: "memory");
      }
    }
    STAMP_FLUSH();
    return;
  }

  ask();
  __builtin_amdgcn_sched_barrier(0);
  unsigned warm = 0;
  unsigned raw0[2][16];
  auto first_loads = [&](bool first) {
    if (xcd < 4) {
      const int wi = wid * 28 + cu - 4;
      if (wi < NI * 4)
        warm = __builtin_amdgcn_raw_buffer_load_b32(rxs, (unsigned)(lane >> 4) * KD + (unsigned)((lane >> 1) & 7) * 512 + (unsigned)(lane & 1) * 64,
                                                    (unsigned)wi * 4 * KD + (unsigned)xcd * 128, 0);
      if (first) resolve();
    } else if (cu < 12) {
      const unsigned so = (unsigned)((cu - 4) * 8 + wid) * 64 * NLOC + ((wq >> (4 * (xcd - 4))) & 3u) * 128;
#pragma unroll
      for (int hh = 0; hh < 2; ++hh)
#pragma unroll
        for (int i = 0; i < 16; ++i) raw0[hh][i] = __builtin_amdgcn_raw_buffer_load_b32(rw, vw + (unsigned)(i * NLOC + hh * 64), so, 0);
      __builtin_amdgcn_sched_barrier(0);
      if (first) resolve();
    } else {
      if (cu < 28)
        warm = __builtin_amdgcn_raw_buffer_load_b32(rw, (unsigned)(lane >> 1) * NLOC + (unsigned)(lane & 1) * 64,
                                                    (unsigned)(((cu - 12) & 7) * 16 + wid * 2 + ((cu - 12) >> 3)) * 32 * NLOC + ((wq >> (4 * (xcd - 4))) & 3u) * 128, 0);
      if (first) resolve();
    }
    __builtin_amdgcn_sched_barrier(0);
  };
  first_loads(true);
  const int vd = meet();
  const unsigned call = __builtin_amdgcn_readlane(vd, 0);
  if (__builtin_expect(__builtin_amdgcn_readlane(vd, 1) != xcd, 0)) {
    xcd = __builtin_amdgcn_readlane(vd, 1);
    pws = p.ws[peer()];
    first_loads(false);
  }
  const bool cross = xcd >= 4;
  if (cross && cu < 12) {
    const __amdgpu_buffer_rsrc_t rd = mk_rsrc(pws + (size_t)(call % NSLOT) * SLOT + (size_t)rank * SHARD, SHARD);
    const int kb = (cu - 4) * 8 + wid;
    u32x4 o[2][4];
    tr16(raw0[0], o[0]);
    tr16(raw0[1], o[1]);
#pragma unroll
    for (int e = 0; e < 8; ++e) {
      if (e > WWIN) asm volatile("s_waitcnt vmcnt(" STR(WWIN) ")" ::: "memory");
      st128(canon(o[e & 1][e >> 1]), rd, vlin + (unsigned)(e >> 1) * 1024, (unsigned)((e & 1) * 64 + kb) * 4096);
      __builtin_amdgcn_sched_barrier(0);
      if (e == 0) STAMP(2);
    }
    STAMP(7);
    __builtin_amdgcn_s_barrier();
  }
  __builtin_amdgcn_sched_barrier(0);

  const int c28 = cu - 4;
  const int ci = xcd * 28 + c28;
  const int strm = cross ? c28 >> 1 : ci >> 3;
  const int slab = cross ? c28 & 1 : ci & 7;
  u64 tl = p.task[cross][strm][0], th = p.task[cross][strm][1];
  typedef const __attribute__((address_space(4))) float cf32;
  float sx = *(cf32*)p.sx, sw = *(cf32*)p.sw;
  float* po = p.out[cross ? xcd : 0];
  bool dead = false;
  auto expired = [&]() -> bool {
    return __builtin_amdgcn_readfirstlane(__builtin_amdgcn_raw_buffer_load_b32(rc, 0, 0, AUX_AGENT)) == call ||
           __builtin_amdgcn_s_memrealtime() - t0 > DEADLINE;
  };
  auto next = [&]() -> int {
    const int c = (int)(tl & 255u);
    tl = (tl >> 8) | (th << 56); th = (th >> 8) | (63ull << 56);
    if (c == 63) return -1;
    int role = c >> 5;
    if (cross && role >= 4) role = 4 + ((role + xcd) & 3);
    return (int)(((rk >> (4 * role)) & 7u) << 8) | c;
  };
  u32x4 xv[8];
  auto issue_own = [&](int code) {
    const __amdgpu_buffer_rsrc_t r = rxs;
    const unsigned so = (unsigned)(code & 31) * GROUP + (unsigned)wid * 512;
#pragma unroll
    for (int q = 0; q < 8; ++q) xv[q] = __builtin_amdgcn_raw_buffer_load_b128(r, vsrc + (unsigned)q * 64, so, 0);
  };
  int code = next();
  u32x4 wf[8][4];
  if (!cross) {
    const unsigned vo = vw + (unsigned)wid * 512 * NLOC + (unsigned)slab * 64;
    unsigned raw[3][16];
#pragma unroll
    for (int u = 0; u < 10; ++u) {
      if (u < 8) {
#pragma unroll
        for (int i = 0; i < 16; ++i) raw[u % 3][i] = __builtin_amdgcn_raw_buffer_load_b32(rw, vo, (unsigned)(u * 64 + i) * NLOC, 0);
      }
      if (u == 8) issue_own(code);
      __builtin_amdgcn_sched_barrier(0);
      if (u >= 2) tr16(raw[(u - 2) % 3], wf[u - 2]);
      __builtin_amdgcn_sched_barrier(0);
    }
    asm volatile("" : "+s"(sx), "+s"(sw), "+s"(po) : "v"(warm));
  } else {
    const __amdgpu_buffer_rsrc_t rq = mk_rsrc(myws + (size_t)(call % NSLOT) * SLOT + (size_t)((rk >> (4 * xcd)) & 7u) * SHARD +
                                              (size_t)(slab * 64 + wid * 8) * 4096, 32768);
    const unsigned vq = (unsigned)(lane & 7) * 4096 + (unsigned)((lane >> 3) & 3) * 1024 + 1020;
    unsigned got = 0;
    asm volatile("" : "+s"(sx), "+s"(sw), "+s"(po) : "v"(warm));
    while (__builtin_amdgcn_s_memrealtime() - t0 < T_W0) __builtin_amdgcn_s_sleep(8);
    asm volatile("" ::: "memory");
    issue_own(code);
#pragma unroll 1
    for (unsigned n = 1; got != 15 && !dead; ++n) {
      __builtin_amdgcn_s_sleep(WSLEEP);
      const unsigned hh = __builtin_amdgcn_raw_buffer_load_b32(rq, vq, 0, AUX_SYS);
      const unsigned miss = (unsigned)__builtin_amdgcn_ballot_w64(hh == 0u);
#pragma unroll
      for (int e = 0; e < 4; ++e) {
        if (!((got >> e) & 1) && !((miss >> (8 * e)) & 255u)) {
          got |= 1u << e;
#pragma unroll
          for (int q = 0; q < 8; ++q) wf[q][e] = __builtin_amdgcn_raw_buffer_load_b128(rq, vlin + (unsigned)e * 1024, (unsigned)q * 4096, AUX_SYS);
        }
      }
      if ((n & 63) == 0 && expired()) dead = true;
      asm volatile("" ::: "memory");
    }
    STAMP(3);
    for (unsigned n = 1;; ++n) {
      unsigned m = 0xffffffffu;
#pragma unroll
      for (int q = 0; q < 8; ++q)
#pragma unroll
        for (int e = 0; e < 4; ++e) m = min4(m, wf[q][e]);
      if (__builtin_expect(__builtin_amdgcn_ballot_w64(m == 0u) == 0ull || dead, 1)) break;
      if ((n & 63) == 0 && expired()) dead = true;
#pragma unroll
      for (int q = 0; q < 8; ++q)
#pragma unroll
        for (int e = 0; e < 4; ++e) wf[q][e] = __builtin_amdgcn_raw_buffer_load_b128(rq, vlin + (unsigned)e * 1024, (unsigned)q * 4096, AUX_SYS);
      asm volatile("" ::: "memory");
    }
  }
  STAMP(1);
  const float scale = sx * sw;
  const __amdgpu_buffer_rsrc_t rs = mk_rsrc(myws + (size_t)(call % NSLOT) * SLOT, SLOT);
  const __amdgpu_buffer_rsrc_t rz = mk_rsrc(myws + (size_t)((call + 2) % NSLOT) * SLOT, SLOT);
  const __amdgpu_buffer_rsrc_t ro = mk_rsrc(po, NR * MLOC * NLOC * 4 - (cross ? 3 * 512 : 0));
  const unsigned vh = (unsigned)(lane & 7) * 1024 + 1020;
  const int orow = 2 * wid + (lg & 1);
  const unsigned vout = lane < 32 ? (unsigned)orow * (NLOC * 4) + (unsigned)slab * 256 + (unsigned)li * 16 : 0x80000000u;
  char* const lw = smem + wid * 4096 + li * 256;
  const char* const lr = smem + orow * 256 + ((li ^ orow) * 16) + (lane >> 5) * 16384;
  const char* const xslot = myws + (size_t)(call % NSLOT) * SLOT;
  auto expected = [&](int code) -> bool {
    const int g = code & 31;
    const int when = (code & 0xe0) == 0 ? 0 : (code & 0x80) == 0 ? 213 - SPECM + 113 * g : g == NI ? 400 : g > NI ? T_SA : T_P + 300;
    return code >= 0 && (int)(__builtin_amdgcn_s_memrealtime() - t0) >= when;
  };
  __builtin_amdgcn_sched_barrier(0);
  int buf = 0;
#pragma unroll 1
  do {
    const int ncode = next();
    if (code & 0xe0) {
      unsigned m = 0xffffffffu;
#pragma unroll
      for (int q = 0; q < 8; ++q) m = min4(m, xv[q]);
      if (__builtin_expect(__builtin_amdgcn_ballot_w64(m == 0u) != 0ull && !dead, 0)) {
        const int s = (code & 31) * 64 + wid * 8;
        const unsigned so = (unsigned)(code >> 8) * SHARD + (unsigned)s * 1024;
        const unsigned ho = (code & 0x80) == 0 && s >= EARLY0 ? so - EARLY * 8192 : so;
        bool poll = true;
        for (unsigned n = 1;; ++n) {
          if (poll) {
            const unsigned hh = __builtin_amdgcn_raw_buffer_load_b32(rs, vh, ho, AUX_SYS);
            poll = __builtin_amdgcn_ballot_w64(hh == 0u) != 0ull;
            if (poll) __builtin_amdgcn_s_sleep(HSLEEP);
          }
          if (!poll) {
#pragma unroll
            for (int q = 0; q < 8; ++q) xv[q] = __builtin_amdgcn_raw_buffer_load_b128(rs, vlin + (unsigned)(q & 3) * 1024, so + (unsigned)(q & 4) * 1024, AUX_SYS);
            m = 0xffffffffu;
#pragma unroll
            for (int q = 0; q < 8; ++q) m = min4(m, xv[q]);
            if (__builtin_amdgcn_ballot_w64(m == 0u) == 0ull) break;
            if (n > 64) __builtin_amdgcn_s_sleep(16);
          }
          if ((n & 63) == 0 && expired()) { dead = true; break; }
          asm volatile("" ::: "memory");
        }
        if (dead) __builtin_amdgcn_raw_buffer_store_b32(call, rc, 0, 0, AUX_AGENT);
      }
    }
    const bool nown = (ncode & 0xe0) == 0;
    __amdgpu_buffer_rsrc_t rn;
    const unsigned nstep = nown ? 64 : 1024;
    const unsigned nso = (unsigned)(ncode & 31) * GROUP + (unsigned)wid * 8 * nstep;
    const unsigned nvo = nown ? vsrc : vlin;
    f32x4 acc[4];
#pragma unroll
    for (int e = 0; e < 4; ++e) acc[e] = f32x4{0.f, 0.f, 0.f, 0.f};
#pragma unroll
    for (int ks = 0; ks < 4; ++ks) {
      const i32x8 b = {(int)xv[2 * ks][0], (int)xv[2 * ks][1], (int)xv[2 * ks][2], (int)xv[2 * ks][3],
                       (int)xv[2 * ks + 1][0], (int)xv[2 * ks + 1][1], (int)xv[2 * ks + 1][2], (int)xv[2 * ks + 1][3]};
#pragma unroll
      for (int e = 0; e < 4; ++e) {
        const u32x4 a0 = wf[2 * ks][e], a1 = wf[2 * ks + 1][e];
        const i32x8 a = {(int)a0[0], (int)a0[1], (int)a0[2], (int)a0[3], (int)a1[0], (int)a1[1], (int)a1[2], (int)a1[3]};
        acc[e] = __builtin_amdgcn_mfma_scale_f32_16x16x128_f8f6f4(a, b, acc[e],   1,   0, 0, 0, 0, 0);
      }
      __builtin_amdgcn_sched_barrier(0);
      if (ks == 0 && !cross) {
        const unsigned zo = (code & 0xe0) ? (unsigned)(code >> 8) * SHARD + (unsigned)(code & 31) * GROUP
                                          : ((rk >> (16 + ((code >> 1) & 12))) & 7u) * SHARD + (unsigned)(code & 7) * GROUP;
        st128(u32x4{0u, 0u, 0u, 0u}, rz, vlin, zo + (unsigned)(slab * 8 + wid) * 1024);
      }
      if (ks == 0) rn = mk_rsrc(nown ? (const char*)px : xslot + (size_t)((ncode >> 8) & 7) * SHARD, expected(ncode) ? SHARD : 0);
      xv[2 * ks] = __builtin_amdgcn_raw_buffer_load_b128(rn, nvo, nso + (unsigned)(2 * ks) * nstep, AUX_SYS);
      xv[2 * ks + 1] = __builtin_amdgcn_raw_buffer_load_b128(rn, nvo, nso + (unsigned)(2 * ks + 1) * nstep, AUX_SYS);
      __builtin_amdgcn_sched_barrier(0);
    }
#pragma unroll
    for (int r = 0; r < 4; ++r) {
      f32x4 v = {acc[0][r], acc[1][r], acc[2][r], acc[3][r]};
      if (dead) v[0] = __builtin_nanf("");
      *(f32x4*)(lw + buf + (((lg * 4 + r) ^ li) * 16)) = v;
    }
    asm volatile("s_waitcnt lgkmcnt(0)" ::: "memory");
    __builtin_amdgcn_s_barrier();
    asm volatile("" ::: "memory");
    f32x4 pr[4], ov;
#pragma unroll
    for (int v = 0; v < 4; ++v) pr[v] = *(const f32x4*)(lr + buf + v * 4096);
    const f32x4 sum = (pr[0] + pr[1]) + (pr[2] + pr[3]);
#pragma unroll
    for (int r = 0; r < 4; ++r) {
      const unsigned su = __float_as_uint(sum[r]);
      const u32x2 sw2 = __builtin_amdgcn_permlane32_swap(su, su, false, false);
      const unsigned s0 = sw2[0], s1 = sw2[1];
      const float y = (__uint_as_float(s0) + __uint_as_float(s1)) * scale;
      ov[r] = y * __builtin_amdgcn_rcpf(1.f + __expf(-__builtin_amdgcn_fmed3f(y, -60.f, 60.f)));
    }
    st128(__builtin_bit_cast(u32x4, ov), ro, vout, ((unsigned)(code >> 8) * MLOC + (unsigned)(code & 31) * 16) * (NLOC * 4));
    __builtin_amdgcn_sched_barrier(0);
    code = ncode;
    buf ^= 32768;
  } while (code >= 0);
  STAMP(5);
  if (cross) {
    asm volatile("s_waitcnt vmcnt(0)" ::: "memory");
    __builtin_amdgcn_s_barrier();
    if (wid == 0) __builtin_amdgcn_raw_buffer_store_b32(call, mk_rsrc(pws + CTRL_OFF, CTRL_BYTES), 0, C_FLAG + rank * 256 + (unsigned)c28 * 4, AUX_SYS);
  }
  STAMP_DRAIN();
  STAMP(6);
  STAMP_FLUSH();
}

#define DUR0 1.2f
#define DUR1 2.0f
#define BEG0 8.f
#define BEG2 13.9f
#define BEG1 12.f
void build_tasks(Params& p) {
  for (int t = 0; t < 2; ++t) {
    float when[168];
    uint8_t what[168];
    int n = 0;
    auto add = [&](float w, int role, int g) {
      int i = n++;
      for (; i > 0 && when[i - 1] > w; --i) { when[i] = when[i - 1]; what[i] = what[i - 1]; }
      when[i] = w; what[i] = (uint8_t)(role << 5 | g);
    };
    for (int g = 0; g < (t ? NNO : 32); ++g) add(0.f, 0, g);
    for (int g = 0; g < (t ? NNO : NI); ++g) for (int r = 1; r < 4; ++r) add(1.f + 1.13f * (g + 1), r, g);
    for (int i = 0; i < (t ? PHI : NX); ++i)
      for (int r = t ? 5 : 4; r < 8; ++r)
        add(i == 0 ? 3.f : i < PHI ? T_SA * 0.01f : T_P * 0.01f + 1.5f * (i - PHI + 1), r, i < PHI ? NI + i : NNO + i - PHI);
    float busy[14];
    int len[14];
    uint8_t lst[14][16];
    for (int j = 0; j < 14; ++j) { busy[j] = t ? (j < 4 ? BEG2 : BEG1) : BEG0; len[j] = 0; }
    for (int i = 0; i < n; ++i) {
      int b = -1;
      for (int j = 0; j < 14; ++j) if (len[j] < 15 && (b < 0 || busy[j] < busy[b])) b = j;
      busy[b] = (busy[b] > when[i] ? busy[b] : when[i]) + (t ? DUR1 : DUR0);
      lst[b][len[b]++] = what[i];
    }
    for (int j = 0; j < 14; ++j) {
      u64 v[2] = {0, 0};
      const int o = 0;
      for (int i = 0; i < 16; ++i) v[i >> 3] |= (u64)(i >= o && i - o < len[j] ? lst[j][i - o] : 63) << (8 * (i & 7));
      p.task[t][j][0] = v[0]; p.task[t][j][1] = v[1];
    }
  }
}

}

extern "C" void kernel_launch(int rank, int nranks, ncclComm_t comm,
                              hipStream_t stream,
                              void* const* d_in, const int* in_sizes, int n_in,
                              void* d_out, int out_size,
                              const int* in_dtypes, int out_dtype,
                              void* const* d_out_all,
                              void* const* d_ws_all, size_t ws_bytes) {
  (void)comm; (void)in_dtypes; (void)out_dtype;
  if (nranks != NR || (unsigned)rank >= (unsigned)NR || n_in < 4 || in_sizes[0] != MLOC * KD || in_sizes[1] != KD * NLOC ||
      out_size != NR * MLOC * NLOC || ws_bytes < CTRL_OFF + CTRL_BYTES) {
    fprintf(stderr, "kernel_launch: unexpected problem shape\n");
    return;
  }
  static bool attr_done[NR];
  if (!attr_done[rank]) {
    HIP_CHECK(hipFuncSetAttribute((const void*)ag_gemm_kernel, hipFuncAttributeMaxDynamicSharedMemorySize, LDS_BYTES));
    attr_done[rank] = true;
  }
  Params p;
  auto desc = [](const void* a, unsigned bytes) { return u32x4{(unsigned)(uintptr_t)a, (unsigned)((uintptr_t)a >> 32) & 0xffffu, bytes, 0x00020000u}; };
  p.dx = desc(d_in[0], SHARD);
  p.dc = desc((char*)d_ws_all[rank] + CTRL_OFF, CTRL_BYTES);
  p.w = (const uint8_t*)d_in[1];
  p.sx = (const float*)d_in[2];
  p.sw = (const float*)d_in[3];
  p.myws = (char*)d_ws_all[rank];
  const int quad = 0x4d;
  auto qidx = [&](int r) { int n = 0; for (int q = 0; q < r; ++q) n += ((quad >> q) & 1) == ((quad >> r) & 1); return n; };
  int role[NR], na = 1, nb = 4;
  role[0] = rank;
  for (int j = 1; j < NR; ++j) {
    const int q = (rank + j) & 7;
    if (((quad >> q) & 1) == ((quad >> rank) & 1)) role[na++] = q; else role[nb++] = q;
  }
  p.rk = 0; p.wq = 0;
  for (int i = 0; i < NR; ++i) {
    p.ws[i] = (char*)d_ws_all[role[i]];
    p.out[i] = i == 0 ? (float*)d_out : (float*)d_out_all[role[i]] + (i >= 4 ? qidx(rank) * 128 : 0);
    p.rk |= (unsigned)role[i] << (4 * i);
    if (i >= 4) p.wq |= (unsigned)qidx(role[i]) << (4 * (i - 4));
  }
  build_tasks(p);
  hipLaunchKernelGGL(ag_gemm_kernel, dim3(NBLK), dim3(NTHR), LDS_BYTES, stream, p);
  HIP_CHECK(hipGetLastError());
}
```

```cpp
#include <hip/hip_runtime.h>
#include <rccl/rccl.h>
#include <cstdio>
#include <cstdint>

#define HIP_CHECK(expr) do { hipError_t _e = (expr); if (_e != hipSuccess) { \
    fprintf(stderr, "HIP error %s at %s:%d\n", hipGetErrorString(_e), __FILE__, __LINE__); \
    return; } } while (0)

typedef unsigned u32x4 __attribute__((ext_vector_type(4)));
__device__ u32x4 raw_load128(u32x4 rsrc, int voff, int soff, int aux) __asm("llvm.amdgcn.raw.buffer.load.v4i32");
__device__ unsigned raw_load32(u32x4 rsrc, int voff, int soff, int aux) __asm("llvm.amdgcn.raw.buffer.load.i32");

namespace {

typedef unsigned u32x2 __attribute__((ext_vector_type(2)));
typedef int i32x8 __attribute__((ext_vector_type(8)));
typedef float f32x4 __attribute__((ext_vector_type(4)));
typedef float f32x2 __attribute__((ext_vector_type(2)));
typedef unsigned long long u64;

constexpr int NR = 8;
constexpr int KD = 4096;
constexpr int MLOC = 512;
constexpr int NLOC = 512;
constexpr unsigned SHARD = MLOC * KD;
constexpr unsigned GROUP = 16 * KD;
constexpr unsigned SLOT = NR * SHARD;
constexpr int NSLOT = 3;
constexpr size_t CTRL_OFF = (size_t)NSLOT * SLOT;
constexpr unsigned CTRL_BYTES = 64 * 1024;
constexpr unsigned C_CNT = 256, C_ROLE = 4096, C_FLAG = 16384, C_TICK = 32768;
constexpr int NBLK = 256;
constexpr int NWAVE = 8;
constexpr int NTHR = NWAVE * 64;
constexpr unsigned LDS_BYTES = 96 * 1024;
constexpr int LROW = 264, LSL = 4224;
constexpr u64 DEADLINE = 50000000ull;
constexpr unsigned CODE_LINES = 136;
constexpr int AUX_SYS = 17;
constexpr int AUX_AGENT = 16;

#define PHI 4
#define PSI 2
constexpr int NNO = 32 - PHI - PSI;
constexpr int NI = NNO + PSI;
constexpr int NX = PHI + PSI;
constexpr int NIP = 16;
constexpr int NUNIT = NI * 32;
#define NSTAT 6
#define NDYNP 10
constexpr int NDYN = NDYNP * NIP;
constexpr int NXT = (NUNIT - NSTAT * NIP - NDYN) / 4;
constexpr int NMAX = 120;
#define AHEAD 6
#define AHEAD1 8
#define EARLY 4
#define EARLY0 ((NUNIT - NDYN) * 2 + 128)
#define HSLEEP 2
#define T_S 1050
#define T_P 1200
#define WWIN 1
#define W1WIN 1
#define SWIN 1
#define T_W2 600
#define T_L 600
#define T_X0 150
#define T_X1 450
#define SPECM 0
#define T_SA 1800
#define T_W0 300
#define T_W1 700
#define WSLEEP 4
#define LATEM 150
#define S2EARLY 1
#define STR2(x) #x
#define STR(x) STR2(x)
static_assert(NNO >= 8 && PHI >= 1, "W quarters live in the unused groups of a cross source");

#define STAMP(i) do {} while (0)
#define STAMP_FLUSH() do {} while (0)
#define STAMP_DRAIN() do {} while (0)

struct Params {
  u32x4 dx, dc;
  char* myws;
  const uint8_t* w;
  unsigned rk;
  unsigned wq;
  char* ws[NR];
  const float* sx;
  const float* sw;
  float* out[NR];
  u64 task[2][14][2];
};

__device__ __forceinline__ __amdgpu_buffer_rsrc_t mk_rsrc(const void* p, unsigned bytes) {
  return __builtin_amdgcn_make_buffer_rsrc(const_cast<void*>(p), 0, bytes, 0x00020000);
}
__device__ __forceinline__ unsigned min3u(unsigned a, unsigned b, unsigned c) { return min(min(a, b), c); }
__device__ __forceinline__ unsigned min4(unsigned m, u32x4 v) { return min3u(min3u(m, v[0], v[1]), v[2], v[3]); }
__device__ __forceinline__ u32x4 canon(u32x4 v) {
#pragma unroll
  for (int d = 0; d < 4; ++d) v[d] = v[d] ? v[d] : 0x80808080u;
  return v;
}
__device__ __forceinline__ void st128(u32x4 v, __amdgpu_buffer_rsrc_t r, unsigned voff, unsigned soff) {
  __builtin_amdgcn_raw_buffer_store_b128(v, r, voff, soff, AUX_SYS);
  asm volatile("s_nop 1" :: "v"(v) : "memory");
}
typedef __attribute__((address_space(3))) int lds_int;
typedef __attribute__((address_space(1))) unsigned gu32;
__device__ __forceinline__ void lds_st(lds_int* a, int v) { __hip_atomic_store(a, v, __ATOMIC_RELAXED, __HIP_MEMORY_SCOPE_WORKGROUP); }
__device__ __forceinline__ int lds_ld(lds_int* a) { return __hip_atomic_load(a, __ATOMIC_RELAXED, __HIP_MEMORY_SCOPE_WORKGROUP); }
__device__ __forceinline__ unsigned src_off(int s) { return (unsigned)(s >> 6) * GROUP + (unsigned)(s & 63) * 64; }
__device__ __forceinline__ void tr16(const unsigned (&r)[16], u32x4 (&o)[4]) {
#pragma unroll
  for (int d = 0; d < 4; ++d) {
    const unsigned a0 = r[4 * d], a1 = r[4 * d + 1], a2 = r[4 * d + 2], a3 = r[4 * d + 3];
    const unsigned t0 = __builtin_amdgcn_perm(a1, a0, 0x05010400u), t1 = __builtin_amdgcn_perm(a1, a0, 0x07030602u);
    const unsigned t2 = __builtin_amdgcn_perm(a3, a2, 0x05010400u), t3 = __builtin_amdgcn_perm(a3, a2, 0x07030602u);
    o[0][d] = __builtin_amdgcn_perm(t2, t0, 0x05040100u);
    o[1][d] = __builtin_amdgcn_perm(t2, t0, 0x07060302u);
    o[2][d] = __builtin_amdgcn_perm(t3, t1, 0x05040100u);
    o[3][d] = __builtin_amdgcn_perm(t3, t1, 0x07060302u);
  }
}

__global__ void __launch_bounds__(NTHR) ag_gemm_kernel(Params p) {
  extern __shared__ __attribute__((aligned(16))) char smem[];
  const u32x4 dx = p.dx, dc = p.dc;
  char* myws = p.myws;
  const uint8_t* pw = p.w;
  unsigned rk = p.rk, wq = p.wq;
  __builtin_amdgcn_sched_barrier(0);
  const int wid = __builtin_amdgcn_readfirstlane(threadIdx.x >> 6);
  const int lane = threadIdx.x & 63;
  const int bid = blockIdx.x;
  const int cu = bid >> 3;
  const unsigned icode = 0;
  const u64 t0 = __builtin_amdgcn_s_memrealtime();
  int xcd = __builtin_amdgcn_s_getreg(20 | (3 << 11)) & 7;
  auto peer = [&]() -> int { return xcd < 4 ? (0x21321321 >> (4 * wid)) & 3 : xcd; };
  const int li = lane & 15, lg = lane >> 4;
  unsigned vsrc = (unsigned)li * KD + (unsigned)lg * 16;
  const unsigned vlin = (unsigned)lane * 16;
  unsigned vw = (unsigned)lg * 16 * NLOC + (unsigned)li * 4;
  const int h = (wid + 5) >> 3;
  const int s5 = cu * 2 + (wid & 1) + (wid >> 1) * 8;
  static_assert(NIP == 16, "the first two pieces of a wave are 32 apart, in one group");
  auto sfirst = [&]() -> unsigned {
    const unsigned m = 0u - (unsigned)(xcd >> 2);
    return ((unsigned)(2 * (cu * 4 + xcd) + h) * 64 & ~m) | ((NI * GROUP + (unsigned)s5 * 64) & m);
  };
  unsigned so0 = sfirst();
  unsigned cnt = C_CNT + (unsigned)(bid * NWAVE + wid) * 4;
  unsigned so1 = so0 + GROUP;
  unsigned vw2 = vw + 8 * NLOC;
  asm volatile("" : "+s"(so0), "+s"(so1), "+s"(cnt), "+v"(vsrc), "+v"(vw), "+v"(vw2));
  __builtin_amdgcn_sched_barrier(0);
  u32x4 r0[4];
  unsigned craw;
  if (__builtin_expect(cu < 4, 1)) {
    r0[0] = raw_load128(dx, vsrc, so0, 0);
    __builtin_amdgcn_sched_barrier(0);
    craw = raw_load32(dc, 0, cnt, AUX_AGENT);
    __builtin_amdgcn_sched_barrier(0);
    r0[1] = raw_load128(dx, vsrc + 2048, so0, 0);
    __builtin_amdgcn_sched_barrier(0);
    r0[2] = raw_load128(dx, vsrc, so1, 0);
    __builtin_amdgcn_sched_barrier(0);
    r0[3] = raw_load128(dx, vsrc + 2048, so1, 0);
    __builtin_amdgcn_sched_barrier(0);
  }
  char* pws = p.ws[peer()];
  const uint8_t* px = (const uint8_t*)((u64)dx[1] << 32 | dx[0]);
  const __amdgpu_buffer_rsrc_t rc = mk_rsrc(myws + CTRL_OFF, CTRL_BYTES);
  const __amdgpu_buffer_rsrc_t rxs = mk_rsrc(px, SHARD);
  const unsigned rank = rk & 7u;
  const __amdgpu_buffer_rsrc_t rw = mk_rsrc(pw, KD * NLOC);
  gu32* const roles = (gu32*)(myws + CTRL_OFF + C_ROLE) + cu * 32 + (lane & 7);
  lds_int* const verdict = (lds_int*)(smem + 16 * LSL);
  unsigned took = 0;
  const bool asker = wid == (cu < 4 ? 7 : 0);
  auto ask = [&]() { if (asker && lane < 8) took = __hip_atomic_fetch_add(roles, lane == xcd ? 1u : 0u, __ATOMIC_RELAXED, __HIP_MEMORY_SCOPE_AGENT); };
  auto resolve = [&]() {
    if (!asker) return;
    int r = xcd;
    unsigned c = __builtin_amdgcn_readlane(took, r);
    unsigned behind = (unsigned)__builtin_amdgcn_ballot_w64((int)(took - c) < 0) & 255u;
    while (__builtin_expect(behind != 0, 0)) {
      const int rn = __builtin_ctz(behind);
      if (lane < 8) took = __hip_atomic_fetch_add(roles, lane == rn ? 1u : lane == r ? ~0u : 0u, __ATOMIC_RELAXED, __HIP_MEMORY_SCOPE_AGENT);
      r = rn;
      c = __builtin_amdgcn_readlane(took, r);
      behind = (unsigned)__builtin_amdgcn_ballot_w64((int)(took - c) < 0) & 255u;
    }
    lds_st(verdict + (lane & 1), lane & 1 ? r : (int)c + 1);
  };
  auto meet = [&]() -> int {
    asm volatile("s_waitcnt lgkmcnt(0)" ::: "memory");
    __builtin_amdgcn_s_barrier();
    asm volatile("" ::: "memory");
    return lds_ld(verdict + (lane & 1));
  };
  if (__builtin_expect(cu < 4, 1)) {
    lds_int* const pos = (lds_int*)smem;
    lds_int* const done = pos + 192;
    auto dfirst = [&](unsigned so) -> unsigned { return (so & ~(GROUP - 1)) | (so & (GROUP - 1)) * 16; };
    unsigned do0 = dfirst(so0), vd0 = xcd < 4 && wid >= 6 ? SHARD : vlin;
    char* pdst = pws + (size_t)rank * SHARD;
    asm volatile("" : "+s"(do0), "+v"(vd0), "+s"(pdst));
    __builtin_amdgcn_sched_barrier(0);
    const unsigned call = __builtin_amdgcn_readfirstlane(craw) + 1;
    __amdgpu_buffer_rsrc_t rd = mk_rsrc(pdst + (size_t)(call % NSLOT) * SLOT, SHARD);
    st128(canon(r0[0]), rd, vd0, do0);
    STAMP(2);
    __builtin_amdgcn_sched_barrier(0);
    st128(canon(r0[1]), rd, vd0, do0 + 32768);
    __builtin_amdgcn_sched_barrier(0);
#pragma unroll 1
    for (bool again = false;; again = true) {
      if (__builtin_expect(xcd < 4, 1)) {
        const int pidx = cu * 4 + xcd;
        if (wid == 6) { lds_st(pos + lane, -1); lds_st(pos + lane + 64, -1); lds_st(pos + lane + 128, -1); lds_st(done + lane, lane >= 6 ? 0x7fffffff : 0); }
        asm volatile("s_waitcnt lgkmcnt(0)" ::: "memory");
        __builtin_amdgcn_s_barrier();
        __builtin_amdgcn_sched_barrier(0);
        if (__builtin_expect(wid < 6, 1)) {
          lds_st(done + wid, 2);
          asm volatile("s_waitcnt vmcnt(1)" ::: "memory");
          st128(canon(r0[2]), rd, vlin, (unsigned)(2 * (pidx + 2 * NIP) + h) * 1024);
          __builtin_amdgcn_sched_barrier(0);
          u32x4 r[2] = {r0[2], r0[3]};
          int up[2] = {pidx + 2 * NIP, pidx + 3 * NIP};
#pragma unroll 1
          for (int n0 = 2; up[0] < NUNIT; n0 += 2) {
#pragma unroll
            for (int i = 0; i < 2; ++i) {
              const int n = n0 + i;
              lds_st(done + wid, n + 1);
              int un = pidx + (n + 2) * NIP;
              if (n + 2 >= NSTAT) {
                for (;;) {
                  un = __builtin_amdgcn_readfirstlane(lds_ld(pos + n + 2));
                  if (un >= 0) break;
                  __builtin_amdgcn_s_sleep(1);
                }
              }
              r[i] = __builtin_amdgcn_raw_buffer_load_b128(rxs, vsrc, src_off(2 * un + h), 0);
              up[i] = un;
              __builtin_amdgcn_sched_barrier(0);
              const int u = up[i ^ 1];
              st128(canon(r[i ^ 1]), rd, u < NUNIT ? vlin : SHARD, (unsigned)(2 * u + h) * 1024);
              __builtin_amdgcn_sched_barrier(0);
            }
          }
          STAMP(5);
          STAMP_DRAIN();
        } else if (wid == 7) {
          if (!again) ask();
          if (pidx == 0) __builtin_amdgcn_raw_buffer_store_b32(0u, rc, (unsigned)(lane & 7) * 128, C_TICK + ((call + 1) & 1) * 1024, AUX_AGENT);
          gu32* tick = (gu32*)(myws + CTRL_OFF + C_TICK + (call & 1) * 1024);
          int n = NSTAT;
          bool mine = true;
          for (;;) {
            int d = lds_ld(done + (lane & 7));
            d = min(d, __builtin_amdgcn_update_dpp(0x7fffffff, d, 0x111, 0xf, 0xf, false));
            d = min(d, __builtin_amdgcn_update_dpp(0x7fffffff, d, 0x112, 0xf, 0xf, false));
            d = min(d, __builtin_amdgcn_update_dpp(0x7fffffff, d, 0x114, 0xf, 0xf, false));
            const int lim = min(__builtin_amdgcn_readlane(d, 7) + (mine ? AHEAD1 : AHEAD), NMAX);
            const int k = lim - n;
            if (k <= 0) {
              if (n >= NMAX) { lds_st(pos + n + (lane & 3), NUNIT); break; }
              __builtin_amdgcn_s_sleep(2);
              continue;
            }
            unsigned t = 0;
            if (lane == 0) t = __hip_atomic_fetch_add(tick + (mine ? xcd * 32 : 128), (unsigned)k, __ATOMIC_RELAXED, __HIP_MEMORY_SCOPE_AGENT);
            if (mine) {
              const int t0 = (int)min(__builtin_amdgcn_readfirstlane(t), (unsigned)NXT);
              const int kk = min(k, NXT - t0);
              if (lane < kk) lds_st(pos + n + lane, NSTAT * NIP + (t0 + lane) * 4 + xcd);
              n += kk;
              mine = t0 + k < NXT;
            } else {
              const int t0 = (int)min(__builtin_amdgcn_readfirstlane(t), (unsigned)NDYN);
              const bool last = t0 + k >= NDYN;
              if (lane < (last ? k + 4 : k)) lds_st(pos + n + lane, min(NUNIT - NDYN + t0 + lane, NUNIT));
              if (last) break;
              n += k;
            }
          }
        }
        STAMP(6);
      } else {
        constexpr int NPW = NX * 2;
        auto spos = [&](int n) -> int { const int g = n >> 1; return (g < PHI ? NI + g : NNO + g - PHI) * 64 + (n & 1) * 32 + s5; };
        u32x4 r[NPW];
        static_assert(PHI >= 2, "the first four pieces of a wave");
        r[2] = r0[2]; r[3] = r0[3];
#pragma unroll
        for (int n = 2; n < 2 + S2EARLY; ++n) {
          asm volatile("s_waitcnt vmcnt(1)" ::: "memory");
          st128(canon(r[n]), rd, vlin, (unsigned)spos(n) * 1024);
          __builtin_amdgcn_sched_barrier(0);
        }
        constexpr int N1 = 2 + S2EARLY;
        __builtin_amdgcn_s_barrier();
        __builtin_amdgcn_sched_barrier(0);
        if (!again) ask();
#pragma unroll
        for (int n = 4; n < NPW; ++n) r[n] = __builtin_amdgcn_raw_buffer_load_b128(rxs, vsrc, src_off(spos(n)), 0);
#pragma unroll
        for (int n = 2; n < NPW; ++n) r[n] = canon(r[n]);
        STAMP(1);
#pragma unroll
        for (int n = N1; n < NPW; ++n) {
          if (n == N1 || n == 2 * PHI) {
            while (__builtin_amdgcn_s_memrealtime() - t0 < (n == 2 * PHI ? T_P : T_S)) __builtin_amdgcn_s_sleep(2);
            asm volatile("" ::: "memory");
          }
          st128(r[n], rd, vlin, (unsigned)spos(n) * 1024);
          asm volatile("s_waitcnt vmcnt(" STR(SWIN) ")" ::: "memory");
        }
        STAMP_DRAIN();
        STAMP(5);
      }
      if (again) break;
      resolve();
      const int role = __builtin_amdgcn_readlane(meet(), 1);
      if (__builtin_expect(role == xcd, 1)) break;
      xcd = role;
      rd = mk_rsrc(p.ws[peer()] + (size_t)(call % NSLOT) * SLOT + (size_t)rank * SHARD, SHARD);
#pragma unroll
      for (int i = 0; i < 4; ++i) r0[i] = __builtin_amdgcn_raw_buffer_load_b128(rxs, vsrc + (unsigned)(i & 1) * 2048, sfirst() + (unsigned)(i >> 1) * GROUP, 0);
#pragma unroll
      for (int i = 0; i < 2; ++i) st128(canon(r0[i]), rd, xcd < 4 && wid >= 6 ? SHARD : vlin, dfirst(sfirst()) + (unsigned)i * 32768);
    }
    asm volatile("" :: "v"(icode));
    __builtin_amdgcn_raw_buffer_store_b32(call, rc, 0, cnt, AUX_AGENT);
    if (cu == 0 && wid == 0 && xcd >= 4) {
      const unsigned fo = C_FLAG + ((rk >> (4 * xcd)) & 7u) * 256;
      for (unsigned n = 1;; ++n) {
        const unsigned f = __builtin_amdgcn_raw_buffer_load_b32(rc, (unsigned)min(lane, 27) * 4, fo, AUX_SYS);
        if (__builtin_amdgcn_ballot_w64(f != call) == 0ull) break;
        if ((n & 63) == 0 && __builtin_amdgcn_s_memrealtime() - t0 > DEADLINE) {
          __builtin_amdgcn_raw_buffer_store_b32(call, rc, 0, 0, AUX_AGENT);
          p.out[0][(size_t)rank * MLOC * NLOC + lane] = __builtin_nanf("");
          break;
        }
        __builtin_amdgcn_s_sleep(1);
        asm volatile("" ::: "memory");
      }
    }
    STAMP_FLUSH();
    return;
  }

  ask();
  __builtin_amdgcn_sched_barrier(0);
  unsigned warm = 0;
  unsigned raw0[2][16];
  auto first_loads = [&](bool first) {
    if (xcd < 4) {
      const int wi = wid * 28 + cu - 4;
      if (wi < NI * 4)
        warm = __builtin_amdgcn_raw_buffer_load_b32(rxs, (unsigned)(lane >> 4) * KD + (unsigned)((lane >> 1) & 7) * 512 + (unsigned)(lane & 1) * 64,
                                                    (unsigned)wi * 4 * KD + (unsigned)xcd * 128, 0);
      if (first) resolve();
    } else if (cu < 12) {
      const unsigned so = (unsigned)((cu - 4) * 8 + wid) * 64 * NLOC + ((wq >> (4 * (xcd - 4))) & 3u) * 128;
#pragma unroll
      for (int hh = 0; hh < 2; ++hh)
#pragma unroll
        for (int i = 0; i < 16; ++i) raw0[hh][i] = __builtin_amdgcn_raw_buffer_load_b32(rw, (i < 8 ? vw : vw2) + (unsigned)((i & 7) * NLOC + hh * 64), so, 0);
      __builtin_amdgcn_sched_barrier(0);
      if (first) resolve();
    } else {
      if (cu < 28)
        warm = __builtin_amdgcn_raw_buffer_load_b32(rw, (unsigned)(lane >> 1) * NLOC + (unsigned)(lane & 1) * 64,
                                                    (unsigned)(((cu - 12) & 7) * 16 + wid * 2 + ((cu - 12) >> 3)) * 32 * NLOC + ((wq >> (4 * (xcd - 4))) & 3u) * 128, 0);
      if (first) resolve();
    }
    __builtin_amdgcn_sched_barrier(0);
  };
  first_loads(true);
  const int vd = meet();
  const unsigned call = __builtin_amdgcn_readlane(vd, 0);
  warm |= icode;
  if (__builtin_expect(__builtin_amdgcn_readlane(vd, 1) != xcd, 0)) {
    xcd = __builtin_amdgcn_readlane(vd, 1);
    pws = p.ws[peer()];
    first_loads(false);
  }
  const bool cross = xcd >= 4;
  if (cross && cu < 12) {
    const __amdgpu_buffer_rsrc_t rd = mk_rsrc(pws + (size_t)(call % NSLOT) * SLOT + (size_t)rank * SHARD, SHARD);
    const int kb = (cu - 4) * 8 + wid;
    u32x4 o[2][4];
    tr16(raw0[0], o[0]);
    tr16(raw0[1], o[1]);
#pragma unroll
    for (int e = 0; e < 8; ++e) {
      if (e > WWIN) asm volatile("s_waitcnt vmcnt(" STR(WWIN) ")" ::: "memory");
      st128(canon(o[e & 1][e >> 1]), rd, vlin + (unsigned)(e >> 1) * 1024, (unsigned)((e & 1) * 64 + kb) * 4096);
      __builtin_amdgcn_sched_barrier(0);
      if (e == 0) STAMP(2);
    }
    STAMP(7);
    __builtin_amdgcn_s_barrier();
  }
  __builtin_amdgcn_sched_barrier(0);

  const int c28 = cu - 4;
  const int ci = xcd * 28 + c28;
  const int strm = cross ? c28 >> 1 : ci >> 3;
  const int slab = cross ? c28 & 1 : ci & 7;
  u64 tl = p.task[cross][strm][0], th = p.task[cross][strm][1];
  typedef const __attribute__((address_space(4))) float cf32;
  float sx = *(cf32*)p.sx, sw = *(cf32*)p.sw;
  float* po = p.out[cross ? xcd : 0];
  bool dead = false;
  auto expired = [&]() -> bool {
    return __builtin_amdgcn_readfirstlane(__builtin_amdgcn_raw_buffer_load_b32(rc, 0, 0, AUX_AGENT)) == call ||
           __builtin_amdgcn_s_memrealtime() - t0 > DEADLINE;
  };
  auto next = [&]() -> int {
    const int c = (int)(tl & 255u);
    tl = (tl >> 8) | (th << 56); th = (th >> 8) | (63ull << 56);
    if (c == 63) return -1;
    int role = c >> 5;
    if (cross && role >= 4) role = 4 + ((role + xcd) & 3);
    return (int)(((rk >> (4 * role)) & 7u) << 8) | c;
  };
  u32x4 xv[8];
  auto issue_own = [&](int code) {
    const __amdgpu_buffer_rsrc_t r = rxs;
    const unsigned so = (unsigned)(code & 31) * GROUP + (unsigned)wid * 512;
#pragma unroll
    for (int q = 0; q < 8; ++q) xv[q] = __builtin_amdgcn_raw_buffer_load_b128(r, vsrc + (unsigned)q * 64, so, 0);
  };
  int code = next();
  u32x4 wf[8][4];
  if (!cross) {
    const unsigned vo = vw + (unsigned)wid * 512 * NLOC + (unsigned)slab * 64;
    unsigned raw[3][16];
#pragma unroll
    for (int u = 0; u < 10; ++u) {
      if (u < 8) {
#pragma unroll
        for (int i = 0; i < 16; ++i) raw[u % 3][i] = __builtin_amdgcn_raw_buffer_load_b32(rw, vo, (unsigned)(u * 64 + i) * NLOC, 0);
      }
      if (u == 8) issue_own(code);
      __builtin_amdgcn_sched_barrier(0);
      if (u >= 2) tr16(raw[(u - 2) % 3], wf[u - 2]);
      __builtin_amdgcn_sched_barrier(0);
    }
    asm volatile("" : "+s"(sx), "+s"(sw), "+s"(po) : "v"(warm));
  } else {
    const __amdgpu_buffer_rsrc_t rq = mk_rsrc(myws + (size_t)(call % NSLOT) * SLOT + (size_t)((rk >> (4 * xcd)) & 7u) * SHARD +
                                              (size_t)(slab * 64 + wid * 8) * 4096, 32768);
    const unsigned vq = (unsigned)(lane & 7) * 4096 + (unsigned)((lane >> 3) & 3) * 1024 + 1020;
    unsigned got = 0;
    asm volatile("" : "+s"(sx), "+s"(sw), "+s"(po) : "v"(warm));
    while (__builtin_amdgcn_s_memrealtime() - t0 < T_W0) __builtin_amdgcn_s_sleep(8);
    asm volatile("" ::: "memory");
    issue_own(code);
#pragma unroll 1
    for (unsigned n = 1; got != 15 && !dead; ++n) {
      __builtin_amdgcn_s_sleep(WSLEEP);
      const unsigned hh = __builtin_amdgcn_raw_buffer_load_b32(rq, vq, 0, AUX_SYS);
      const unsigned miss = (unsigned)__builtin_amdgcn_ballot_w64(hh == 0u);
#pragma unroll
      for (int e = 0; e < 4; ++e) {
        if (!((got >> e) & 1) && !((miss >> (8 * e)) & 255u)) {
          got |= 1u << e;
#pragma unroll
          for (int q = 0; q < 8; ++q) wf[q][e] = __builtin_amdgcn_raw_buffer_load_b128(rq, vlin + (unsigned)e * 1024, (unsigned)q * 4096, AUX_SYS);
        }
      }
      if ((n & 63) == 0 && expired()) dead = true;
      asm volatile("" ::: "memory");
    }
    STAMP(3);
    for (unsigned n = 1;; ++n) {
      unsigned m = 0xffffffffu;
#pragma unroll
      for (int q = 0; q < 8; ++q)
#pragma unroll
        for (int e = 0; e < 4; ++e) m = min4(m, wf[q][e]);
      if (__builtin_expect(__builtin_amdgcn_ballot_w64(m == 0u) == 0ull || dead, 1)) break;
      if ((n & 63) == 0 && expired()) dead = true;
#pragma unroll
      for (int q = 0; q < 8; ++q)
#pragma unroll
        for (int e = 0; e < 4; ++e) wf[q][e] = __builtin_amdgcn_raw_buffer_load_b128(rq, vlin + (unsigned)e * 1024, (unsigned)q * 4096, AUX_SYS);
      asm volatile("" ::: "memory");
    }
  }
  STAMP(1);
  float scale = dead ? __builtin_nanf("") : sx * sw;
  const __amdgpu_buffer_rsrc_t rs = mk_rsrc(myws + (size_t)(call % NSLOT) * SLOT, SLOT);
  const __amdgpu_buffer_rsrc_t rz = mk_rsrc(myws + (size_t)((call + 2) % NSLOT) * SLOT, SLOT);
  const __amdgpu_buffer_rsrc_t ro = mk_rsrc(po, NR * MLOC * NLOC * 4 - (cross ? 3 * 512 : 0));
  const unsigned vh = (unsigned)(lane & 7) * 1024 + 1020;
  const int orow = 2 * wid + (lane >> 5);
  const unsigned vout = (unsigned)orow * (NLOC * 4) + (unsigned)slab * 256 + (unsigned)(lane & 31) * 8;
  typedef __attribute__((address_space(3))) char lds_char;
  lds_char* const lw = (lds_char*)smem + wid * LSL + li * LROW + lg * 64;
  const lds_char* const lr = (lds_char*)smem + orow * LROW + (lane & 31) * 8;
  const unsigned vz = cross ? 0x80000000u : vlin;
  const char* const xslot = myws + (size_t)(call % NSLOT) * SLOT;
  auto when = [&](int code) -> int {
    const int g = code & 31;
    int a = 213 - SPECM + 113 * g, b = g == NI ? 400 : g > NI ? T_SA : T_P + 300;
    asm volatile("" : "+s"(a), "+s"(b));
    a = code & 0x80 ? b : a;
    a = code & 0xe0 ? a : 0;
    return code < 0 ? 0x7fffffff : a;
  };
  __builtin_amdgcn_sched_barrier(0);
  int buf = 0;
  bool again = false;
  __builtin_amdgcn_raw_buffer_store_b32(0u, ro, 0x80000000u, 0, AUX_SYS);
  __builtin_amdgcn_sched_barrier(0);
#pragma unroll 1
  do {
    const int ncode = next();
    const bool nown = (ncode & 0xe0) == 0;
    unsigned nstep = nown ? 64 : 1024;
    unsigned nso = (unsigned)(ncode & 31) * GROUP + (unsigned)wid * 8 * nstep;
    unsigned nvo = nown ? vsrc : vlin;
    const char* nbase = nown ? (const char*)px : xslot + (size_t)((ncode >> 8) & 7) * SHARD;
    int nwhen = when(ncode);
    unsigned zo = (unsigned)(code >> 8) * SHARD + (unsigned)(code & 31) * GROUP, zw = ((rk >> (16 + ((code >> 1) & 12))) & 7u) * SHARD + (unsigned)(code & 7) * GROUP;
    asm volatile("" : "+s"(zo), "+s"(zw));
    zo = (code & 0xe0 ? zo : zw) + (unsigned)(slab * 8 + wid) * 1024;
    unsigned oo = ((unsigned)(code >> 8) * MLOC + (unsigned)(code & 31) * 16) * (NLOC * 4);
    lds_char* lwb = lw + buf;
    const lds_char* lrb = lr + buf;
    int far = ncode < 0 ? 0 : ncode & 0xe0;
    asm volatile("" : "+s"(nstep), "+s"(nso), "+s"(nbase), "+s"(nwhen), "+s"(zo), "+s"(oo), "+s"(far), "+s"(tl), "+s"(th));
    asm volatile("" : "+v"(nvo), "+v"(lwb), "+v"(lrb));
    __builtin_amdgcn_sched_barrier(0);
    asm volatile("" :: "v"(xv[0]), "v"(xv[1]), "v"(xv[2]), "v"(xv[3]), "v"(xv[4]), "v"(xv[5]), "v"(xv[6]), "v"(xv[7]));
    __builtin_amdgcn_sched_barrier(0);
    if (__builtin_expect(again, 0)) {
      unsigned m = 0xffffffffu, mx = 0;
#pragma unroll
      for (int q = 0; q < 8; ++q) { m = min4(m, xv[q]); mx |= xv[q][3]; }
      if (__builtin_expect(__builtin_amdgcn_ballot_w64(m == 0u) != 0ull && !dead, 0)) {
        const int s = (code & 31) * 64 + wid * 8;
        const unsigned so = (unsigned)(code >> 8) * SHARD + (unsigned)s * 1024;
        const unsigned ho = (code & 0x80) == 0 && s >= EARLY0 ? so - EARLY * 8192 : so;
        bool poll = __builtin_amdgcn_ballot_w64(mx != 0u) == 0ull;
        for (unsigned n = 1;; ++n) {
          if (poll) {
            const unsigned hh = __builtin_amdgcn_raw_buffer_load_b32(rs, vh, ho, AUX_SYS);
            poll = __builtin_amdgcn_ballot_w64(hh == 0u) != 0ull;
            if (poll) __builtin_amdgcn_s_sleep(HSLEEP);
          }
          if (!poll) {
#pragma unroll
            for (int q = 0; q < 8; ++q)
                xv[q] = __builtin_amdgcn_raw_buffer_load_b128(rs, vlin + (unsigned)(q & 3) * 1024, so + (unsigned)(q & 4) * 1024, AUX_SYS);
            m = 0xffffffffu;
#pragma unroll
            for (int q = 0; q < 8; ++q) m = min4(m, xv[q]);
            if (__builtin_amdgcn_ballot_w64(m == 0u) == 0ull) break;
            if (n > 64) __builtin_amdgcn_s_sleep(16);
          }
          if ((n & 63) == 0 && expired()) { dead = true; break; }
          asm volatile("" ::: "memory");
        }
        if (dead) { __builtin_amdgcn_raw_buffer_store_b32(call, rc, 0, 0, AUX_AGENT); scale = __builtin_nanf(""); }
      }
    }
    unsigned now = (unsigned)__builtin_amdgcn_s_memrealtime();
    __builtin_amdgcn_sched_barrier(0);
    __amdgpu_buffer_rsrc_t rn;
    f32x4 acc[4];
#pragma unroll
    for (int e = 0; e < 4; ++e) acc[e] = f32x4{0.f, 0.f, 0.f, 0.f};
    auto fetch = [&](int q) { xv[q] = __builtin_amdgcn_raw_buffer_load_b128(rn, nvo, nso + (unsigned)q * nstep, AUX_SYS); };
    auto part = [&](int e) {
#pragma unroll
      for (int r = 0; r < 4; ++r) *(__attribute__((address_space(3))) float*)(lwb + r * 16 + e * 4) = acc[e][r];
    };
#pragma unroll
    for (int ks = 0; ks < 4; ++ks) {
      const i32x8 b = {(int)xv[2 * ks][0], (int)xv[2 * ks][1], (int)xv[2 * ks][2], (int)xv[2 * ks][3],
                       (int)xv[2 * ks + 1][0], (int)xv[2 * ks + 1][1], (int)xv[2 * ks + 1][2], (int)xv[2 * ks + 1][3]};
#pragma unroll
      for (int e = 0; e < 4; ++e) {
        const u32x4 a0 = wf[2 * ks][e], a1 = wf[2 * ks + 1][e];
        const i32x8 a = {(int)a0[0], (int)a0[1], (int)a0[2], (int)a0[3], (int)a1[0], (int)a1[1], (int)a1[2], (int)a1[3]};
        acc[e] = __builtin_amdgcn_mfma_scale_f32_16x16x128_f8f6f4(a, b, acc[e],   1,   0, 0, 0, 0, 0);
        __builtin_amdgcn_sched_barrier(0);
        if (ks == 0 && e == 0) st128(u32x4{0u, 0u, 0u, 0u}, rz, vz, zo);
        if (ks == 0 && e == 3) {
          asm volatile("" : "+s"(now));
          rn = mk_rsrc(nbase, (int)(now - (unsigned)t0) >= nwhen ? SHARD : 0);
        }
        if (ks > 0 && e < 2) fetch(2 * ks - 2 + e);
        if (ks == 3 && e >= 2) part(e - 2);
        __builtin_amdgcn_sched_barrier(0);
      }
    }
    part(2);
    __builtin_amdgcn_sched_barrier(0);
    part(3);
    __builtin_amdgcn_sched_barrier(0);
    fetch(6);
    fetch(7);
    __builtin_amdgcn_sched_barrier(0);
    asm volatile("s_waitcnt lgkmcnt(0)" ::: "memory");
    __builtin_amdgcn_s_barrier();
    asm volatile("" ::: "memory");
    f32x2 pr[8], ov;
#pragma unroll
    for (int v = 0; v < 8; ++v) pr[v] = *(const __attribute__((address_space(3))) f32x2*)(lrb + v * LSL);
    const f32x2 y = (((pr[0] + pr[1]) + (pr[2] + pr[3])) + ((pr[4] + pr[5]) + (pr[6] + pr[7]))) * scale;
#pragma unroll
    for (int r = 0; r < 2; ++r) ov[r] = y[r] * __builtin_amdgcn_rcpf(1.f + __expf(-__builtin_amdgcn_fmed3f(y[r], -60.f, 60.f)));
    asm volatile("" : "+v"(ov));
    __builtin_amdgcn_sched_barrier(0);
    again = false;
    if (far) {
      unsigned m = 0xffffffffu;
#pragma unroll
      for (int q = 0; q < 8; ++q) m = min4(m, xv[q]);
      again = __builtin_amdgcn_ballot_w64(m == 0u) != 0ull;
      if (again && (int)(__builtin_amdgcn_s_memrealtime() - t0) + LATEM >= nwhen) {
        const unsigned so = (unsigned)(ncode >> 8) * SHARD + nso;
#pragma unroll
        for (int q = 0; q < 8; ++q) xv[q] = __builtin_amdgcn_raw_buffer_load_b128(rs, vlin + (unsigned)(q & 3) * 1024, so + (unsigned)(q & 4) * 1024, AUX_SYS);
      }
      if (cross) __builtin_amdgcn_s_barrier();
    }
    __builtin_amdgcn_sched_barrier(0);
    __builtin_amdgcn_raw_buffer_store_b64(__builtin_bit_cast(u32x2, ov), ro, vout, oo, AUX_SYS);
    __builtin_amdgcn_sched_barrier(0);
    code = ncode;
    buf = 8 * LSL - buf;
  } while (code >= 0);
  STAMP(5);
  if (cross) {
    asm volatile("s_waitcnt vmcnt(0)" ::: "memory");
    __builtin_amdgcn_s_barrier();
    if (wid == 0) __builtin_amdgcn_raw_buffer_store_b32(call, mk_rsrc(pws + CTRL_OFF, CTRL_BYTES), 0, C_FLAG + rank * 256 + (unsigned)c28 * 4, AUX_SYS);
  }
  STAMP_DRAIN();
  STAMP(6);
  STAMP_FLUSH();
}

#define DUR0 1.2f
#define DUR1 2.0f
#define BEG0 8.f
#define BEG2 13.9f
#define BEG1 12.f
void build_tasks(Params& p) {
  for (int t = 0; t < 2; ++t) {
    float when[168];
    uint8_t what[168];
    int n = 0;
    auto add = [&](float w, int role, int g) {
      int i = n++;
      for (; i > 0 && when[i - 1] > w; --i) { when[i] = when[i - 1]; what[i] = what[i - 1]; }
      when[i] = w; what[i] = (uint8_t)(role << 5 | g);
    };
    for (int g = 0; g < (t ? NNO : 32); ++g) add(0.f, 0, g);
    for (int g = 0; g < (t ? NNO : NI); ++g) for (int r = 1; r < 4; ++r) add(1.f + 1.13f * (g + 1), r, g);
    for (int i = 0; i < (t ? PHI : NX); ++i)
      for (int r = t ? 5 : 4; r < 8; ++r)
        add(i == 0 ? 3.f : i < PHI ? T_SA * 0.01f : T_P * 0.01f + 1.5f * (i - PHI + 1), r, i < PHI ? NI + i : NNO + i - PHI);
    float busy[14];
    int len[14];
    uint8_t lst[14][16];
    for (int j = 0; j < 14; ++j) { busy[j] = t ? (j < 4 ? BEG2 : BEG1) : BEG0; len[j] = 0; }
    for (int i = 0; i < n; ++i) {
      int b = -1;
      for (int j = 0; j < 14; ++j) if (len[j] < 15 && (b < 0 || busy[j] < busy[b])) b = j;
      busy[b] = (busy[b] > when[i] ? busy[b] : when[i]) + (t ? DUR1 : DUR0);
      lst[b][len[b]++] = what[i];
    }
    for (int j = 0; j < 14; ++j) {
      u64 v[2] = {0, 0};
      const int o = 0;
      for (int i = 0; i < 16; ++i) v[i >> 3] |= (u64)(i >= o && i - o < len[j] ? lst[j][i - o] : 63) << (8 * (i & 7));
      p.task[t][j][0] = v[0]; p.task[t][j][1] = v[1];
    }
  }
}

}

extern "C" void kernel_launch(int rank, int nranks, ncclComm_t comm,
                              hipStream_t stream,
                              void* const* d_in, const int* in_sizes, int n_in,
                              void* d_out, int out_size,
                              const int* in_dtypes, int out_dtype,
                              void* const* d_out_all,
                              void* const* d_ws_all, size_t ws_bytes) {
  (void)comm; (void)in_dtypes; (void)out_dtype;
  if (nranks != NR || (unsigned)rank >= (unsigned)NR || n_in < 4 || in_sizes[0] != MLOC * KD || in_sizes[1] != KD * NLOC ||
      out_size != NR * MLOC * NLOC || ws_bytes < CTRL_OFF + CTRL_BYTES) {
    fprintf(stderr, "kernel_launch: unexpected problem shape\n");
    return;
  }
  static bool attr_done[NR];
  if (!attr_done[rank]) {
    HIP_CHECK(hipFuncSetAttribute((const void*)ag_gemm_kernel, hipFuncAttributeMaxDynamicSharedMemorySize, LDS_BYTES));
    attr_done[rank] = true;
  }
  Params p;
  auto desc = [](const void* a, unsigned bytes) { return u32x4{(unsigned)(uintptr_t)a, (unsigned)((uintptr_t)a >> 32) & 0xffffu, bytes, 0x00020000u}; };
  p.dx = desc(d_in[0], SHARD);
  p.dc = desc((char*)d_ws_all[rank] + CTRL_OFF, CTRL_BYTES);
  p.w = (const uint8_t*)d_in[1];
  p.sx = (const float*)d_in[2];
  p.sw = (const float*)d_in[3];
  p.myws = (char*)d_ws_all[rank];
  const int quad = 0x4d;
  auto qidx = [&](int r) { int n = 0; for (int q = 0; q < r; ++q) n += ((quad >> q) & 1) == ((quad >> r) & 1); return n; };
  int role[NR], na = 1, nb = 4;
  role[0] = rank;
  for (int j = 1; j < NR; ++j) {
    const int q = (rank + j) & 7;
    if (((quad >> q) & 1) == ((quad >> rank) & 1)) role[na++] = q; else role[nb++] = q;
  }
  p.rk = 0; p.wq = 0;
  for (int i = 0; i < NR; ++i) {
    p.ws[i] = (char*)d_ws_all[role[i]];
    p.out[i] = i == 0 ? (float*)d_out : (float*)d_out_all[role[i]] + (i >= 4 ? qidx(rank) * 128 : 0);
    p.rk |= (unsigned)role[i] << (4 * i);
    if (i >= 4) p.wq |= (unsigned)qidx(role[i]) << (4 * (i - 4));
  }
  build_tasks(p);
  hipLaunchKernelGGL(ag_gemm_kernel, dim3(NBLK), dim3(NTHR), LDS_BYTES, stream, p);
  HIP_CHECK(hipGetLastError());
}
```

```cpp
#include <hip/hip_runtime.h>
#include <rccl/rccl.h>
#include <cstdio>
#include <cstdint>

#define HIP_CHECK(expr) do { hipError_t _e = (expr); if (_e != hipSuccess) { \
    fprintf(stderr, "HIP error %s at %s:%d\n", hipGetErrorString(_e), __FILE__, __LINE__); \
    return; } } while (0)

typedef unsigned u32x4 __attribute__((ext_vector_type(4)));
__device__ u32x4 raw_load128(u32x4 rsrc, int voff, int soff, int aux) __asm("llvm.amdgcn.raw.buffer.load.v4i32");
__device__ unsigned raw_load32(u32x4 rsrc, int voff, int soff, int aux) __asm("llvm.amdgcn.raw.buffer.load.i32");

namespace {

typedef unsigned u32x2 __attribute__((ext_vector_type(2)));
typedef int i32x8 __attribute__((ext_vector_type(8)));
typedef float f32x4 __attribute__((ext_vector_type(4)));
typedef float f32x2 __attribute__((ext_vector_type(2)));
typedef unsigned long long u64;

constexpr int NR = 8;
constexpr int KD = 4096;
constexpr int MLOC = 512;
constexpr int NLOC = 512;
constexpr unsigned SHARD = MLOC * KD;
constexpr unsigned GROUP = 16 * KD;
constexpr unsigned SLOT = NR * SHARD;
constexpr int NSLOT = 3;
constexpr size_t CTRL_OFF = (size_t)NSLOT * SLOT;
constexpr unsigned CTRL_BYTES = 64 * 1024;
constexpr unsigned C_CNT = 256, C_ROLE = 4096, C_FLAG = 16384, C_TICK = 32768;
constexpr int NBLK = 256;
constexpr int NWAVE = 8;
constexpr int NTHR = NWAVE * 64;
constexpr unsigned LDS_BYTES = 96 * 1024;
constexpr int LROW = 264, LSL = 4224;
constexpr u64 DEADLINE = 50000000ull;
constexpr unsigned CODE_LINES = 136;
constexpr int AUX_SYS = 17;
constexpr int AUX_AGENT = 16;

#define PHI 5
#define PSI 1
constexpr int NNO = 32 - PHI - PSI;
constexpr int NI = NNO + PSI;
constexpr int NX = PHI + PSI;
constexpr int NIP = 16;
constexpr int NUNIT = NI * 32;
#define NSTAT 6
#define NDYNP 10
constexpr int NDYN = NDYNP * NIP;
constexpr int NXT = (NUNIT - NSTAT * NIP - NDYN) / 4;
constexpr int NMAX = 120;
#define AHEAD 6
#define AHEAD1 8
#define EARLY 4
#define EARLY0 ((NUNIT - NDYN) * 2 + 128)
#define HSLEEP 2
#define T_S 1050
#define T_P 1200
#define WWIN 1
#define W1WIN 1
#define SWIN 1
#define T_W2 600
#define T_L 600
#define T_X0 150
#define T_X1 450
#define SPECM 0
#define T_SA 1800
#define T_W0 300
#define T_W1 700
#define WSLEEP 4
#define FAUX AUX_SYS
#define RAUX AUX_SYS
#define ROT
#define GLATE 21
#define T_F (156 + 113 * NNO)
#define T_FL 2000
#define FWM 8
#define FWIN 0
#define TP0 1575
#define PD 56
#define ADJ0 0
#define ADJ1 -42
#define ADJ2 0
#define T_POLL 2800
#define PSLEEP 1
#define LATEM 150
#define S2EARLY 1
#define STR2(x) #x
#define STR(x) STR2(x)
static_assert(NNO >= 8 && PHI >= 1, "W quarters live in the unused groups of a cross source");

#define STAMP(i) do {} while (0)
#define STAMP_FLUSH() do {} while (0)
#define STAMP_DRAIN() do {} while (0)

struct Params {
  u32x4 dx, dc;
  char* myws;
  const uint8_t* w;
  unsigned rk;
  unsigned wq;
  char* ws[NR];
  const float* sx;
  const float* sw;
  float* out[NR];
  u64 task[2][14][2];
  u64 last[4][2];
};

__device__ __forceinline__ __amdgpu_buffer_rsrc_t mk_rsrc(const void* p, unsigned bytes) {
  return __builtin_amdgcn_make_buffer_rsrc(const_cast<void*>(p), 0, bytes, 0x00020000);
}
__device__ __forceinline__ unsigned min3u(unsigned a, unsigned b, unsigned c) { return min(min(a, b), c); }
__device__ __forceinline__ unsigned min4(unsigned m, u32x4 v) { return min3u(min3u(m, v[0], v[1]), v[2], v[3]); }
__device__ __forceinline__ u32x4 canon(u32x4 v) {
#pragma unroll
  for (int d = 0; d < 4; ++d) v[d] = v[d] ? v[d] : 0x80808080u;
  return v;
}
__device__ __forceinline__ void st128(u32x4 v, __amdgpu_buffer_rsrc_t r, unsigned voff, unsigned soff) {
  __builtin_amdgcn_raw_buffer_store_b128(v, r, voff, soff, AUX_SYS);
  asm volatile("s_nop 1" :: "v"(v) : "memory");
}
typedef __attribute__((address_space(3))) int lds_int;
typedef __attribute__((address_space(1))) unsigned gu32;
__device__ __forceinline__ void lds_st(lds_int* a, int v) { __hip_atomic_store(a, v, __ATOMIC_RELAXED, __HIP_MEMORY_SCOPE_WORKGROUP); }
__device__ __forceinline__ int lds_ld(lds_int* a) { return __hip_atomic_load(a, __ATOMIC_RELAXED, __HIP_MEMORY_SCOPE_WORKGROUP); }
__device__ __forceinline__ unsigned src_off(int s) { return (unsigned)(s >> 6) * GROUP + (unsigned)(s & 63) * 64; }
__device__ __forceinline__ void tr16(const unsigned (&r)[16], u32x4 (&o)[4]) {
#pragma unroll
  for (int d = 0; d < 4; ++d) {
    const unsigned a0 = r[4 * d], a1 = r[4 * d + 1], a2 = r[4 * d + 2], a3 = r[4 * d + 3];
    const unsigned t0 = __builtin_amdgcn_perm(a1, a0, 0x05010400u), t1 = __builtin_amdgcn_perm(a1, a0, 0x07030602u);
    const unsigned t2 = __builtin_amdgcn_perm(a3, a2, 0x05010400u), t3 = __builtin_amdgcn_perm(a3, a2, 0x07030602u);
    o[0][d] = __builtin_amdgcn_perm(t2, t0, 0x05040100u);
    o[1][d] = __builtin_amdgcn_perm(t2, t0, 0x07060302u);
    o[2][d] = __builtin_amdgcn_perm(t3, t1, 0x05040100u);
    o[3][d] = __builtin_amdgcn_perm(t3, t1, 0x07060302u);
  }
}

__global__ void __launch_bounds__(NTHR) ag_gemm_kernel(Params p) {
  extern __shared__ __attribute__((aligned(16))) char smem[];
  const u32x4 dx = p.dx, dc = p.dc;
  char* myws = p.myws;
  const uint8_t* pw = p.w;
  unsigned rk = p.rk, wq = p.wq;
  __builtin_amdgcn_sched_barrier(0);
  const int wid = __builtin_amdgcn_readfirstlane(threadIdx.x >> 6);
  const int lane = threadIdx.x & 63;
  const int bid = blockIdx.x;
  const int cu = bid >> 3;
  const unsigned icode = 0;
  const u64 t0 = __builtin_amdgcn_s_memrealtime();
  int xcd = __builtin_amdgcn_s_getreg(20 | (3 << 11)) & 7;
  auto peer = [&]() -> int { return xcd < 4 ? (0x21321321 >> (4 * wid)) & 3 : xcd; };
  const int li = lane & 15, lg = lane >> 4;
  unsigned vsrc = (unsigned)li * KD + (unsigned)lg * 16;
  const unsigned vlin = (unsigned)lane * 16;
  unsigned vw = (unsigned)lg * 16 * NLOC + (unsigned)li * 4;
  const int h = (wid + 5) >> 3;
  const int s5 = cu * 2 + (wid & 1) + (wid >> 1) * 8;
  static_assert(NIP == 16, "the first two pieces of a wave are 32 apart, in one group");
  auto sfirst = [&]() -> unsigned {
    const unsigned m = 0u - (unsigned)(xcd >> 2);
    return ((unsigned)(2 * (cu * 4 + xcd) + h) * 64 & ~m) | ((NI * GROUP + (unsigned)s5 * 64) & m);
  };
  unsigned so0 = sfirst();
  unsigned cnt = C_CNT + (unsigned)(bid * NWAVE + wid) * 4;
  unsigned so1 = so0 + GROUP;
  unsigned vw2 = vw + 8 * NLOC;
  asm volatile("" : "+s"(so0), "+s"(so1), "+s"(cnt), "+v"(vsrc), "+v"(vw), "+v"(vw2));
  __builtin_amdgcn_sched_barrier(0);
  u32x4 r0[4];
  unsigned craw;
  if (__builtin_expect(cu < 4, 1)) {
    r0[0] = raw_load128(dx, vsrc, so0, 0);
    __builtin_amdgcn_sched_barrier(0);
    craw = raw_load32(dc, 0, cnt, AUX_AGENT);
    __builtin_amdgcn_sched_barrier(0);
    r0[1] = raw_load128(dx, vsrc + 2048, so0, 0);
    __builtin_amdgcn_sched_barrier(0);
    r0[2] = raw_load128(dx, vsrc, so1, 0);
    __builtin_amdgcn_sched_barrier(0);
    r0[3] = raw_load128(dx, vsrc + 2048, so1, 0);
    __builtin_amdgcn_sched_barrier(0);
  }
  char* pws = p.ws[peer()];
  const uint8_t* px = (const uint8_t*)((u64)dx[1] << 32 | dx[0]);
  const __amdgpu_buffer_rsrc_t rc = mk_rsrc(myws + CTRL_OFF, CTRL_BYTES);
  const __amdgpu_buffer_rsrc_t rxs = mk_rsrc(px, SHARD);
  const unsigned rank = rk & 7u;
  const __amdgpu_buffer_rsrc_t rw = mk_rsrc(pw, KD * NLOC);
  gu32* const roles = (gu32*)(myws + CTRL_OFF + C_ROLE) + cu * 32 + (lane & 7);
  lds_int* const verdict = (lds_int*)(smem + 16 * LSL);
  unsigned took = 0;
  const bool asker = wid == (cu < 4 ? 6 : 0);
  auto ask = [&]() { if (asker && lane < 8) took = __hip_atomic_fetch_add(roles, lane == xcd ? 1u : 0u, __ATOMIC_RELAXED, __HIP_MEMORY_SCOPE_AGENT); };
  auto resolve = [&]() -> int {
    if (!asker) return 0;
    int r = xcd;
    unsigned c = __builtin_amdgcn_readlane(took, r);
    unsigned behind = (unsigned)__builtin_amdgcn_ballot_w64((int)(took - c) < 0) & 255u;
    while (__builtin_expect(behind != 0, 0)) {
      const int rn = __builtin_ctz(behind);
      if (lane < 8) took = __hip_atomic_fetch_add(roles, lane == rn ? 1u : lane == r ? ~0u : 0u, __ATOMIC_RELAXED, __HIP_MEMORY_SCOPE_AGENT);
      r = rn;
      c = __builtin_amdgcn_readlane(took, r);
      behind = (unsigned)__builtin_amdgcn_ballot_w64((int)(took - c) < 0) & 255u;
    }
    lds_st(verdict + (lane & 1), lane & 1 ? r : (int)c + 1);
    return r;
  };
  auto meet = [&]() -> int {
    asm volatile("s_waitcnt lgkmcnt(0)" ::: "memory");
    __builtin_amdgcn_s_barrier();
    asm volatile("" ::: "memory");
    return lds_ld(verdict + (lane & 1));
  };
  if (__builtin_expect(cu < 4, 1)) {
    lds_int* const pos = (lds_int*)smem;
    lds_int* const done = pos + 192;
    auto dfirst = [&](unsigned so) -> unsigned { return (so & ~(GROUP - 1)) | (so & (GROUP - 1)) * 16; };
    unsigned do0 = dfirst(so0), vd0 = xcd < 4 && wid >= 6 ? SHARD : vlin;
    char* pdst = pws + (size_t)rank * SHARD;
    asm volatile("" : "+s"(do0), "+v"(vd0), "+s"(pdst));
    __builtin_amdgcn_sched_barrier(0);
    const unsigned call = __builtin_amdgcn_readfirstlane(craw) + 1;
    __amdgpu_buffer_rsrc_t rd = mk_rsrc(pdst + (size_t)(call % NSLOT) * SLOT, SHARD);
    st128(canon(r0[0]), rd, vd0, do0);
    STAMP(2);
    __builtin_amdgcn_sched_barrier(0);
    st128(canon(r0[1]), rd, vd0, do0 + 32768);
    __builtin_amdgcn_sched_barrier(0);
#pragma unroll 1
    for (bool again = false;; again = true) {
      if (__builtin_expect(xcd < 4, 1)) {
        const int pidx = cu * 4 + xcd;
        if (wid == 6) { lds_st(pos + lane, -1); lds_st(pos + lane + 64, -1); lds_st(pos + lane + 128, -1); lds_st(done + lane, lane >= 6 ? 0x7fffffff : 0); }
        asm volatile("s_waitcnt lgkmcnt(0)" ::: "memory");
        __builtin_amdgcn_s_barrier();
        __builtin_amdgcn_sched_barrier(0);
        if (__builtin_expect(wid < 6, 1)) {
          lds_st(done + wid, 2);
          asm volatile("s_waitcnt vmcnt(1)" ::: "memory");
          st128(canon(r0[2]), rd, vlin, (unsigned)(2 * (pidx + 2 * NIP) + h) * 1024);
          __builtin_amdgcn_sched_barrier(0);
          u32x4 r[2] = {r0[2], r0[3]};
          int up[2] = {pidx + 2 * NIP, pidx + 3 * NIP};
#pragma unroll 1
          for (int n0 = 2; up[0] < NUNIT; n0 += 2) {
#pragma unroll
            for (int i = 0; i < 2; ++i) {
              const int n = n0 + i;
              lds_st(done + wid, n + 1);
              int un = pidx + (n + 2) * NIP;
              if (n + 2 >= NSTAT) {
                for (;;) {
                  un = __builtin_amdgcn_readfirstlane(lds_ld(pos + n + 2));
                  if (un >= 0) break;
                  __builtin_amdgcn_s_sleep(1);
                }
              }
              r[i] = __builtin_amdgcn_raw_buffer_load_b128(rxs, vsrc, src_off(2 * un + h), 0);
              up[i] = un;
              __builtin_amdgcn_sched_barrier(0);
              const int u = up[i ^ 1];
              st128(canon(r[i ^ 1]), rd, u < NUNIT ? vlin : SHARD, (unsigned)(2 * u + h) * 1024);
              __builtin_amdgcn_sched_barrier(0);
            }
          }
          STAMP(5);
          STAMP_DRAIN();
        } else if (wid == 6) {
          int role = xcd;
          if (!again) { ask(); role = resolve(); }
          if (role == xcd) {
            const size_t fo = (size_t)(call % NSLOT) * SLOT + (size_t)((rk >> (16 + 4 * xcd)) & 7u) * SHARD + NNO * GROUP + ((unsigned)cu * 16 + (wq >> 16)) * 1024;
            const __amdgpu_buffer_rsrc_t rm = mk_rsrc(myws + fo, 16384);
            while (__builtin_amdgcn_s_memrealtime() - t0 < T_FL) __builtin_amdgcn_s_sleep(8);
            asm volatile("" ::: "memory");
            u32x4 f[2];
            unsigned vok = vlin, tl = 0;
#pragma unroll
            for (int i = 1; i < 4; ++i) tl |= __builtin_amdgcn_raw_buffer_load_b32(mk_rsrc(p.ws[i] + fo, 16384), 0, 0, AUX_SYS);
            for (unsigned k = 1;; ++k) {
#pragma unroll
              for (int i = 0; i < 2; ++i) f[i] = __builtin_amdgcn_raw_buffer_load_b128(rm, vlin, (unsigned)i * 8192, AUX_SYS);
              if (__builtin_amdgcn_ballot_w64(min4(min4(0xffffffffu, f[0]), f[1]) == 0u) == 0ull) break;
              if ((k & 63) == 0 && __builtin_amdgcn_s_memrealtime() - t0 > DEADLINE) { vok = 16384; break; }
              __builtin_amdgcn_s_sleep(16);
              asm volatile("" ::: "memory");
            }
            asm volatile("" :: "v"(tl));
            while (__builtin_amdgcn_readfirstlane(lds_ld(pos + lds_ld(done))) < NNO * 32 - FWM) __builtin_amdgcn_s_sleep(2);
#pragma unroll
            for (int i = 0; i < 6; ++i) {
              if (i == 3) asm volatile("s_waitcnt vmcnt(" STR(FWIN) ")" ::: "memory");
              st128(f[i / 3], mk_rsrc(p.ws[1 + i % 3] + fo, 16384), vok, (unsigned)(i / 3) * 8192);
            }
          }
        } else {
          if (pidx == 0) __builtin_amdgcn_raw_buffer_store_b32(0u, rc, (unsigned)(lane & 7) * 128, C_TICK + ((call + 1) & 1) * 1024, AUX_AGENT);
          gu32* tick = (gu32*)(myws + CTRL_OFF + C_TICK + (call & 1) * 1024);
          int n = NSTAT;
          bool mine = true;
          for (;;) {
            int d = lds_ld(done + (lane & 7));
            d = min(d, __builtin_amdgcn_update_dpp(0x7fffffff, d, 0x111, 0xf, 0xf, false));
            d = min(d, __builtin_amdgcn_update_dpp(0x7fffffff, d, 0x112, 0xf, 0xf, false));
            d = min(d, __builtin_amdgcn_update_dpp(0x7fffffff, d, 0x114, 0xf, 0xf, false));
            const int lim = min(__builtin_amdgcn_readlane(d, 7) + (mine ? AHEAD1 : AHEAD), NMAX);
            const int k = lim - n;
            if (k <= 0) {
              if (n >= NMAX) { lds_st(pos + n + (lane & 3), NUNIT); break; }
              __builtin_amdgcn_s_sleep(2);
              continue;
            }
            unsigned t = 0;
            if (lane == 0) t = __hip_atomic_fetch_add(tick + (mine ? xcd * 32 : 128), (unsigned)k, __ATOMIC_RELAXED, __HIP_MEMORY_SCOPE_AGENT);
            if (mine) {
              const int t0 = (int)min(__builtin_amdgcn_readfirstlane(t), (unsigned)NXT);
              const int kk = min(k, NXT - t0);
              if (lane < kk) lds_st(pos + n + lane, NSTAT * NIP + (t0 + lane) * 4 + xcd);
              n += kk;
              mine = t0 + k < NXT;
            } else {
              const int t0 = (int)min(__builtin_amdgcn_readfirstlane(t), (unsigned)NDYN);
              const bool last = t0 + k >= NDYN;
              if (lane < (last ? k + 4 : k)) lds_st(pos + n + lane, min(NUNIT - NDYN + t0 + lane, NUNIT));
              if (last) break;
              n += k;
            }
          }
        }
        STAMP(6);
      } else {
        constexpr int NPW = NX * 2;
        int s6 = s5;
        asm volatile("" : "+s"(s6));
        auto spos = [&](int n) -> int { const int g = n >> 1; return (g < PHI ? NI + g : NNO + g - PHI) * 64 + (n & 1) * 32 + s6; };
        u32x4 r[NPW];
        const unsigned vpl = (s6 & 4) == 0 && (unsigned)(s6 & 3) != ((wq >> (4 * (xcd - 4))) & 3u) ? SHARD : vlin;
        static_assert(PHI >= 2, "the first four pieces of a wave");
        r[2] = r0[2]; r[3] = r0[3];
#pragma unroll
        for (int n = 2; n < 2 + S2EARLY; ++n) {
          asm volatile("s_waitcnt vmcnt(1)" ::: "memory");
          st128(canon(r[n]), rd, vlin, (unsigned)spos(n) * 1024);
          __builtin_amdgcn_sched_barrier(0);
        }
        constexpr int N1 = 2 + S2EARLY;
        __builtin_amdgcn_s_barrier();
        __builtin_amdgcn_sched_barrier(0);
        if (!again) ask();
#pragma unroll
        for (int n = 4; n < NPW; ++n) r[n] = __builtin_amdgcn_raw_buffer_load_b128(rxs, vsrc, src_off(spos(n)), 0);
#pragma unroll
        for (int n = 2; n < NPW; ++n) r[n] = canon(r[n]);
        STAMP(1);
#pragma unroll
        for (int n = N1; n < NPW; ++n) {
          if (n == N1 || n == 2 * PHI) {
            while (__builtin_amdgcn_s_memrealtime() - t0 < (n == 2 * PHI ? T_P : T_S)) __builtin_amdgcn_s_sleep(2);
            asm volatile("" ::: "memory");
          }
          st128(r[n], rd, n >> 1 == PHI ? vpl : vlin, (unsigned)spos(n) * 1024);
          asm volatile("s_waitcnt vmcnt(" STR(SWIN) ")" ::: "memory");
        }
        STAMP_DRAIN();
        STAMP(5);
        if (!again) resolve();
      }
      if (again) break;
      const int role = __builtin_amdgcn_readlane(meet(), 1);
      if (__builtin_expect(role == xcd, 1)) break;
      xcd = role;
      rd = mk_rsrc(p.ws[peer()] + (size_t)(call % NSLOT) * SLOT + (size_t)rank * SHARD, SHARD);
#pragma unroll
      for (int i = 0; i < 4; ++i) r0[i] = __builtin_amdgcn_raw_buffer_load_b128(rxs, vsrc + (unsigned)(i & 1) * 2048, sfirst() + (unsigned)(i >> 1) * GROUP, 0);
#pragma unroll
      for (int i = 0; i < 2; ++i) st128(canon(r0[i]), rd, xcd < 4 && wid >= 6 ? SHARD : vlin, dfirst(sfirst()) + (unsigned)i * 32768);
    }
    asm volatile("" :: "v"(icode));
    __builtin_amdgcn_raw_buffer_store_b32(call, rc, 0, cnt, AUX_AGENT);
    if (xcd >= 4 && cu * 8 + wid < 28) {
      const int c = cu * 8 + wid;
      const unsigned rg = (unsigned)(p.last[xcd - 4][c >> 4] >> (((c >> 1) & 7) * 8)) & 255u;
      const __amdgpu_buffer_rsrc_t rt = mk_rsrc(p.out[0], NR * MLOC * NLOC * 4);
      const unsigned so = rg * 16 * (NLOC * 4) + ((wq >> (4 * (xcd - 4))) & 3u) * 512 + (unsigned)(c & 1) * 256;
      const unsigned vt = (unsigned)lg * (NLOC * 4) + (unsigned)li * 16;
#pragma unroll
      for (int q = 0; q < 4; ++q) st128(u32x4{~0u, ~0u, ~0u, ~0u}, rt, vt, so + (unsigned)q * 4 * (NLOC * 4));
      asm volatile("s_waitcnt vmcnt(0)" ::: "memory");
      __builtin_amdgcn_raw_buffer_store_b32(call, mk_rsrc(p.ws[xcd] + CTRL_OFF, CTRL_BYTES), 0, C_FLAG + rank * 256 + (unsigned)c * 4, AUX_SYS);
      while (__builtin_amdgcn_s_memrealtime() - t0 < T_POLL) __builtin_amdgcn_s_sleep(8);
      asm volatile("" ::: "memory");
      for (unsigned n = 1;; ++n) {
        u32x4 t[4];
#pragma unroll
        for (int q = 0; q < 4; ++q) t[q] = __builtin_amdgcn_raw_buffer_load_b128(rt, vt, so + (unsigned)q * 4 * (NLOC * 4), AUX_SYS);
        unsigned m = 0;
#pragma unroll
        for (int q = 0; q < 4; ++q) m = max(max(max(m, t[q][0]), t[q][1]), max(t[q][2], t[q][3]));
        if (__builtin_amdgcn_ballot_w64(m == ~0u) == 0ull) break;
        if ((n & 63) == 0 && __builtin_amdgcn_s_memrealtime() - t0 > DEADLINE) {
          __builtin_amdgcn_raw_buffer_store_b32(call, rc, 0, 0, AUX_AGENT);
          break;
        }
        __builtin_amdgcn_s_sleep(PSLEEP);
        asm volatile("" ::: "memory");
      }
    }
    STAMP_FLUSH();
    return;
  }

  ask();
  __builtin_amdgcn_sched_barrier(0);
  unsigned warm = 0;
  unsigned raw0[2][16];
  auto first_loads = [&](bool first) {
    if (xcd < 4) {
      const int wi = wid * 28 + cu - 4;
      if (wi < NI * 4)
        warm = __builtin_amdgcn_raw_buffer_load_b32(rxs, (unsigned)(lane >> 4) * KD + (unsigned)((lane >> 1) & 7) * 512 + (unsigned)(lane & 1) * 64,
                                                    (unsigned)wi * 4 * KD + (unsigned)xcd * 128, 0);
      if (first) resolve();
    } else if (cu < 12) {
      const unsigned so = (unsigned)((cu - 4) * 8 + wid) * 64 * NLOC + ((wq >> (4 * (xcd - 4))) & 3u) * 128;
#pragma unroll
      for (int hh = 0; hh < 2; ++hh)
#pragma unroll
        for (int i = 0; i < 16; ++i) raw0[hh][i] = __builtin_amdgcn_raw_buffer_load_b32(rw, (i < 8 ? vw : vw2) + (unsigned)((i & 7) * NLOC + hh * 64), so, 0);
      __builtin_amdgcn_sched_barrier(0);
      if (first) resolve();
    } else {
      if (cu < 28)
        warm = __builtin_amdgcn_raw_buffer_load_b32(rw, (unsigned)(lane >> 1) * NLOC + (unsigned)(lane & 1) * 64,
                                                    (unsigned)(((cu - 12) & 7) * 16 + wid * 2 + ((cu - 12) >> 3)) * 32 * NLOC + ((wq >> (4 * (xcd - 4))) & 3u) * 128, 0);
      if (first) resolve();
    }
    __builtin_amdgcn_sched_barrier(0);
  };
  first_loads(true);
  const int vd = meet();
  const unsigned call = __builtin_amdgcn_readlane(vd, 0);
  warm |= icode;
  if (__builtin_expect(__builtin_amdgcn_readlane(vd, 1) != xcd, 0)) {
    xcd = __builtin_amdgcn_readlane(vd, 1);
    pws = p.ws[peer()];
    first_loads(false);
  }
  const bool cross = xcd >= 4;
  if (cross && cu < 12) {
    const __amdgpu_buffer_rsrc_t rd = mk_rsrc(pws + (size_t)(call % NSLOT) * SLOT + (size_t)rank * SHARD, SHARD);
    const int kb = (cu - 4) * 8 + wid;
    u32x4 o[2][4];
    tr16(raw0[0], o[0]);
    tr16(raw0[1], o[1]);
#pragma unroll
    for (int e = 0; e < 8; ++e) {
      if (e > WWIN) asm volatile("s_waitcnt vmcnt(" STR(WWIN) ")" ::: "memory");
      st128(canon(o[e & 1][e >> 1]), rd, vlin + (unsigned)(e >> 1) * 1024, (unsigned)((e & 1) * 64 + kb) * 4096);
      __builtin_amdgcn_sched_barrier(0);
      if (e == 0) STAMP(2);
    }
    STAMP(7);
    __builtin_amdgcn_s_barrier();
  }
  __builtin_amdgcn_sched_barrier(0);

  const int c28 = cu - 4;
  const int ci = xcd * 28 + c28;
  const int strm = cross ? c28 >> 1 : ci >> 3;
  const int slab = cross ? c28 & 1 : ci & 7;
  const unsigned rot = cross ? (unsigned)slab * 4 : 0u;
  u64 tl = p.task[cross][strm][0], th = p.task[cross][strm][1];
  typedef const __attribute__((address_space(4))) float cf32;
  float sx = *(cf32*)p.sx, sw = *(cf32*)p.sw;
  float* po = p.out[cross ? xcd : 0];
  bool dead = false;
  auto expired = [&]() -> bool {
    return __builtin_amdgcn_readfirstlane(__builtin_amdgcn_raw_buffer_load_b32(rc, 0, 0, AUX_AGENT)) == call ||
           __builtin_amdgcn_s_memrealtime() - t0 > DEADLINE;
  };
  auto next = [&]() -> int {
    const int c = (int)(tl & 255u);
    tl = (tl >> 8) | (th << 56); th = (th >> 8) | (63ull << 56);
    if (c == 63) return -1;
    int role = c >> 5;
    if (cross && role >= 4) role = 4 + ((role + xcd) & 3);
    return (int)(((rk >> (4 * role)) & 7u) << 8) | c;
  };
  u32x4 xv[8];
  auto issue_own = [&](int code) {
    const __amdgpu_buffer_rsrc_t r = rxs;
    const unsigned so = (unsigned)(code & 31) * GROUP + (unsigned)wid * 512;
#pragma unroll
    for (int q = 0; q < 8; ++q) xv[q] = __builtin_amdgcn_raw_buffer_load_b128(r, vsrc + (unsigned)(q & 3) * 64, so + ((unsigned)(q & 4) ^ rot) * 64, 0);
  };
  int code = next();
  u32x4 wf[8][4];
  if (!cross) {
    const unsigned vo = vw + (unsigned)wid * 512 * NLOC + (unsigned)slab * 64;
    unsigned raw[3][16];
#pragma unroll
    for (int u = 0; u < 10; ++u) {
      if (u < 8) {
#pragma unroll
        for (int i = 0; i < 16; ++i) raw[u % 3][i] = __builtin_amdgcn_raw_buffer_load_b32(rw, vo, (unsigned)(u * 64 + i) * NLOC, 0);
      }
      if (u == 8) issue_own(code);
      __builtin_amdgcn_sched_barrier(0);
      if (u >= 2) tr16(raw[(u - 2) % 3], wf[u - 2]);
      __builtin_amdgcn_sched_barrier(0);
    }
    asm volatile("" : "+s"(sx), "+s"(sw), "+s"(po) : "v"(warm));
  } else {
    const __amdgpu_buffer_rsrc_t rq = mk_rsrc(myws + (size_t)(call % NSLOT) * SLOT + (size_t)((rk >> (4 * xcd)) & 7u) * SHARD +
                                              (size_t)(slab * 64 + wid * 8) * 4096, 32768);
    const unsigned vq = (unsigned)(lane & 7) * 4096 + (unsigned)((lane >> 3) & 3) * 1024 + 1020;
    unsigned got = 0;
    asm volatile("" : "+s"(sx), "+s"(sw), "+s"(po) : "v"(warm));
    while (__builtin_amdgcn_s_memrealtime() - t0 < T_W0) __builtin_amdgcn_s_sleep(8);
    asm volatile("" ::: "memory");
    issue_own(code);
#pragma unroll 1
    for (unsigned n = 1; got != 15 && !dead; ++n) {
      __builtin_amdgcn_s_sleep(WSLEEP);
      const unsigned hh = __builtin_amdgcn_raw_buffer_load_b32(rq, vq, 0, AUX_SYS);
      const unsigned miss = (unsigned)__builtin_amdgcn_ballot_w64(hh == 0u);
#pragma unroll
      for (int e = 0; e < 4; ++e) {
        if (!((got >> e) & 1) && !((miss >> (8 * e)) & 255u)) {
          got |= 1u << e;
#pragma unroll
          for (int q = 0; q < 8; ++q) wf[q][e] = __builtin_amdgcn_raw_buffer_load_b128(rq, vlin + (unsigned)e * 1024, ((unsigned)q ^ rot) * 4096, RAUX);
        }
      }
      if ((n & 63) == 0 && expired()) dead = true;
      asm volatile("" ::: "memory");
    }
    STAMP(3);
    for (unsigned n = 1;; ++n) {
      unsigned m = 0xffffffffu;
#pragma unroll
      for (int q = 0; q < 8; ++q)
#pragma unroll
        for (int e = 0; e < 4; ++e) m = min4(m, wf[q][e]);
      if (__builtin_expect(__builtin_amdgcn_ballot_w64(m == 0u) == 0ull || dead, 1)) break;
      if ((n & 63) == 0 && expired()) dead = true;
#pragma unroll
      for (int q = 0; q < 8; ++q)
#pragma unroll
        for (int e = 0; e < 4; ++e) wf[q][e] = __builtin_amdgcn_raw_buffer_load_b128(rq, vlin + (unsigned)e * 1024, ((unsigned)q ^ rot) * 4096, RAUX);
      asm volatile("" ::: "memory");
    }
  }
  STAMP(1);
  float scale = dead ? __builtin_nanf("") : sx * sw;
  const __amdgpu_buffer_rsrc_t rs = mk_rsrc(myws + (size_t)(call % NSLOT) * SLOT, SLOT);
  const __amdgpu_buffer_rsrc_t rz = mk_rsrc(myws + (size_t)((call + 2) % NSLOT) * SLOT, SLOT);
  const __amdgpu_buffer_rsrc_t ro = mk_rsrc(po, NR * MLOC * NLOC * 4 - (cross ? 3 * 512 : 0));
  const unsigned vh = (unsigned)(lane & 7) * 1024 + 1020;
  const int orow = 2 * wid + (lane >> 5);
  const unsigned vout = (unsigned)orow * (NLOC * 4) + (unsigned)slab * 256 + (unsigned)(lane & 31) * 8;
  typedef __attribute__((address_space(3))) char lds_char;
  lds_char* const lw = (lds_char*)smem + wid * LSL + li * LROW + lg * 64;
  const lds_char* const lr = (lds_char*)smem + orow * LROW + (lane & 31) * 8;
  const unsigned vz = cross ? 0x80000000u : vlin;
  const char* const xslot = myws + (size_t)(call % NSLOT) * SLOT;
  auto when = [&](int code) -> int {
    const int g = code & 31;
    int a = (g < NNO ? 213 : 269) + 113 * g, b = g == NI ? 400 : g > NI ? T_SA : g == NNO ? T_F : T_P + 300;
    asm volatile("" : "+s"(a), "+s"(b));
    a = code & 0x80 ? b : a;
    a = code & 0xe0 ? a : 0;
    return code < 0 ? 0x7fffffff : a;
  };
  __builtin_amdgcn_sched_barrier(0);
  int buf = 0;
  bool again = false;
  const unsigned fo = C_FLAG + ((rk >> (4 * xcd)) & 7u) * 256 + (unsigned)c28 * 4;
  int slot = cross ? TP0 * 8 + ((strm < 4 ? strm + 10 : strm - 4) * 2 + slab) * PD : -0x10000000;
  const int period = 28 * PD + ((unsigned)(strm - 4) < 3u ? ADJ0 : (unsigned)(strm - 4) < 7u ? ADJ1 : ADJ2);
  unsigned marked = call - 1;
  __builtin_amdgcn_raw_buffer_store_b32(0u, ro, 0x80000000u, 0, AUX_SYS);
  __builtin_amdgcn_sched_barrier(0);
#pragma unroll 1
  do {
    const int ncode = next();
    const bool nown = (ncode & 0xe0) == 0;
    unsigned nstep = nown ? 64 : 1024;
    unsigned nso = (unsigned)(ncode & 31) * GROUP + (unsigned)wid * 8 * nstep;
    unsigned nvo = nown ? vsrc : vlin;
    unsigned nlo = nso + rot * nstep, nhi = nso + (rot ^ 4) * nstep;
    asm volatile("" : "+s"(nlo), "+s"(nhi));
    const char* nbase = nown ? (const char*)px : xslot + (size_t)((ncode >> 8) & 7) * SHARD;
    int nwhen = when(ncode);
    unsigned zo = (unsigned)(code >> 8) * SHARD + (unsigned)(code & 31) * GROUP, zw = ((rk >> (16 + ((code >> 1) & 12))) & 7u) * SHARD + (unsigned)(code & 7) * GROUP;
    asm volatile("" : "+s"(zo), "+s"(zw));
    zo = (code & 0xe0 ? zo : zw) + (unsigned)(slab * 8 + wid) * 1024;
    unsigned oo = ((unsigned)(code >> 8) * MLOC + (unsigned)(code & 31) * 16) * (NLOC * 4);
    lds_char* lwb = lw + buf;
    const lds_char* lrb = lr + buf;
    int far = ncode < 0 ? 0 : ncode & 0xe0;
    int old = cross ? (ncode & 0x9f) - GLATE : 0;
    asm volatile("" : "+s"(old));
    far = (unsigned)old >= 32u - GLATE ? -far : far;
    unsigned vf = cross && ncode >= 0 && (tl & 255u) == 63 ? 0u : 0x80000000u;
    asm volatile("" : "+s"(nstep), "+s"(nso), "+s"(nbase), "+s"(nwhen), "+s"(zo), "+s"(oo), "+s"(far), "+s"(tl), "+s"(th));
    asm volatile("" : "+v"(nvo), "+v"(lwb), "+v"(lrb), "+v"(vf));
    __builtin_amdgcn_sched_barrier(0);
    asm volatile("" :: "v"(xv[0]), "v"(xv[1]), "v"(xv[2]), "v"(xv[3]), "v"(xv[4]), "v"(xv[5]), "v"(xv[6]), "v"(xv[7]));
    __builtin_amdgcn_sched_barrier(0);
    unsigned mk = __builtin_amdgcn_readfirstlane(marked);
    asm volatile("" : "+s"(mk));
    __builtin_amdgcn_sched_barrier(0);
    if (__builtin_expect(again, 0)) {
      unsigned m = 0xffffffffu, mx = 0;
#pragma unroll
      for (int q = 0; q < 8; ++q) { m = min4(m, xv[q]); mx |= xv[q][3]; }
      if (__builtin_expect(__builtin_amdgcn_ballot_w64(m == 0u) != 0ull && !dead, 0)) {
        const int s = (code & 31) * 64 + wid * 8;
        const unsigned so = (unsigned)(code >> 8) * SHARD + (unsigned)s * 1024;
        const unsigned ho = (code & 0x80) == 0 && s >= EARLY0 ? so - EARLY * 8192 : so;
        bool poll = __builtin_amdgcn_ballot_w64(mx != 0u) == 0ull;
        for (unsigned n = 1;; ++n) {
          if (poll) {
            const unsigned hh = __builtin_amdgcn_raw_buffer_load_b32(rs, vh, ho, AUX_SYS);
            poll = __builtin_amdgcn_ballot_w64(hh == 0u) != 0ull;
            if (poll) __builtin_amdgcn_s_sleep(HSLEEP);
          }
          if (!poll) {
#pragma unroll
            for (int q = 0; q < 8; ++q)
                xv[q] = __builtin_amdgcn_raw_buffer_load_b128(rs, vlin + (unsigned)(q & 3) * 1024, so + ((unsigned)(q & 4) ^ rot) * 1024, RAUX);
            m = 0xffffffffu;
#pragma unroll
            for (int q = 0; q < 8; ++q) m = min4(m, xv[q]);
            if (__builtin_amdgcn_ballot_w64(m == 0u) == 0ull) break;
            if (n > 64) __builtin_amdgcn_s_sleep(16);
          }
          if ((n & 63) == 0 && expired()) { dead = true; break; }
          asm volatile("" ::: "memory");
        }
        if (dead) { __builtin_amdgcn_raw_buffer_store_b32(call, rc, 0, 0, AUX_AGENT); scale = __builtin_nanf(""); }
      }
    }
    unsigned now = (unsigned)__builtin_amdgcn_s_memrealtime();
    __builtin_amdgcn_sched_barrier(0);
    __amdgpu_buffer_rsrc_t rn;
    f32x4 acc[4];
#pragma unroll
    for (int e = 0; e < 4; ++e) acc[e] = f32x4{0.f, 0.f, 0.f, 0.f};
    auto fetch = [&](int q) { xv[q] = __builtin_amdgcn_raw_buffer_load_b128(rn, nvo, (q < 4 ? nlo : nhi) + (unsigned)(q & 3) * nstep, FAUX); };
    auto part = [&](int e) {
#pragma unroll
      for (int r = 0; r < 4; ++r) *(__attribute__((address_space(3))) float*)(lwb + r * 16 + e * 4) = acc[e][r];
    };
#pragma unroll
    for (int ks = 0; ks < 4; ++ks) {
      const i32x8 b = {(int)xv[2 * ks][0], (int)xv[2 * ks][1], (int)xv[2 * ks][2], (int)xv[2 * ks][3],
                       (int)xv[2 * ks + 1][0], (int)xv[2 * ks + 1][1], (int)xv[2 * ks + 1][2], (int)xv[2 * ks + 1][3]};
#pragma unroll
      for (int e = 0; e < 4; ++e) {
        const u32x4 a0 = wf[2 * ks][e], a1 = wf[2 * ks + 1][e];
        const i32x8 a = {(int)a0[0], (int)a0[1], (int)a0[2], (int)a0[3], (int)a1[0], (int)a1[1], (int)a1[2], (int)a1[3]};
        acc[e] = __builtin_amdgcn_mfma_scale_f32_16x16x128_f8f6f4(a, b, acc[e],   1,   0, 0, 0, 0, 0);
        __builtin_amdgcn_sched_barrier(0);
        if (ks == 0 && e == 0) st128(u32x4{0u, 0u, 0u, 0u}, rz, vz, zo);
        if (ks == 0 && e == 1) marked = __builtin_amdgcn_raw_buffer_load_b32(rc, vf, fo, AUX_SYS);
        if (ks == 0 && e == 3) {
          asm volatile("" : "+s"(now));
          rn = mk_rsrc(nbase, (int)(now - (unsigned)t0) >= nwhen ? SHARD : 0);
        }
        if (ks > 0 && e < 2) fetch(2 * ks - 2 + e);
        if (ks == 3 && e >= 2) part(e - 2);
        __builtin_amdgcn_sched_barrier(0);
      }
    }
    part(2);
    __builtin_amdgcn_sched_barrier(0);
    part(3);
    __builtin_amdgcn_sched_barrier(0);
    fetch(6);
    fetch(7);
    __builtin_amdgcn_sched_barrier(0);
    asm volatile("s_waitcnt lgkmcnt(0)" ::: "memory");
    __builtin_amdgcn_s_barrier();
    asm volatile("" ::: "memory");
    f32x2 pr[8], ov;
#pragma unroll
    for (int v = 0; v < 8; ++v) pr[v] = *(const __attribute__((address_space(3))) f32x2*)(lrb + v * LSL);
    const f32x2 y = (((pr[0] + pr[1]) + (pr[2] + pr[3])) + ((pr[4] + pr[5]) + (pr[6] + pr[7]))) * scale;
#pragma unroll
    for (int r = 0; r < 2; ++r) ov[r] = y[r] * __builtin_amdgcn_rcpf(1.f + __expf(-__builtin_amdgcn_fmed3f(y[r], -60.f, 60.f)));
    u32x2 ob = __builtin_bit_cast(u32x2, ov);
#pragma unroll
    for (int r = 0; r < 2; ++r) ob[r] = min(ob[r], 0xfffffffeu);
    asm volatile("" : "+v"(ob));
    __builtin_amdgcn_sched_barrier(0);
    auto pace = [&]() {
      if ((int)(now - (unsigned)t0) * 8 < slot) {
        while ((int)((unsigned)__builtin_amdgcn_s_memrealtime() - (unsigned)t0) * 8 < slot) __builtin_amdgcn_s_sleep(1);
        asm volatile("" ::: "memory");
      }
    };
    again = far < 0;
    if (far > 0) {
      unsigned m = 0xffffffffu;
#pragma unroll
      for (int q = 0; q < 8; ++q) m = min4(m, xv[q]);
      again = __builtin_amdgcn_ballot_w64(m == 0u) != 0ull;
      const unsigned so = (unsigned)(ncode >> 8) * SHARD + nso;
      auto refetch = [&]() {
#pragma unroll
        for (int q = 0; q < 8; ++q) xv[q] = __builtin_amdgcn_raw_buffer_load_b128(rs, vlin + (unsigned)(q & 3) * 1024, so + ((unsigned)(q & 4) ^ rot) * 1024, RAUX);
      };
      if (again && (int)(__builtin_amdgcn_s_memrealtime() - t0) + LATEM >= nwhen) refetch();
      if (cross) __builtin_amdgcn_s_barrier();
    }
    if (cross && ncode < 0) {
      for (unsigned n = 1; __builtin_expect(mk != call, 0); ++n) {
        mk = __builtin_amdgcn_readfirstlane(__builtin_amdgcn_raw_buffer_load_b32(rc, 0, fo, AUX_SYS));
        if ((n & 63) == 0 && expired()) break;
        __builtin_amdgcn_s_sleep(2);
        asm volatile("" ::: "memory");
      }
      asm volatile("s_waitcnt vmcnt(0)" ::: "memory");
    }
    pace();
    slot += period;
    __builtin_amdgcn_sched_barrier(0);
    __builtin_amdgcn_raw_buffer_store_b64(ob, ro, vout, oo, AUX_SYS);
    __builtin_amdgcn_sched_barrier(0);
    code = ncode;
    buf = 8 * LSL - buf;
  } while (code >= 0);
  STAMP(5);
  STAMP_DRAIN();
  STAMP(6);
  STAMP_FLUSH();
}

#define DUR0 1.2f
#define DUR1 2.0f
#define BEG0 8.f
#define BEG2 13.9f
#define BEG1 12.f
void build_tasks(Params& p) {
  for (int t = 0; t < 2; ++t) {
    float when[168];
    uint8_t what[168];
    int n = 0;
    auto add = [&](float w, int role, int g) {
      int i = n++;
      for (; i > 0 && when[i - 1] > w; --i) { when[i] = when[i - 1]; what[i] = what[i - 1]; }
      when[i] = w; what[i] = (uint8_t)(role << 5 | g);
    };
    for (int g = 0; g < (t ? NNO : 32); ++g) add(0.f, 0, g);
    for (int g = 0; g < (t ? NNO : NI); ++g) for (int r = 1; r < 4; ++r) add((g < NNO ? 1.f : 1.56f) + 1.13f * (g + 1), r, g);
    for (int i = 0; i < (t ? PHI : NX); ++i)
      for (int r = t ? 5 : 4; r < 8; ++r)
        add(i == 0 ? 3.f : i < PHI ? T_SA * 0.01f : i == PHI ? T_F * 0.01f : T_P * 0.01f + 1.5f * (i - PHI + 1), r, i < PHI ? NI + i : NNO + i - PHI);
    float busy[14];
    int len[14];
    uint8_t lst[14][16];
    for (int j = 0; j < 14; ++j) { busy[j] = t ? (j < 4 ? BEG2 : BEG1) : BEG0; len[j] = 0; }
    for (int i = 0; i < n; ++i) {
      int b = -1;
      for (int j = 0; j < 14; ++j) if (len[j] < 15 && (b < 0 || busy[j] < busy[b])) b = j;
      busy[b] = (busy[b] > when[i] ? busy[b] : when[i]) + (t ? DUR1 : DUR0);
      lst[b][len[b]++] = what[i];
    }
    if (t) {
      static_assert((NNO * 4 + PHI * 3) % 14 == 7, "lists 4..10 are the long ones");
      uint8_t e[7];
      for (int j = 0; j < 7; ++j) e[j] = lst[4 + j][len[4 + j] - 1];
      for (int j = 0; j < 7; ++j) lst[4 + j][len[4 + j] - 1] = e[(j + 4) % 7];
    }
    for (int j = 0; j < 14; ++j) {
      u64 v[2] = {0, 0};
      const int o = 0;
      for (int i = 0; i < 16; ++i) v[i >> 3] |= (u64)(i >= o && i - o < len[j] ? lst[j][i - o] : 63) << (8 * (i & 7));
      p.task[t][j][0] = v[0]; p.task[t][j][1] = v[1];
    }
  }
}

}

extern "C" void kernel_launch(int rank, int nranks, ncclComm_t comm,
                              hipStream_t stream,
                              void* const* d_in, const int* in_sizes, int n_in,
                              void* d_out, int out_size,
                              const int* in_dtypes, int out_dtype,
                              void* const* d_out_all,
                              void* const* d_ws_all, size_t ws_bytes) {
  (void)comm; (void)in_dtypes; (void)out_dtype;
  if (nranks != NR || (unsigned)rank >= (unsigned)NR || n_in < 4 || in_sizes[0] != MLOC * KD || in_sizes[1] != KD * NLOC ||
      out_size != NR * MLOC * NLOC || ws_bytes < CTRL_OFF + CTRL_BYTES) {
    fprintf(stderr, "kernel_launch: unexpected problem shape\n");
    return;
  }
  static bool attr_done[NR];
  if (!attr_done[rank]) {
    HIP_CHECK(hipFuncSetAttribute((const void*)ag_gemm_kernel, hipFuncAttributeMaxDynamicSharedMemorySize, LDS_BYTES));
    attr_done[rank] = true;
  }
  Params p;
  auto desc = [](const void* a, unsigned bytes) { return u32x4{(unsigned)(uintptr_t)a, (unsigned)((uintptr_t)a >> 32) & 0xffffu, bytes, 0x00020000u}; };
  p.dx = desc(d_in[0], SHARD);
  p.dc = desc((char*)d_ws_all[rank] + CTRL_OFF, CTRL_BYTES);
  p.w = (const uint8_t*)d_in[1];
  p.sx = (const float*)d_in[2];
  p.sw = (const float*)d_in[3];
  p.myws = (char*)d_ws_all[rank];
  const int quad = 0x4d;
  auto qidx = [&](int r) { int n = 0; for (int q = 0; q < r; ++q) n += ((quad >> q) & 1) == ((quad >> r) & 1); return n; };
  auto roles = [&](int r, int (&role)[NR]) {
    int na = 1, nb = 4;
    role[0] = r;
    for (int j = 1; j < NR; ++j) {
      const int q = (r + j) & 7;
      if (((quad >> q) & 1) == ((quad >> r) & 1)) role[na++] = q; else role[nb++] = q;
    }
  };
  int role[NR];
  roles(rank, role);
  p.rk = 0; p.wq = 0;
  for (int i = 0; i < NR; ++i) {
    p.ws[i] = (char*)d_ws_all[role[i]];
    p.out[i] = i == 0 ? (float*)d_out : (float*)d_out_all[role[i]] + (i >= 4 ? qidx(rank) * 128 : 0);
    p.rk |= (unsigned)role[i] << (4 * i);
    if (i >= 4) p.wq |= (unsigned)qidx(role[i]) << (4 * (i - 4));
  }
  p.wq |= (unsigned)qidx(rank) << 16;
  build_tasks(p);
  for (int i = 0; i < 4; ++i) {
    int their[NR], me = 4;
    roles(role[4 + i], their);
    while (their[me] != rank) ++me;
    p.last[i][0] = p.last[i][1] = 0;
    for (int j = 0; j < 14; ++j) {
      int c = 63;
      for (int n = 15; c == 63; --n) c = (int)(p.task[1][j][n >> 3] >> (8 * (n & 7))) & 255;
      const int r = c >> 5 < 4 ? c >> 5 : 4 + (((c >> 5) + me) & 3);
      p.last[i][j >> 3] |= (u64)(their[r] * 32 + (c & 31)) << (8 * (j & 7));
    }
  }
  hipLaunchKernelGGL(ag_gemm_kernel, dim3(NBLK), dim3(NTHR), LDS_BYTES, stream, p);
  HIP_CHECK(hipGetLastError());
}
```

```cpp
#include <hip/hip_runtime.h>
#include <rccl/rccl.h>
#include <cstdio>
#include <cstdint>

#define HIP_CHECK(expr) do { hipError_t _e = (expr); if (_e != hipSuccess) { \
    fprintf(stderr, "HIP error %s at %s:%d\n", hipGetErrorString(_e), __FILE__, __LINE__); \
    return; } } while (0)

typedef unsigned u32x4 __attribute__((ext_vector_type(4)));
__device__ u32x4 raw_load128(u32x4 rsrc, int voff, int soff, int aux) __asm("llvm.amdgcn.raw.buffer.load.v4i32");
__device__ unsigned raw_load32(u32x4 rsrc, int voff, int soff, int aux) __asm("llvm.amdgcn.raw.buffer.load.i32");

namespace {

typedef unsigned u32x2 __attribute__((ext_vector_type(2)));
typedef int i32x8 __attribute__((ext_vector_type(8)));
typedef float f32x4 __attribute__((ext_vector_type(4)));
typedef float f32x2 __attribute__((ext_vector_type(2)));
typedef unsigned long long u64;

constexpr int NR = 8;
constexpr int KD = 4096;
constexpr int MLOC = 512;
constexpr int NLOC = 512;
constexpr unsigned SHARD = MLOC * KD;
constexpr unsigned GROUP = 16 * KD;
constexpr unsigned SLOT = NR * SHARD;
constexpr int NSLOT = 3;
constexpr size_t CTRL_OFF = (size_t)NSLOT * SLOT;
constexpr unsigned CTRL_BYTES = 64 * 1024;
constexpr unsigned C_CNT = 256, C_ROLE = 4096, C_FLAG = 16384, C_TICK = 32768;
constexpr int NBLK = 256;
constexpr int NWAVE = 8;
constexpr int NTHR = NWAVE * 64;
constexpr unsigned LDS_BYTES = 96 * 1024;
constexpr int LROW = 264, LSL = 4224;
constexpr u64 DEADLINE = 50000000ull;
constexpr unsigned CODE_LINES = 136;
constexpr int AUX_SYS = 17;
constexpr int AUX_AGENT = 16;

#define PHI 5
#define PSI 1
constexpr int NNO = 32 - PHI - PSI;
constexpr int NI = NNO + PSI;
constexpr int NX = PHI + PSI;
constexpr int NIP = 16;
constexpr int NUNIT = NI * 32;
#define NSTAT 6
#define NDYNP 10
constexpr int NDYN = NDYNP * NIP;
constexpr int NXT = (NUNIT - NSTAT * NIP - NDYN) / 4;
constexpr int NMAX = 120;
#define AHEAD 6
#define AHEAD1 8
#define EARLY 4
#define EARLY0 ((NUNIT - NDYN) * 2 + 128)
#define HSLEEP 2
#define T_S 1050
#define T_P 1200
#define WWIN 1
#define W1WIN 1
#define SWIN 1
#define T_W2 600
#define T_L 600
#define T_X0 150
#define T_X1 450
#define SPECM 0
#define T_SA 1800
#define TOUCH 20
#define T_W0 300
#define T_W1 700
#define WSLEEP 4
#define FAUX AUX_SYS
#define RAUX AUX_SYS
#define ROT
#define GLATE 21
#define T_F (156 + 113 * NNO)
#define T_FL 2000
#define FWM 8
#define FWIN 0
#define TP0 1560
#define PD 56
#define ADJ0 0
#define ADJ1 -42
#define ADJ2 0
#define T_POLL 2800
#define PSLEEP 1
#define LATEM 150
#define S2EARLY 1
#define STR2(x) #x
#define STR(x) STR2(x)
static_assert(NNO >= 8 && PHI >= 1, "W quarters live in the unused groups of a cross source");

#define STAMP(i) do {} while (0)
#define STAMP_FLUSH() do {} while (0)
#define STAMP_DRAIN() do {} while (0)

struct Params {
  u32x4 dx, dc;
  char* myws;
  const uint8_t* w;
  unsigned rk;
  unsigned wq;
  char* ws[NR];
  const float* sx;
  const float* sw;
  float* out[NR];
  u64 task[2][14][2];
  u64 last[4][2];
};

__device__ __forceinline__ __amdgpu_buffer_rsrc_t mk_rsrc(const void* p, unsigned bytes) {
  return __builtin_amdgcn_make_buffer_rsrc(const_cast<void*>(p), 0, bytes, 0x00020000);
}
__device__ __forceinline__ unsigned min3u(unsigned a, unsigned b, unsigned c) { return min(min(a, b), c); }
__device__ __forceinline__ unsigned min4(unsigned m, u32x4 v) { return min3u(min3u(m, v[0], v[1]), v[2], v[3]); }
__device__ __forceinline__ u32x4 canon(u32x4 v) {
#pragma unroll
  for (int d = 0; d < 4; ++d) v[d] = v[d] ? v[d] : 0x80808080u;
  return v;
}
__device__ __forceinline__ void st128(u32x4 v, __amdgpu_buffer_rsrc_t r, unsigned voff, unsigned soff) {
  __builtin_amdgcn_raw_buffer_store_b128(v, r, voff, soff, AUX_SYS);
  asm volatile("s_nop 1" :: "v"(v) : "memory");
}
typedef __attribute__((address_space(3))) int lds_int;
typedef __attribute__((address_space(1))) unsigned gu32;
__device__ __forceinline__ void lds_st(lds_int* a, int v) { __hip_atomic_store(a, v, __ATOMIC_RELAXED, __HIP_MEMORY_SCOPE_WORKGROUP); }
__device__ __forceinline__ int lds_ld(lds_int* a) { return __hip_atomic_load(a, __ATOMIC_RELAXED, __HIP_MEMORY_SCOPE_WORKGROUP); }
__device__ __forceinline__ unsigned src_off(int s) { return (unsigned)(s >> 6) * GROUP + (unsigned)(s & 63) * 64; }
__device__ __forceinline__ void tr16(const unsigned (&r)[16], u32x4 (&o)[4]) {
#pragma unroll
  for (int d = 0; d < 4; ++d) {
    const unsigned a0 = r[4 * d], a1 = r[4 * d + 1], a2 = r[4 * d + 2], a3 = r[4 * d + 3];
    const unsigned t0 = __builtin_amdgcn_perm(a1, a0, 0x05010400u), t1 = __builtin_amdgcn_perm(a1, a0, 0x07030602u);
    const unsigned t2 = __builtin_amdgcn_perm(a3, a2, 0x05010400u), t3 = __builtin_amdgcn_perm(a3, a2, 0x07030602u);
    o[0][d] = __builtin_amdgcn_perm(t2, t0, 0x05040100u);
    o[1][d] = __builtin_amdgcn_perm(t2, t0, 0x07060302u);
    o[2][d] = __builtin_amdgcn_perm(t3, t1, 0x05040100u);
    o[3][d] = __builtin_amdgcn_perm(t3, t1, 0x07060302u);
  }
}

__global__ void __launch_bounds__(NTHR) ag_gemm_kernel(Params p) {
  extern __shared__ __attribute__((aligned(16))) char smem[];
  const u32x4 dx = p.dx, dc = p.dc;
  char* myws = p.myws;
  const uint8_t* pw = p.w;
  unsigned rk = p.rk, wq = p.wq;
  __builtin_amdgcn_sched_barrier(0);
  const int wid = __builtin_amdgcn_readfirstlane(threadIdx.x >> 6);
  const int lane = threadIdx.x & 63;
  const int bid = blockIdx.x;
  const int cu = bid >> 3;
  unsigned icode = 0;
  int xcd = __builtin_amdgcn_s_getreg(20 | (3 << 11)) & 7;
  const unsigned tj = (unsigned)(cu - TOUCH);
  const int mate = (((0x21321321 >> (4 * wid)) & ((cu - 4) >> 31)) | ((1 + (int)(tj * 11 >> 5)) & ~((cu - 4) >> 31))) & 3;
  auto peer = [&]() -> int { return xcd < 4 ? mate : xcd; };
  const int li = lane & 15, lg = lane >> 4;
  unsigned vsrc = (unsigned)li * KD + (unsigned)lg * 16;
  const unsigned vlin = (unsigned)lane * 16;
  unsigned vw = (unsigned)lg * 16 * NLOC + (unsigned)li * 4;
  const int h = (wid + 5) >> 3;
  const int s5 = cu * 2 + (wid & 1) + (wid >> 1) * 8;
  static_assert(NIP == 16, "the first two pieces of a wave are 32 apart, in one group");
  auto sfirst = [&]() -> unsigned {
    const unsigned m = 0u - (unsigned)(xcd >> 2);
    return ((unsigned)(2 * (cu * 4 + xcd) + h) * 64 & ~m) | ((NI * GROUP + (unsigned)s5 * 64) & m);
  };
  unsigned so0 = sfirst();
  unsigned cnt = C_CNT + (unsigned)(bid * NWAVE + wid) * 4;
  unsigned so1 = so0 + GROUP;
  unsigned vw2 = vw + 8 * NLOC;
  asm volatile("" : "+s"(so0), "+s"(so1), "+s"(cnt), "+v"(vsrc), "+v"(vw), "+v"(vw2));
  __builtin_amdgcn_sched_barrier(0);
  u32x4 r0[4];
  unsigned craw;
  if (__builtin_expect(cu < 4, 1)) {
    r0[0] = raw_load128(dx, vsrc, so0, 0);
    __builtin_amdgcn_sched_barrier(0);
    craw = raw_load32(dc, 0, cnt, AUX_AGENT);
    __builtin_amdgcn_sched_barrier(0);
    r0[1] = raw_load128(dx, vsrc + 2048, so0, 0);
    __builtin_amdgcn_sched_barrier(0);
    r0[2] = raw_load128(dx, vsrc, so1, 0);
    __builtin_amdgcn_sched_barrier(0);
    r0[3] = raw_load128(dx, vsrc + 2048, so1, 0);
    __builtin_amdgcn_sched_barrier(0);
  }
  char* pws = p.ws[peer()];
  asm volatile("" :: "s"(rk));
  __builtin_amdgcn_sched_barrier(0);
  const u64 t0 = __builtin_amdgcn_s_memrealtime();
  __builtin_amdgcn_sched_barrier(0);
  const uint8_t* px = (const uint8_t*)((u64)dx[1] << 32 | dx[0]);
  const __amdgpu_buffer_rsrc_t rc = mk_rsrc(myws + CTRL_OFF, CTRL_BYTES);
  const __amdgpu_buffer_rsrc_t rxs = mk_rsrc(px, SHARD);
  const unsigned rank = rk & 7u;
  const __amdgpu_buffer_rsrc_t rw = mk_rsrc(pw, KD * NLOC);
  gu32* const roles = (gu32*)(myws + CTRL_OFF + C_ROLE) + cu * 32 + (lane & 7);
  lds_int* const verdict = (lds_int*)(smem + 16 * LSL);
  unsigned took = 0;
  const bool asker = wid == (cu < 4 ? 6 : 0);
  auto ask = [&]() { if (asker && lane < 8) took = __hip_atomic_fetch_add(roles, lane == xcd ? 1u : 0u, __ATOMIC_RELAXED, __HIP_MEMORY_SCOPE_AGENT); };
  auto resolve = [&]() -> int {
    if (!asker) return 0;
    int r = xcd;
    unsigned c = __builtin_amdgcn_readlane(took, r);
    unsigned behind = (unsigned)__builtin_amdgcn_ballot_w64((int)(took - c) < 0) & 255u;
    while (__builtin_expect(behind != 0, 0)) {
      const int rn = __builtin_ctz(behind);
      if (lane < 8) took = __hip_atomic_fetch_add(roles, lane == rn ? 1u : lane == r ? ~0u : 0u, __ATOMIC_RELAXED, __HIP_MEMORY_SCOPE_AGENT);
      r = rn;
      c = __builtin_amdgcn_readlane(took, r);
      behind = (unsigned)__builtin_amdgcn_ballot_w64((int)(took - c) < 0) & 255u;
    }
    lds_st(verdict + (lane & 1), lane & 1 ? r : (int)c + 1);
    return r;
  };
  auto meet = [&]() -> int {
    asm volatile("s_waitcnt lgkmcnt(0)" ::: "memory");
    __builtin_amdgcn_s_barrier();
    asm volatile("" ::: "memory");
    return lds_ld(verdict + (lane & 1));
  };
  if (__builtin_expect(cu < 4, 1)) {
    lds_int* const pos = (lds_int*)smem;
    lds_int* const done = pos + 192;
    auto dfirst = [&](unsigned so) -> unsigned { return (so & ~(GROUP - 1)) | (so & (GROUP - 1)) * 16; };
    unsigned do0 = dfirst(so0), vd0 = xcd < 4 && wid >= 6 ? SHARD : vlin;
    char* pdst = pws + (size_t)rank * SHARD;
    asm volatile("" : "+s"(do0), "+v"(vd0), "+s"(pdst));
    __builtin_amdgcn_sched_barrier(0);
    const unsigned call = __builtin_amdgcn_readfirstlane(craw) + 1;
    __amdgpu_buffer_rsrc_t rd = mk_rsrc(pdst + (size_t)(call % NSLOT) * SLOT, SHARD);
    st128(canon(r0[0]), rd, vd0, do0);
    STAMP(2);
    __builtin_amdgcn_sched_barrier(0);
    st128(canon(r0[1]), rd, vd0, do0 + 32768);
    __builtin_amdgcn_sched_barrier(0);
#pragma unroll 1
    for (bool again = false;; again = true) {
      if (__builtin_expect(xcd < 4, 1)) {
        const int pidx = cu * 4 + xcd;
        if (wid == 6) { lds_st(pos + lane, -1); lds_st(pos + lane + 64, -1); lds_st(pos + lane + 128, -1); lds_st(done + lane, lane >= 6 ? 0x7fffffff : 0); }
        asm volatile("s_waitcnt lgkmcnt(0)" ::: "memory");
        __builtin_amdgcn_s_barrier();
        __builtin_amdgcn_sched_barrier(0);
        if (__builtin_expect(wid < 6, 1)) {
          lds_st(done + wid, 2);
          asm volatile("s_waitcnt vmcnt(1)" ::: "memory");
          st128(canon(r0[2]), rd, vlin, (unsigned)(2 * (pidx + 2 * NIP) + h) * 1024);
          __builtin_amdgcn_sched_barrier(0);
          u32x4 r[2] = {r0[2], r0[3]};
          int up[2] = {pidx + 2 * NIP, pidx + 3 * NIP};
#pragma unroll 1
          for (int n0 = 2; up[0] < NUNIT; n0 += 2) {
#pragma unroll
            for (int i = 0; i < 2; ++i) {
              const int n = n0 + i;
              lds_st(done + wid, n + 1);
              int un = pidx + (n + 2) * NIP;
              if (n + 2 >= NSTAT) {
                for (;;) {
                  un = __builtin_amdgcn_readfirstlane(lds_ld(pos + n + 2));
                  if (un >= 0) break;
                  __builtin_amdgcn_s_sleep(1);
                }
              }
              r[i] = __builtin_amdgcn_raw_buffer_load_b128(rxs, vsrc, src_off(2 * un + h), 0);
              up[i] = un;
              __builtin_amdgcn_sched_barrier(0);
              const int u = up[i ^ 1];
              st128(canon(r[i ^ 1]), rd, u < NUNIT ? vlin : SHARD, (unsigned)(2 * u + h) * 1024);
              __builtin_amdgcn_sched_barrier(0);
            }
          }
          STAMP(5);
          STAMP_DRAIN();
        } else if (wid == 6) {
          int role = xcd;
          if (!again) { ask(); role = resolve(); }
          if (role == xcd) {
            const size_t fo = (size_t)(call % NSLOT) * SLOT + (size_t)((rk >> (16 + 4 * xcd)) & 7u) * SHARD + NNO * GROUP + ((unsigned)cu * 16 + (wq >> 16)) * 1024;
            const __amdgpu_buffer_rsrc_t rm = mk_rsrc(myws + fo, 16384);
            while (__builtin_amdgcn_s_memrealtime() - t0 < T_FL) __builtin_amdgcn_s_sleep(8);
            asm volatile("" ::: "memory");
            u32x4 f[2];
            unsigned vok = vlin, tl = 0;
#pragma unroll
            for (int i = 1; i < 4; ++i) tl |= __builtin_amdgcn_raw_buffer_load_b32(mk_rsrc(p.ws[i] + fo, 16384), 0, 0, AUX_SYS);
            for (unsigned k = 1;; ++k) {
#pragma unroll
              for (int i = 0; i < 2; ++i) f[i] = __builtin_amdgcn_raw_buffer_load_b128(rm, vlin, (unsigned)i * 8192, AUX_SYS);
              if (__builtin_amdgcn_ballot_w64(min4(min4(0xffffffffu, f[0]), f[1]) == 0u) == 0ull) break;
              if ((k & 63) == 0 && __builtin_amdgcn_s_memrealtime() - t0 > DEADLINE) { vok = 16384; break; }
              __builtin_amdgcn_s_sleep(16);
              asm volatile("" ::: "memory");
            }
            asm volatile("" :: "v"(tl));
            while (__builtin_amdgcn_readfirstlane(lds_ld(pos + lds_ld(done))) < NNO * 32 - FWM) __builtin_amdgcn_s_sleep(2);
#pragma unroll
            for (int i = 0; i < 6; ++i) {
              if (i == 3) asm volatile("s_waitcnt vmcnt(" STR(FWIN) ")" ::: "memory");
              st128(f[i / 3], mk_rsrc(p.ws[1 + i % 3] + fo, 16384), vok, (unsigned)(i / 3) * 8192);
            }
          }
        } else {
          if (pidx == 0) __builtin_amdgcn_raw_buffer_store_b32(0u, rc, (unsigned)(lane & 7) * 128, C_TICK + ((call + 1) & 1) * 1024, AUX_AGENT);
          gu32* tick = (gu32*)(myws + CTRL_OFF + C_TICK + (call & 1) * 1024);
          int n = NSTAT;
          bool mine = true;
          for (;;) {
            int d = lds_ld(done + (lane & 7));
            d = min(d, __builtin_amdgcn_update_dpp(0x7fffffff, d, 0x111, 0xf, 0xf, false));
            d = min(d, __builtin_amdgcn_update_dpp(0x7fffffff, d, 0x112, 0xf, 0xf, false));
            d = min(d, __builtin_amdgcn_update_dpp(0x7fffffff, d, 0x114, 0xf, 0xf, false));
            const int lim = min(__builtin_amdgcn_readlane(d, 7) + (mine ? AHEAD1 : AHEAD), NMAX);
            const int k = lim - n;
            if (k <= 0) {
              if (n >= NMAX) { lds_st(pos + n + (lane & 3), NUNIT); break; }
              __builtin_amdgcn_s_sleep(2);
              continue;
            }
            unsigned t = 0;
            if (lane == 0) t = __hip_atomic_fetch_add(tick + (mine ? xcd * 32 : 128), (unsigned)k, __ATOMIC_RELAXED, __HIP_MEMORY_SCOPE_AGENT);
            if (mine) {
              const int t0 = (int)min(__builtin_amdgcn_readfirstlane(t), (unsigned)NXT);
              const int kk = min(k, NXT - t0);
              if (lane < kk) lds_st(pos + n + lane, NSTAT * NIP + (t0 + lane) * 4 + xcd);
              n += kk;
              mine = t0 + k < NXT;
            } else {
              const int t0 = (int)min(__builtin_amdgcn_readfirstlane(t), (unsigned)NDYN);
              const bool last = t0 + k >= NDYN;
              if (lane < (last ? k + 4 : k)) lds_st(pos + n + lane, min(NUNIT - NDYN + t0 + lane, NUNIT));
              if (last) break;
              n += k;
            }
          }
        }
        STAMP(6);
      } else {
        constexpr int NPW = NX * 2;
        int s6 = s5;
        asm volatile("" : "+s"(s6));
        auto spos = [&](int n) -> int { const int g = n >> 1; return (g < PHI ? NI + g : NNO + g - PHI) * 64 + (n & 1) * 32 + s6; };
        u32x4 r[NPW];
        const unsigned vpl = (s6 & 4) == 0 && (unsigned)(s6 & 3) != ((wq >> (4 * (xcd - 4))) & 3u) ? SHARD : vlin;
        static_assert(PHI >= 2, "the first four pieces of a wave");
        r[2] = r0[2]; r[3] = r0[3];
#pragma unroll
        for (int n = 2; n < 2 + S2EARLY; ++n) {
          asm volatile("s_waitcnt vmcnt(1)" ::: "memory");
          st128(canon(r[n]), rd, vlin, (unsigned)spos(n) * 1024);
          __builtin_amdgcn_sched_barrier(0);
        }
        constexpr int N1 = 2 + S2EARLY;
        __builtin_amdgcn_s_barrier();
        __builtin_amdgcn_sched_barrier(0);
        if (!again) ask();
#pragma unroll
        for (int n = 4; n < NPW; ++n) r[n] = __builtin_amdgcn_raw_buffer_load_b128(rxs, vsrc, src_off(spos(n)), 0);
#pragma unroll
        for (int n = 2; n < NPW; ++n) r[n] = canon(r[n]);
        STAMP(1);
#pragma unroll
        for (int n = N1; n < NPW; ++n) {
          if (n == N1 || n == 2 * PHI) {
            while (__builtin_amdgcn_s_memrealtime() - t0 < (n == 2 * PHI ? T_P : T_S)) __builtin_amdgcn_s_sleep(2);
            asm volatile("" ::: "memory");
          }
          st128(r[n], rd, n >> 1 == PHI ? vpl : vlin, (unsigned)spos(n) * 1024);
          asm volatile("s_waitcnt vmcnt(" STR(SWIN) ")" ::: "memory");
        }
        STAMP_DRAIN();
        STAMP(5);
        if (!again) resolve();
      }
      if (again) break;
      const int role = __builtin_amdgcn_readlane(meet(), 1);
      if (__builtin_expect(role == xcd, 1)) break;
      xcd = role;
      rd = mk_rsrc(p.ws[peer()] + (size_t)(call % NSLOT) * SLOT + (size_t)rank * SHARD, SHARD);
#pragma unroll
      for (int i = 0; i < 4; ++i) r0[i] = __builtin_amdgcn_raw_buffer_load_b128(rxs, vsrc + (unsigned)(i & 1) * 2048, sfirst() + (unsigned)(i >> 1) * GROUP, 0);
#pragma unroll
      for (int i = 0; i < 2; ++i) st128(canon(r0[i]), rd, xcd < 4 && wid >= 6 ? SHARD : vlin, dfirst(sfirst()) + (unsigned)i * 32768);
    }
    asm volatile("" :: "v"(icode));
    __builtin_amdgcn_raw_buffer_store_b32(call, rc, 0, cnt, AUX_AGENT);
    if (xcd >= 4 && cu * 8 + wid < 28) {
      const int c = cu * 8 + wid;
      const unsigned rg = (unsigned)(p.last[xcd - 4][c >> 4] >> (((c >> 1) & 7) * 8)) & 255u;
      const __amdgpu_buffer_rsrc_t rt = mk_rsrc(p.out[0], NR * MLOC * NLOC * 4);
      const unsigned so = rg * 16 * (NLOC * 4) + ((wq >> (4 * (xcd - 4))) & 3u) * 512 + (unsigned)(c & 1) * 256;
      const unsigned vt = (unsigned)lg * (NLOC * 4) + (unsigned)li * 16;
#pragma unroll
      for (int q = 0; q < 4; ++q) st128(u32x4{~0u, ~0u, ~0u, ~0u}, rt, vt, so + (unsigned)q * 4 * (NLOC * 4));
      asm volatile("s_waitcnt vmcnt(0)" ::: "memory");
      __builtin_amdgcn_raw_buffer_store_b32(call, mk_rsrc(p.ws[xcd] + CTRL_OFF, CTRL_BYTES), 0, C_FLAG + rank * 256 + (unsigned)c * 4, AUX_SYS);
      while (__builtin_amdgcn_s_memrealtime() - t0 < T_POLL) __builtin_amdgcn_s_sleep(8);
      asm volatile("" ::: "memory");
      for (unsigned n = 1;; ++n) {
        u32x4 t[4];
#pragma unroll
        for (int q = 0; q < 4; ++q) t[q] = __builtin_amdgcn_raw_buffer_load_b128(rt, vt, so + (unsigned)q * 4 * (NLOC * 4), AUX_SYS);
        unsigned m = 0;
#pragma unroll
        for (int q = 0; q < 4; ++q) m = max(max(max(m, t[q][0]), t[q][1]), max(t[q][2], t[q][3]));
        if (__builtin_amdgcn_ballot_w64(m == ~0u) == 0ull) break;
        if ((n & 63) == 0 && __builtin_amdgcn_s_memrealtime() - t0 > DEADLINE) {
          __builtin_amdgcn_raw_buffer_store_b32(call, rc, 0, 0, AUX_AGENT);
          break;
        }
        __builtin_amdgcn_s_sleep(PSLEEP);
        asm volatile("" ::: "memory");
      }
    }
    STAMP_FLUSH();
    return;
  }

  if (tj < (xcd < 4 ? 9u : 3u))
    icode = __builtin_amdgcn_raw_buffer_load_b32(mk_rsrc(pws + (size_t)rank * SHARD + (size_t)(tj - (tj * 11 >> 5) * 3) * SLOT, 64), 0, 0, AUX_SYS);
  if (xcd >= 4 && tj - 3u < 2u) icode = __builtin_amdgcn_raw_buffer_load_b32(rw, 0, (tj - 3u) * (KD * NLOC - 64), 0);
  __builtin_amdgcn_sched_barrier(0);
  ask();
  __builtin_amdgcn_sched_barrier(0);
  unsigned warm = 0;
  unsigned raw0[2][16];
  auto first_loads = [&](bool first) {
    if (xcd < 4) {
      const int wi = wid * 28 + cu - 4;
      if (wi < NI * 4)
        warm = __builtin_amdgcn_raw_buffer_load_b32(rxs, (unsigned)(lane >> 4) * KD + (unsigned)((lane >> 1) & 7) * 512 + (unsigned)(lane & 1) * 64,
                                                    (unsigned)wi * 4 * KD + (unsigned)xcd * 128, 0);
      if (first) resolve();
    } else if (cu < 12) {
      const unsigned so = (unsigned)((cu - 4) * 8 + wid) * 64 * NLOC + ((wq >> (4 * (xcd - 4))) & 3u) * 128;
#pragma unroll
      for (int hh = 0; hh < 2; ++hh)
#pragma unroll
        for (int i = 0; i < 16; ++i) raw0[hh][i] = __builtin_amdgcn_raw_buffer_load_b32(rw, (i < 8 ? vw : vw2) + (unsigned)((i & 7) * NLOC + hh * 64), so, 0);
      __builtin_amdgcn_sched_barrier(0);
      if (first) resolve();
    } else {
      if (cu < 28)
        warm = __builtin_amdgcn_raw_buffer_load_b32(rw, (unsigned)(lane >> 1) * NLOC + (unsigned)(lane & 1) * 64,
                                                    (unsigned)(((cu - 12) & 7) * 16 + wid * 2 + ((cu - 12) >> 3)) * 32 * NLOC + ((wq >> (4 * (xcd - 4))) & 3u) * 128, 0);
      if (first) resolve();
    }
    __builtin_amdgcn_sched_barrier(0);
  };
  first_loads(true);
  const int vd = meet();
  const unsigned call = __builtin_amdgcn_readlane(vd, 0);
  warm |= icode;
  if (__builtin_expect(__builtin_amdgcn_readlane(vd, 1) != xcd, 0)) {
    xcd = __builtin_amdgcn_readlane(vd, 1);
    pws = p.ws[peer()];
    first_loads(false);
  }
  const bool cross = xcd >= 4;
  if (cross && cu < 12) {
    const __amdgpu_buffer_rsrc_t rd = mk_rsrc(pws + (size_t)(call % NSLOT) * SLOT + (size_t)rank * SHARD, SHARD);
    const int kb = (cu - 4) * 8 + wid;
    u32x4 o[2][4];
    tr16(raw0[0], o[0]);
    tr16(raw0[1], o[1]);
#pragma unroll
    for (int e = 0; e < 8; ++e) {
      if (e > WWIN) asm volatile("s_waitcnt vmcnt(" STR(WWIN) ")" ::: "memory");
      st128(canon(o[e & 1][e >> 1]), rd, vlin + (unsigned)(e >> 1) * 1024, (unsigned)((e & 1) * 64 + kb) * 4096);
      __builtin_amdgcn_sched_barrier(0);
      if (e == 0) STAMP(2);
    }
    STAMP(7);
    __builtin_amdgcn_s_barrier();
  }
  __builtin_amdgcn_sched_barrier(0);

  const int c28 = cu - 4;
  const int ci = xcd * 28 + c28;
  const int strm = cross ? c28 >> 1 : ci >> 3;
  const int slab = cross ? c28 & 1 : ci & 7;
  const unsigned rot = cross ? (unsigned)slab * 4 : 0u;
  u64 tl = p.task[cross][strm][0], th = p.task[cross][strm][1];
  typedef const __attribute__((address_space(4))) float cf32;
  float sx = *(cf32*)p.sx, sw = *(cf32*)p.sw;
  float* po = p.out[cross ? xcd : 0];
  bool dead = false;
  auto expired = [&]() -> bool {
    return __builtin_amdgcn_readfirstlane(__builtin_amdgcn_raw_buffer_load_b32(rc, 0, 0, AUX_AGENT)) == call ||
           __builtin_amdgcn_s_memrealtime() - t0 > DEADLINE;
  };
  auto next = [&]() -> int {
    const int c = (int)(tl & 255u);
    tl = (tl >> 8) | (th << 56); th = (th >> 8) | (63ull << 56);
    if (c == 63) return -1;
    int role = c >> 5;
    if (cross && role >= 4) role = 4 + ((role + xcd) & 3);
    return (int)(((rk >> (4 * role)) & 7u) << 8) | c;
  };
  u32x4 xv[8];
  auto issue_own = [&](int code) {
    const __amdgpu_buffer_rsrc_t r = rxs;
    const unsigned so = (unsigned)(code & 31) * GROUP + (unsigned)wid * 512;
#pragma unroll
    for (int q = 0; q < 8; ++q) xv[q] = __builtin_amdgcn_raw_buffer_load_b128(r, vsrc + (unsigned)(q & 3) * 64, so + ((unsigned)(q & 4) ^ rot) * 64, 0);
  };
  int code = next();
  u32x4 wf[8][4];
  if (!cross) {
    const unsigned vo = vw + (unsigned)wid * 512 * NLOC + (unsigned)slab * 64;
    unsigned raw[3][16];
#pragma unroll
    for (int u = 0; u < 10; ++u) {
      if (u < 8) {
#pragma unroll
        for (int i = 0; i < 16; ++i) raw[u % 3][i] = __builtin_amdgcn_raw_buffer_load_b32(rw, vo, (unsigned)(u * 64 + i) * NLOC, 0);
      }
      if (u == 8) issue_own(code);
      __builtin_amdgcn_sched_barrier(0);
      if (u >= 2) tr16(raw[(u - 2) % 3], wf[u - 2]);
      __builtin_amdgcn_sched_barrier(0);
    }
    asm volatile("" : "+s"(sx), "+s"(sw), "+s"(po) : "v"(warm));
  } else {
    const __amdgpu_buffer_rsrc_t rq = mk_rsrc(myws + (size_t)(call % NSLOT) * SLOT + (size_t)((rk >> (4 * xcd)) & 7u) * SHARD +
                                              (size_t)(slab * 64 + wid * 8) * 4096, 32768);
    const unsigned vq = (unsigned)(lane & 7) * 4096 + (unsigned)((lane >> 3) & 3) * 1024 + 1020;
    unsigned got = 0;
    asm volatile("" : "+s"(sx), "+s"(sw), "+s"(po) : "v"(warm));
    while (__builtin_amdgcn_s_memrealtime() - t0 < T_W0) __builtin_amdgcn_s_sleep(8);
    asm volatile("" ::: "memory");
    issue_own(code);
    if (cu == 31 && wid == 0)
      warm = __builtin_amdgcn_raw_buffer_load_b32(mk_rsrc(po, NR * MLOC * NLOC * 4 - 3 * 512), lane < 8 ? (unsigned)lane << 20 : 0x80000000u, 0, AUX_SYS);
#pragma unroll 1
    for (unsigned n = 1; got != 15 && !dead; ++n) {
      __builtin_amdgcn_s_sleep(WSLEEP);
      const unsigned hh = __builtin_amdgcn_raw_buffer_load_b32(rq, vq, 0, AUX_SYS);
      const unsigned miss = (unsigned)__builtin_amdgcn_ballot_w64(hh == 0u);
#pragma unroll
      for (int e = 0; e < 4; ++e) {
        if (!((got >> e) & 1) && !((miss >> (8 * e)) & 255u)) {
          got |= 1u << e;
#pragma unroll
          for (int q = 0; q < 8; ++q) wf[q][e] = __builtin_amdgcn_raw_buffer_load_b128(rq, vlin + (unsigned)e * 1024, ((unsigned)q ^ rot) * 4096, RAUX);
        }
      }
      if ((n & 63) == 0 && expired()) dead = true;
      asm volatile("" ::: "memory");
    }
    asm volatile("" :: "v"(warm));
    STAMP(3);
    for (unsigned n = 1;; ++n) {
      unsigned m = 0xffffffffu;
#pragma unroll
      for (int q = 0; q < 8; ++q)
#pragma unroll
        for (int e = 0; e < 4; ++e) m = min4(m, wf[q][e]);
      if (__builtin_expect(__builtin_amdgcn_ballot_w64(m == 0u) == 0ull || dead, 1)) break;
      if ((n & 63) == 0 && expired()) dead = true;
#pragma unroll
      for (int q = 0; q < 8; ++q)
#pragma unroll
        for (int e = 0; e < 4; ++e) wf[q][e] = __builtin_amdgcn_raw_buffer_load_b128(rq, vlin + (unsigned)e * 1024, ((unsigned)q ^ rot) * 4096, RAUX);
      asm volatile("" ::: "memory");
    }
  }
  STAMP(1);
  float scale = dead ? __builtin_nanf("") : sx * sw;
  const __amdgpu_buffer_rsrc_t rs = mk_rsrc(myws + (size_t)(call % NSLOT) * SLOT, SLOT);
  const __amdgpu_buffer_rsrc_t rz = mk_rsrc(myws + (size_t)((call + 2) % NSLOT) * SLOT, SLOT);
  const __amdgpu_buffer_rsrc_t ro = mk_rsrc(po, NR * MLOC * NLOC * 4 - (cross ? 3 * 512 : 0));
  const unsigned vh = (unsigned)(lane & 7) * 1024 + 1020;
  const int orow = 2 * wid + (lane >> 5);
  const unsigned vout = (unsigned)orow * (NLOC * 4) + (unsigned)slab * 256 + (unsigned)(lane & 31) * 8;
  typedef __attribute__((address_space(3))) char lds_char;
  lds_char* const lw = (lds_char*)smem + wid * LSL + li * LROW + lg * 64;
  const lds_char* const lr = (lds_char*)smem + orow * LROW + (lane & 31) * 8;
  const unsigned vz = cross ? 0x80000000u : vlin;
  const char* const xslot = myws + (size_t)(call % NSLOT) * SLOT;
  auto when = [&](int code) -> int {
    const int g = code & 31;
    int a = (g < NNO ? 213 : 269) + 113 * g, b = g == NI ? 400 : g > NI ? T_SA : g == NNO ? T_F : T_P + 300;
    asm volatile("" : "+s"(a), "+s"(b));
    a = code & 0x80 ? b : a;
    a = code & 0xe0 ? a : 0;
    return code < 0 ? 0x7fffffff : a;
  };
  __builtin_amdgcn_sched_barrier(0);
  int buf = 0;
  bool again = false;
  const unsigned fo = C_FLAG + ((rk >> (4 * xcd)) & 7u) * 256 + (unsigned)c28 * 4;
  int slot = cross ? TP0 * 8 + ((strm < 4 ? strm + 10 : strm - 4) * 2 + slab) * PD : -0x10000000;
  const int period = 28 * PD + ((unsigned)(strm - 4) < 3u ? ADJ0 : (unsigned)(strm - 4) < 7u ? ADJ1 : ADJ2);
  unsigned marked = call - 1;
  __builtin_amdgcn_raw_buffer_store_b32(0u, ro, 0x80000000u, 0, AUX_SYS);
  __builtin_amdgcn_sched_barrier(0);
#pragma unroll 1
  do {
    const int ncode = next();
    const bool nown = (ncode & 0xe0) == 0;
    unsigned nstep = nown ? 64 : 1024;
    unsigned nso = (unsigned)(ncode & 31) * GROUP + (unsigned)wid * 8 * nstep;
    unsigned nvo = nown ? vsrc : vlin;
    unsigned nlo = nso + rot * nstep, nhi = nso + (rot ^ 4) * nstep;
    asm volatile("" : "+s"(nlo), "+s"(nhi));
    const char* nbase = nown ? (const char*)px : xslot + (size_t)((ncode >> 8) & 7) * SHARD;
    int nwhen = when(ncode);
    unsigned zo = (unsigned)(code >> 8) * SHARD + (unsigned)(code & 31) * GROUP, zw = ((rk >> (16 + ((code >> 1) & 12))) & 7u) * SHARD + (unsigned)(code & 7) * GROUP;
    asm volatile("" : "+s"(zo), "+s"(zw));
    zo = (code & 0xe0 ? zo : zw) + (unsigned)(slab * 8 + wid) * 1024;
    unsigned oo = ((unsigned)(code >> 8) * MLOC + (unsigned)(code & 31) * 16) * (NLOC * 4);
    lds_char* lwb = lw + buf;
    const lds_char* lrb = lr + buf;
    int far = ncode < 0 ? 0 : ncode & 0xe0;
    int old = cross ? (ncode & 0x9f) - GLATE : 0;
    asm volatile("" : "+s"(old));
    far = (unsigned)old >= 32u - GLATE ? -far : far;
    unsigned vf = cross && ncode >= 0 && (tl & 255u) == 63 ? 0u : 0x80000000u;
    asm volatile("" : "+s"(nstep), "+s"(nso), "+s"(nbase), "+s"(nwhen), "+s"(zo), "+s"(oo), "+s"(far), "+s"(tl), "+s"(th));
    asm volatile("" : "+v"(nvo), "+v"(lwb), "+v"(lrb), "+v"(vf));
    __builtin_amdgcn_sched_barrier(0);
    asm volatile("" :: "v"(xv[0]), "v"(xv[1]), "v"(xv[2]), "v"(xv[3]), "v"(xv[4]), "v"(xv[5]), "v"(xv[6]), "v"(xv[7]));
    __builtin_amdgcn_sched_barrier(0);
    unsigned mk = __builtin_amdgcn_readfirstlane(marked);
    asm volatile("" : "+s"(mk));
    __builtin_amdgcn_sched_barrier(0);
    if (__builtin_expect(again, 0)) {
      unsigned m = 0xffffffffu, mx = 0;
#pragma unroll
      for (int q = 0; q < 8; ++q) { m = min4(m, xv[q]); mx |= xv[q][3]; }
      if (__builtin_expect(__builtin_amdgcn_ballot_w64(m == 0u) != 0ull && !dead, 0)) {
        const int s = (code & 31) * 64 + wid * 8;
        const unsigned so = (unsigned)(code >> 8) * SHARD + (unsigned)s * 1024;
        const unsigned ho = (code & 0x80) == 0 && s >= EARLY0 ? so - EARLY * 8192 : so;
        bool poll = __builtin_amdgcn_ballot_w64(mx != 0u) == 0ull;
        for (unsigned n = 1;; ++n) {
          if (poll) {
            const unsigned hh = __builtin_amdgcn_raw_buffer_load_b32(rs, vh, ho, AUX_SYS);
            poll = __builtin_amdgcn_ballot_w64(hh == 0u) != 0ull;
            if (poll) __builtin_amdgcn_s_sleep(HSLEEP);
          }
          if (!poll) {
#pragma unroll
            for (int q = 0; q < 8; ++q)
                xv[q] = __builtin_amdgcn_raw_buffer_load_b128(rs, vlin + (unsigned)(q & 3) * 1024, so + ((unsigned)(q & 4) ^ rot) * 1024, RAUX);
            m = 0xffffffffu;
#pragma unroll
            for (int q = 0; q < 8; ++q) m = min4(m, xv[q]);
            if (__builtin_amdgcn_ballot_w64(m == 0u) == 0ull) break;
            if (n > 64) __builtin_amdgcn_s_sleep(16);
          }
          if ((n & 63) == 0 && expired()) { dead = true; break; }
          asm volatile("" ::: "memory");
        }
        if (dead) { __builtin_amdgcn_raw_buffer_store_b32(call, rc, 0, 0, AUX_AGENT); scale = __builtin_nanf(""); }
      }
    }
    unsigned now = (unsigned)__builtin_amdgcn_s_memrealtime();
    __builtin_amdgcn_sched_barrier(0);
    __amdgpu_buffer_rsrc_t rn;
    f32x4 acc[4];
#pragma unroll
    for (int e = 0; e < 4; ++e) acc[e] = f32x4{0.f, 0.f, 0.f, 0.f};
    auto fetch = [&](int q) { xv[q] = __builtin_amdgcn_raw_buffer_load_b128(rn, nvo, (q < 4 ? nlo : nhi) + (unsigned)(q & 3) * nstep, FAUX); };
    auto part = [&](int e) {
#pragma unroll
      for (int r = 0; r < 4; ++r) *(__attribute__((address_space(3))) float*)(lwb + r * 16 + e * 4) = acc[e][r];
    };
#pragma unroll
    for (int ks = 0; ks < 4; ++ks) {
      const i32x8 b = {(int)xv[2 * ks][0], (int)xv[2 * ks][1], (int)xv[2 * ks][2], (int)xv[2 * ks][3],
                       (int)xv[2 * ks + 1][0], (int)xv[2 * ks + 1][1], (int)xv[2 * ks + 1][2], (int)xv[2 * ks + 1][3]};
#pragma unroll
      for (int e = 0; e < 4; ++e) {
        const u32x4 a0 = wf[2 * ks][e], a1 = wf[2 * ks + 1][e];
        const i32x8 a = {(int)a0[0], (int)a0[1], (int)a0[2], (int)a0[3], (int)a1[0], (int)a1[1], (int)a1[2], (int)a1[3]};
        acc[e] = __builtin_amdgcn_mfma_scale_f32_16x16x128_f8f6f4(a, b, acc[e],   1,   0, 0, 0, 0, 0);
        __builtin_amdgcn_sched_barrier(0);
        if (ks == 0 && e == 0) st128(u32x4{0u, 0u, 0u, 0u}, rz, vz, zo);
        if (ks == 0 && e == 1) marked = __builtin_amdgcn_raw_buffer_load_b32(rc, vf, fo, AUX_SYS);
        if (ks == 0 && e == 3) {
          asm volatile("" : "+s"(now));
          rn = mk_rsrc(nbase, (int)(now - (unsigned)t0) >= nwhen ? SHARD : 0);
        }
        if (ks > 0 && e < 2) fetch(2 * ks - 2 + e);
        if (ks == 3 && e >= 2) part(e - 2);
        __builtin_amdgcn_sched_barrier(0);
      }
    }
    part(2);
    __builtin_amdgcn_sched_barrier(0);
    part(3);
    __builtin_amdgcn_sched_barrier(0);
    fetch(6);
    fetch(7);
    __builtin_amdgcn_sched_barrier(0);
    asm volatile("s_waitcnt lgkmcnt(0)" ::: "memory");
    __builtin_amdgcn_s_barrier();
    asm volatile("" ::: "memory");
    f32x2 pr[8], ov;
#pragma unroll
    for (int v = 0; v < 8; ++v) pr[v] = *(const __attribute__((address_space(3))) f32x2*)(lrb + v * LSL);
    const f32x2 y = (((pr[0] + pr[1]) + (pr[2] + pr[3])) + ((pr[4] + pr[5]) + (pr[6] + pr[7]))) * scale;
#pragma unroll
    for (int r = 0; r < 2; ++r) ov[r] = y[r] * __builtin_amdgcn_rcpf(1.f + __expf(-__builtin_amdgcn_fmed3f(y[r], -60.f, 60.f)));
    u32x2 ob = __builtin_bit_cast(u32x2, ov);
#pragma unroll
    for (int r = 0; r < 2; ++r) ob[r] = min(ob[r], 0xfffffffeu);
    asm volatile("" : "+v"(ob));
    __builtin_amdgcn_sched_barrier(0);
    auto pace = [&]() {
      if ((int)(now - (unsigned)t0) * 8 < slot) {
        while ((int)((unsigned)__builtin_amdgcn_s_memrealtime() - (unsigned)t0) * 8 < slot) __builtin_amdgcn_s_sleep(1);
        asm volatile("" ::: "memory");
      }
    };
    again = far < 0;
    if (far > 0) {
      unsigned m = 0xffffffffu;
#pragma unroll
      for (int q = 0; q < 8; ++q) m = min4(m, xv[q]);
      again = __builtin_amdgcn_ballot_w64(m == 0u) != 0ull;
      const unsigned so = (unsigned)(ncode >> 8) * SHARD + nso;
      auto refetch = [&]() {
#pragma unroll
        for (int q = 0; q < 8; ++q) xv[q] = __builtin_amdgcn_raw_buffer_load_b128(rs, vlin + (unsigned)(q & 3) * 1024, so + ((unsigned)(q & 4) ^ rot) * 1024, RAUX);
      };
      if (again && (int)(__builtin_amdgcn_s_memrealtime() - t0) + LATEM >= nwhen) refetch();
      if (cross) __builtin_amdgcn_s_barrier();
    }
    if (cross && ncode < 0) {
      for (unsigned n = 1; __builtin_expect(mk != call, 0); ++n) {
        mk = __builtin_amdgcn_readfirstlane(__builtin_amdgcn_raw_buffer_load_b32(rc, 0, fo, AUX_SYS));
        if ((n & 63) == 0 && expired()) break;
        __builtin_amdgcn_s_sleep(2);
        asm volatile("" ::: "memory");
      }
      asm volatile("s_waitcnt vmcnt(0)" ::: "memory");
    }
    pace();
    slot += period;
    __builtin_amdgcn_sched_barrier(0);
    __builtin_amdgcn_raw_buffer_store_b64(ob, ro, vout, oo, AUX_SYS);
    __builtin_amdgcn_sched_barrier(0);
    code = ncode;
    buf = 8 * LSL - buf;
  } while (code >= 0);
  STAMP(5);
  STAMP_DRAIN();
  STAMP(6);
  STAMP_FLUSH();
}

#define DUR0 1.2f
#define DUR1 2.0f
#define BEG0 8.f
#define BEG2 13.9f
#define BEG1 12.f
void build_tasks(Params& p) {
  for (int t = 0; t < 2; ++t) {
    float when[168];
    uint8_t what[168];
    int n = 0;
    auto add = [&](float w, int role, int g) {
      int i = n++;
      for (; i > 0 && when[i - 1] > w; --i) { when[i] = when[i - 1]; what[i] = what[i - 1]; }
      when[i] = w; what[i] = (uint8_t)(role << 5 | g);
    };
    for (int g = 0; g < (t ? NNO : 32); ++g) add(0.f, 0, g);
    for (int g = 0; g < (t ? NNO : NI); ++g) for (int r = 1; r < 4; ++r) add((g < NNO ? 1.f : 1.56f) + 1.13f * (g + 1), r, g);
    for (int i = 0; i < (t ? PHI : NX); ++i)
      for (int r = t ? 5 : 4; r < 8; ++r)
        add(i == 0 ? 3.f : i < PHI ? T_SA * 0.01f : i == PHI ? T_F * 0.01f : T_P * 0.01f + 1.5f * (i - PHI + 1), r, i < PHI ? NI + i : NNO + i - PHI);
    float busy[14];
    int len[14];
    uint8_t lst[14][16];
    for (int j = 0; j < 14; ++j) { busy[j] = t ? (j < 4 ? BEG2 : BEG1) : BEG0; len[j] = 0; }
    for (int i = 0; i < n; ++i) {
      int b = -1;
      for (int j = 0; j < 14; ++j) if (len[j] < 15 && (b < 0 || busy[j] < busy[b])) b = j;
      busy[b] = (busy[b] > when[i] ? busy[b] : when[i]) + (t ? DUR1 : DUR0);
      lst[b][len[b]++] = what[i];
    }
    if (t) {
      static_assert((NNO * 4 + PHI * 3) % 14 == 7, "lists 4..10 are the long ones");
      uint8_t e[7];
      for (int j = 0; j < 7; ++j) e[j] = lst[4 + j][len[4 + j] - 1];
      for (int j = 0; j < 7; ++j) lst[4 + j][len[4 + j] - 1] = e[(j + 4) % 7];
    }
    for (int j = 0; j < 14; ++j) {
      u64 v[2] = {0, 0};
      const int o = 0;
      for (int i = 0; i < 16; ++i) v[i >> 3] |= (u64)(i >= o && i - o < len[j] ? lst[j][i - o] : 63) << (8 * (i & 7));
      p.task[t][j][0] = v[0]; p.task[t][j][1] = v[1];
    }
  }
}

}

extern "C" void kernel_launch(int rank, int nranks, ncclComm_t comm,
                              hipStream_t stream,
                              void* const* d_in, const int* in_sizes, int n_in,
                              void* d_out, int out_size,
                              const int* in_dtypes, int out_dtype,
                              void* const* d_out_all,
                              void* const* d_ws_all, size_t ws_bytes) {
  (void)comm; (void)in_dtypes; (void)out_dtype;
  if (nranks != NR || (unsigned)rank >= (unsigned)NR || n_in < 4 || in_sizes[0] != MLOC * KD || in_sizes[1] != KD * NLOC ||
      out_size != NR * MLOC * NLOC || ws_bytes < CTRL_OFF + CTRL_BYTES) {
    fprintf(stderr, "kernel_launch: unexpected problem shape\n");
    return;
  }
  static bool attr_done[NR];
  if (!attr_done[rank]) {
    HIP_CHECK(hipFuncSetAttribute((const void*)ag_gemm_kernel, hipFuncAttributeMaxDynamicSharedMemorySize, LDS_BYTES));
    attr_done[rank] = true;
  }
  Params p;
  auto desc = [](const void* a, unsigned bytes) { return u32x4{(unsigned)(uintptr_t)a, (unsigned)((uintptr_t)a >> 32) & 0xffffu, bytes, 0x00020000u}; };
  p.dx = desc(d_in[0], SHARD);
  p.dc = desc((char*)d_ws_all[rank] + CTRL_OFF, CTRL_BYTES);
  p.w = (const uint8_t*)d_in[1];
  p.sx = (const float*)d_in[2];
  p.sw = (const float*)d_in[3];
  p.myws = (char*)d_ws_all[rank];
  const int quad = 0x4d;
  auto qidx = [&](int r) { int n = 0; for (int q = 0; q < r; ++q) n += ((quad >> q) & 1) == ((quad >> r) & 1); return n; };
  auto roles = [&](int r, int (&role)[NR]) {
    int na = 1, nb = 4;
    role[0] = r;
    for (int j = 1; j < NR; ++j) {
      const int q = (r + j) & 7;
      if (((quad >> q) & 1) == ((quad >> r) & 1)) role[na++] = q; else role[nb++] = q;
    }
  };
  int role[NR];
  roles(rank, role);
  p.rk = 0; p.wq = 0;
  for (int i = 0; i < NR; ++i) {
    p.ws[i] = (char*)d_ws_all[role[i]];
    p.out[i] = i == 0 ? (float*)d_out : (float*)d_out_all[role[i]] + (i >= 4 ? qidx(rank) * 128 : 0);
    p.rk |= (unsigned)role[i] << (4 * i);
    if (i >= 4) p.wq |= (unsigned)qidx(role[i]) << (4 * (i - 4));
  }
  p.wq |= (unsigned)qidx(rank) << 16;
  build_tasks(p);
  for (int i = 0; i < 4; ++i) {
    int their[NR], me = 4;
    roles(role[4 + i], their);
    while (their[me] != rank) ++me;
    p.last[i][0] = p.last[i][1] = 0;
    for (int j = 0; j < 14; ++j) {
      int c = 63;
      for (int n = 15; c == 63; --n) c = (int)(p.task[1][j][n >> 3] >> (8 * (n & 7))) & 255;
      const int r = c >> 5 < 4 ? c >> 5 : 4 + (((c >> 5) + me) & 3);
      p.last[i][j >> 3] |= (u64)(their[r] * 32 + (c & 31)) << (8 * (j & 7));
    }
  }
  hipLaunchKernelGGL(ag_gemm_kernel, dim3(NBLK), dim3(NTHR), LDS_BYTES, stream, p);
  HIP_CHECK(hipGetLastError());
}
```
